```python
import numpy as np

kernel_name = "gpu_a2a_gemm_m4096_k4096_n32768_bf16_relu"


ABSMAX_RTOL = 0.008

COMPUTE_DTYPE = "bfloat16"
EPILOGUE = "relu"
IN_QUANT = None
OUT_QUANT = None
IN_DTYPES = {"x": "bfloat16", "w_mat": "bfloat16"}
WORLD_SIZE = 8

def _to_bf16(x):
    u = np.ascontiguousarray(x, dtype=np.float32).view(np.uint32)
    return ((u + 0x7FFF + ((u >> 16) & 1)) & 0xFFFF0000).view(np.float32)

def setup_inputs():

    x = _to_bf16(np.random.randn(4096, 4096).astype(np.float32))
    w_mat = _to_bf16((np.random.randn(4096, 32768) / np.sqrt(4096)).astype(np.float32))
    return {"x": x, "w_mat": w_mat}

def _epilogue(y):
    return np.maximum(y, 0.0).astype(np.float32)

def reference(x, w_mat):
    y = (x @ w_mat).astype(np.float32)
    return _epilogue(y)
```

```cpp
#include <hip/hip_runtime.h>
#include <rccl/rccl.h>
#include <cstdio>

#define CFG_STAMP 0
#define CFG_EPI 2
#define CFG_NOCOMM 0
#define CFG_AEPI 1

typedef float f32x16 __attribute__((ext_vector_type(16)));
typedef float f32x2 __attribute__((ext_vector_type(2)));
typedef __bf16 bf16x2 __attribute__((ext_vector_type(2)));
typedef short s16x2 __attribute__((ext_vector_type(2)));
typedef unsigned u32x2 __attribute__((ext_vector_type(2)));
typedef unsigned u32x4 __attribute__((ext_vector_type(4)));
typedef unsigned u32x8 __attribute__((ext_vector_type(8)));
typedef unsigned u32x16 __attribute__((ext_vector_type(16)));
typedef unsigned short u16;

constexpr int NOUT = 32768;
constexpr unsigned WS_CNT = 4096;
constexpr unsigned WS_TMO = 2048;
constexpr unsigned WS_ANCH = 3072;
constexpr unsigned WS_FLAG = 8192;
constexpr unsigned WS_DATA = 1u << 20;
constexpr unsigned WS_SINK = 16u << 20;
constexpr size_t WS_NEED = WS_SINK + (4u << 20);

struct WsTab { char* p[8]; };

__device__ __forceinline__ unsigned pk_relu(float lo, float hi) {
  f32x2 v = {lo, hi}; bf16x2 b = __builtin_convertvector(v, bf16x2);
  s16x2 s = __builtin_bit_cast(s16x2, b); s16x2 z = {0, 0};
  s = __builtin_elementwise_max(s, z);
  return __builtin_bit_cast(unsigned, s);
}

__global__ __launch_bounds__(256, 1) void a2a_gemm(const u16* __restrict__ X, const u16* __restrict__ W, u16* __restrict__ out, WsTab ws, int rank) {
  __shared__ __attribute__((aligned(1024))) char lds[5 * 32768];
  const unsigned b = blockIdx.x, xcd = b & 7, idx = b >> 3, pm = idx & 1, pn = xcd * 16 + (idx >> 1);
  const unsigned tid = threadIdx.x, l = tid & 63;
  const unsigned w = __builtin_amdgcn_readfirstlane(tid >> 6), wm = w >> 1, wn = w & 1;
  const unsigned ldsb = (unsigned)(size_t)lds;
  u32x16 va; u32x8 vb, vd; u32x4 vc; u32x2 ve;
  {
    unsigned r = l >> 2, c = (l & 3) ^ (((r >> 3) & 1) << 1);
    for (int i = 0; i < 4; i++) va[i] = ((w * 4 + i) * 16 + r) * 1024 + c * 16 - 1024 * i;
    for (int kb = 0; kb < 4; kb++) {
      unsigned row = 8 * kb + (l >> 3), sl = l & 7, seg = (sl >> 1) ^ (((l >> 4) & 1) | ((kb & 1) << 1));
      va[4 + kb] = row * 65536 + (64 * w + 16 * seg + 8 * (sl & 1)) * 2 - 1024 * kb;
    }
    unsigned rr = l & 15, g = l >> 4;
    unsigned abase = ldsb + wm * 8192 + rr * 64 + ((g ^ (((rr >> 3) & 1) << 1)) * 16);
    va[8] = abase; va[9] = abase + 65536;
    va[10] = w * 4096 + l * 16;
    unsigned U = (l - 4) & 31; if (U > 27) U = 0;
    va[11] = WS_FLAG + pm * 512 + U * 16;
    unsigned q = (l >> 2) & 3, p = l & 3, swz = (q >> 1) | ((g & 1) << 1);
    for (int s = 0; s < 4; s++) {
      unsigned v = ldsb + 16384 + wn * 8192 + (8 * g + q) * 128 + ((s ^ swz) * 32) + p * 8;
      va[12 + s] = v; vb[s] = v + 65536;
    }
    vb[4] = 0; vb[5] = 0; vb[6] = WS_CNT + b * 16; vb[7] = 0;
    for (int i = 0; i < 4; i++) vc[i] = ((w * 4 + i) * 8 + (l >> 3)) * 1024 + (((l & 7) ^ (4 * (i & 1) + (l >> 4))) * 16) - 1024 * i;
    for (int par = 0; par < 2; par++) {
      unsigned v = ldsb + rr * 128 + (((par * 4 + g) ^ (rr >> 1)) * 16);
      vd[par] = v + wm * 32768; vd[2 + par] = v + wm * 32768 + 65536; vd[4 + par] = v + wm * 16384 + 131072;
    }
    vd[6] = (pm * 256 + wm * 128 + rr) * (NOUT * 2) + (pn * 256 + wn * 128 + (g & 1) * 16 + (g >> 1) * 8) * 2; vd[7] = 0;
    ve[0] = (pm * 256 + wm * 128 + (rr & 7)) * (NOUT * 2) + (pn * 256 + wn * 128 + (rr >> 3) * 32 + (g & 1) * 16 + (g >> 1) * 8) * 2;
    ve[1] = ve[0] + 8 * (NOUT * 2);
  }
  const unsigned pj = xcd, ppm = idx & 1, q = idx >> 1;
  const unsigned dest = ((unsigned)rank - pj) & 7;
  const bool pusher = pj != 0 && !CFG_NOCOMM;
  const unsigned ipush = pusher ? q >> 1 : 0xffffu;
  const unsigned iflag = (pusher && w == 0) ? (q >> 1) + (q >= 2 ? 1u : 0u) : 0xffffu;
  char* myws = ws.p[0]; char* dws = ws.p[pj];
  u32x16 sa, sb, sc, sd;
  {
    size_t xb = (size_t)X, wb = (size_t)W + (size_t)pn * 512;
    sa[0] = (unsigned)xb; sa[1] = (unsigned)(xb >> 32) & 0xffff; sa[2] = 4u << 20; sa[3] = 0x20000;
    sa[4] = (unsigned)wb; sa[5] = (unsigned)(wb >> 32) & 0xffff; sa[6] = (256u << 20) - pn * 512; sa[7] = 0x20000;
    sa[8] = 0; sa[9] = (unsigned)rank * (32u << 20); sa[10] = ldsb + w * 4096;
    sa[11] = 0; sa[12] = 16 * 16384; sa[13] = 0; sa[14] = 0; sa[15] = 0;
    size_t ab = (size_t)myws + WS_DATA + (size_t)pm * (2u << 20);
    sb[0] = (unsigned)ab; sb[1] = (unsigned)(ab >> 32) & 0xffff; sb[2] = 2u << 20; sb[3] = 0x20000;
    unsigned rs = 7 * q + (pj ? pj - 1 : 0);
    size_t db = (size_t)dws + WS_DATA + (size_t)ppm * (2u << 20) + (size_t)(16 + rs) * 16384;
    sb[4] = (unsigned)db; sb[5] = (unsigned)(db >> 32);
    size_t kb = (size_t)myws + WS_SINK + (size_t)b * 16384;
    sb[6] = (unsigned)kb; sb[7] = (unsigned)(kb >> 32);
    size_t fb = (size_t)dws + WS_FLAG + ppm * 512 + rs * 4;
    sb[8] = (unsigned)fb; sb[9] = (unsigned)(fb >> 32);
    sb[10] = ipush; sb[11] = 3; sb[12] = 0;
    const bool poller = (w == 0) && !CFG_NOCOMM;
    sb[13] = poller ? 0xfu : 0xffffffffu; sb[14] = poller ? 0xfffffff0u : 0u; sb[15] = 0;
    sc[0] = 4; sc[1] = __builtin_amdgcn_readfirstlane(iflag); sc[2] = (unsigned)(size_t)myws; sc[3] = (unsigned)((size_t)myws >> 32);
    sc[4] = (dest * 512 + ppm * 256) * 1024 + q * 64; sc[5] = 0;
    sc[6] = ((unsigned)rank * 512 + pm * 256) * 1024; sc[7] = 66u << 20; sc[8] = 32u << 20; sc[9] = 0;
    for (int i = 10; i < 16; i++) sc[i] = 0;
    for (int i = 0; i < 16; i++) sd[i] = 0;
    sd[9] = ((idx >> 1) & 3) * 0;
    sd[12] = __builtin_amdgcn_readfirstlane((b == 0 && w == 0) ? 1u : 0u);
    sd[10] = ((idx >> 1) & 3) * (0 * 16);
    sd[13] = __builtin_amdgcn_readfirstlane((w == 0) ? 0xffffffffu : 0u);
  }
  f32x16 c[16];
  u32x8 st; unsigned fail, prof; u32x2 so = {(unsigned)(size_t)out, (unsigned)((size_t)out >> 32)};
  asm volatile(
    "s_memrealtime s[28:29]\n"
    "global_load_dwordx3 v[180:182], v150, s[74:75] sc0 sc1\n"
    "s_add_u32 m0, s50, 16384\n"
    "s_nop 0\n"
    "buffer_load_dwordx4 v132, s[44:47], s49 offen offset:0 nt lds\n"
    "buffer_load_dwordx4 v133, s[44:47], s49 offen offset:1024 nt lds\n"
    "buffer_load_dwordx4 v134, s[44:47], s49 offen offset:2048 nt lds\n"
    "buffer_load_dwordx4 v135, s[44:47], s49 offen offset:3072 nt lds\n"
    "s_add_u32 s49, s49, 0x200000\n"
    "s_and_b32 s49, s49, 0xfffffff\n"
    "s_add_u32 s48, s78, 0\n"
    "s_add_u32 m0, s50, 65536\n"
    "s_nop 0\n"
    "buffer_load_dwordx4 v154, s[40:43], s48 offen offset:0 lds\n"
    "buffer_load_dwordx4 v155, s[40:43], s48 offen offset:1024 lds\n"
    "buffer_load_dwordx4 v156, s[40:43], s48 offen offset:2048 lds\n"
    "buffer_load_dwordx4 v157, s[40:43], s48 offen offset:3072 lds\n"
    "s_add_u32 s48, s78, 131072\n"
    "s_add_u32 m0, s50, 98304\n"
    "s_nop 0\n"
    "buffer_load_dwordx4 v154, s[40:43], s48 offen offset:0 lds\n"
    "buffer_load_dwordx4 v155, s[40:43], s48 offen offset:1024 lds\n"
    "buffer_load_dwordx4 v156, s[40:43], s48 offen offset:2048 lds\n"
    "buffer_load_dwordx4 v157, s[40:43], s48 offen offset:3072 lds\n"
    "s_add_u32 m0, s50, 49152\n"
    "s_nop 0\n"
    "buffer_load_dwordx4 v132, s[44:47], s49 offen offset:0 nt lds\n"
    "buffer_load_dwordx4 v133, s[44:47], s49 offen offset:1024 nt lds\n"
    "buffer_load_dwordx4 v134, s[44:47], s49 offen offset:2048 nt lds\n"
    "buffer_load_dwordx4 v135, s[44:47], s49 offen offset:3072 nt lds\n"
    "s_add_u32 s49, s49, 0x200000\n"
    "s_and_b32 s49, s49, 0xfffffff\n"
    "s_cmp_lg_u32 s66, 0\n"
    "s_cbranch_scc1 NOPUSHLD_%=\n"
    "buffer_load_dwordx4 v[160:163], v128, s[40:43], s76 offen offset:0\n"
    "buffer_load_dwordx4 v[164:167], v129, s[40:43], s76 offen offset:1024\n"
    "buffer_load_dwordx4 v[168:171], v130, s[40:43], s76 offen offset:2048\n"
    "buffer_load_dwordx4 v[172:175], v131, s[40:43], s76 offen offset:3072\n"
    "NOPUSHLD_%=:\n"
    "s_add_u32 s48, s78, 128\n"
    "s_add_u32 m0, s50, 131072\n"
    "s_nop 0\n"
    "buffer_load_dwordx4 v154, s[40:43], s48 offen offset:0 lds\n"
    "buffer_load_dwordx4 v155, s[40:43], s48 offen offset:1024 lds\n"
    "buffer_load_dwordx4 v156, s[40:43], s48 offen offset:2048 lds\n"
    "buffer_load_dwordx4 v157, s[40:43], s48 offen offset:3072 lds\n"
    "s_add_u32 s48, s78, 131200\n"
    "s_add_u32 m0, s50, 147456\n"
    "s_nop 0\n"
    "buffer_load_dwordx4 v154, s[40:43], s48 offen offset:0 lds\n"
    "buffer_load_dwordx4 v155, s[40:43], s48 offen offset:1024 lds\n"
    "buffer_load_dwordx4 v156, s[40:43], s48 offen offset:2048 lds\n"
    "buffer_load_dwordx4 v157, s[40:43], s48 offen offset:3072 lds\n"
    "s_add_u32 m0, s50, 81920\n"
    "s_nop 0\n"
    "buffer_load_dwordx4 v132, s[44:47], s49 offen offset:0 nt lds\n"
    "buffer_load_dwordx4 v133, s[44:47], s49 offen offset:1024 nt lds\n"
    "buffer_load_dwordx4 v134, s[44:47], s49 offen offset:2048 nt lds\n"
    "buffer_load_dwordx4 v135, s[44:47], s49 offen offset:3072 nt lds\n"
    "s_add_u32 s49, s49, 0x200000\n"
    "s_and_b32 s49, s49, 0xfffffff\n"
    "s_waitcnt vmcnt(16)\n"
    "s_barrier\n"
    "ds_read_b64_tr_b16 v[64:65], v140 offset:0\n"
    "ds_read_b64_tr_b16 v[66:67], v140 offset:512\n"
    "ds_read_b128 v[0:3], v192 offset:0\n"
    "ds_read_b128 v[4:7], v192 offset:2048\n"
    "ds_read_b128 v[8:11], v192 offset:4096\n"
    "ds_read_b128 v[12:15], v192 offset:6144\n"
    "ds_read_b128 v[16:19], v192 offset:8192\n"
    "ds_read_b128 v[20:23], v192 offset:10240\n"
    "ds_read_b128 v[24:27], v192 offset:12288\n"
    "ds_read_b128 v[28:31], v192 offset:14336\n"
    "ds_read_b64_tr_b16 v[68:69], v141 offset:0\n"
    "ds_read_b64_tr_b16 v[70:71], v141 offset:512\n"
    "ds_read_b64_tr_b16 v[72:73], v142 offset:0\n"
    "ds_read_b64_tr_b16 v[74:75], v142 offset:512\n"
    "ds_read_b64_tr_b16 v[76:77], v143 offset:0\n"
    "ds_read_b64_tr_b16 v[78:79], v143 offset:512\n"
    "ds_read_b64_tr_b16 v[80:81], v140 offset:4096\n"
    "ds_read_b64_tr_b16 v[82:83], v140 offset:4608\n"
    "ds_read_b64_tr_b16 v[84:85], v141 offset:4096\n"
    "ds_read_b64_tr_b16 v[86:87], v141 offset:4608\n"
    "ds_read_b64_tr_b16 v[88:89], v142 offset:4096\n"
    "ds_read_b64_tr_b16 v[90:91], v142 offset:4608\n"
    "ds_read_b64_tr_b16 v[92:93], v143 offset:4096\n"
    "ds_read_b64_tr_b16 v[94:95], v143 offset:4608\n"
    "s_add_u32 m0, s50, 114688\n"
    "s_nop 0\n"
    "buffer_load_dwordx4 v132, s[44:47], s49 offen offset:0 nt lds\n"
    "buffer_load_dwordx4 v133, s[44:47], s49 offen offset:1024 nt lds\n"
    "buffer_load_dwordx4 v134, s[44:47], s49 offen offset:2048 nt lds\n"
    "buffer_load_dwordx4 v135, s[44:47], s49 offen offset:3072 nt lds\n"
    "s_add_u32 s49, s49, 0x200000\n"
    "s_and_b32 s49, s49, 0xfffffff\n"
    "s_add_u32 s48, s78, 256\n"
    "s_add_u32 m0, s50, 0\n"
    "s_nop 0\n"
    "buffer_load_dwordx4 v154, s[40:43], s48 offen offset:0 lds\n"
    "buffer_load_dwordx4 v155, s[40:43], s48 offen offset:1024 lds\n"
    "buffer_load_dwordx4 v156, s[40:43], s48 offen offset:2048 lds\n"
    "buffer_load_dwordx4 v157, s[40:43], s48 offen offset:3072 lds\n"
    "v_readfirstlane_b32 s32, v181\n"
    "v_readfirstlane_b32 s34, v182\n"
    "v_readfirstlane_b32 s68, v180\n"
    "s_nop 3\n"
    "s_add_u32 s68, s68, 1\n"
    "s_and_b32 s54, s68, 1\n"
    "s_lshl_b32 s54, s54, 22\n"
    "v_mov_b32 v152, s68\n"
    "s_add_u32 s56, s56, s54\n"
    "s_addc_u32 s57, s57, 0\n"
    "s_add_u32 s60, s60, s54\n"
    "s_addc_u32 s61, s61, 0\n"
    "s_cmp_eq_u32 s66, 0\n"
    "s_cselect_b64 exec, -1, 1\n"
    "s_cselect_b64 s[26:27], s[60:61], s[62:63]\n"
    "s_waitcnt vmcnt(20)\n"
    "global_store_dwordx4 v138, v[160:163], s[26:27] offset:0 sc0 sc1\n"
    "global_store_dwordx4 v138, v[164:167], s[26:27] offset:1024 sc0 sc1\n"
    "global_store_dwordx4 v138, v[168:171], s[26:27] offset:2048 sc0 sc1\n"
    "global_store_dwordx4 v138, v[172:175], s[26:27] offset:3072 sc0 sc1\n"
    "s_mov_b64 exec, -1\n"
    "s_waitcnt lgkmcnt(0)\n"
    "s_barrier\n"
    "s_sub_u32 s33, s28, s33\n"
    "s_add_u32 s33, s33, 200\n"
    "s_and_b32 s24, s37, 2720\n"
    "s_and_b32 s35, s37, 2128\n"
    "s_min_u32 s34, s34, 192\n"
    "s_and_b32 s34, s34, s37\n"
    "s_add_u32 s55, s34, 56\n"
    "s_min_u32 s55, s55, 192\n"
    "s_cmp_ge_u32 s34, 40\n"
    "s_cselect_b32 s55, s55, 0\n"
    "s_sub_u32 s54, s28, s32\n"
    "s_cmp_lt_u32 s54, 4000\n"
    "s_cselect_b32 s55, s34, s55\n"
    "s_cselect_b32 s81, 24, 40\n"
    "s_cselect_b32 s24, 2352, s24\n"
    "s_cselect_b32 s35, 2352, s35\n"
    "s_mov_b32 s25, 2128\n"
    "s_cselect_b32 s25, 2352, s25\n"
    "s_add_u32 s24, s24, s55\n"
    "s_add_u32 s35, s35, s55\n"
    "s_add_u32 s25, s25, s55\n"
    "s_and_b32 s24, s24, s37\n"
    "s_and_b32 s35, s35, s37\n"
    "s_and_b32 s25, s25, s37\n"
    "s_mov_b32 s32, s33\n"
    "s_mov_b32 s36, 0\n"
    "v_mfma_f32_16x16x32_bf16 a[0:3], v[64:67], v[0:3], 0\n"
    "ds_read_b64_tr_b16 v[96:97], v140 offset:32768\n"
    "v_mfma_f32_16x16x32_bf16 a[4:7], v[68:71], v[0:3], 0\n"
    "v_mfma_f32_16x16x32_bf16 a[32:35], v[64:67], v[4:7], 0\n"
    "ds_read_b64_tr_b16 v[98:99], v140 offset:33280\n"
    "s_add_u32 m0, s50, 16384\n"
    "buffer_load_dwordx4 v[160:163], v128, s[40:43], s76 offen offset:0\n"
    "v_mfma_f32_16x16x32_bf16 a[36:39], v[68:71], v[4:7], 0\n"
    "buffer_load_dwordx4 v132, s[44:47], s49 offen offset:0 nt lds\n"
    "v_mfma_f32_16x16x32_bf16 a[64:67], v[64:67], v[8:11], 0\n"
    "ds_read_b128 v[32:35], v193 offset:0\n"
    "v_mfma_f32_16x16x32_bf16 a[68:71], v[68:71], v[8:11], 0\n"
    "v_mfma_f32_16x16x32_bf16 a[96:99], v[64:67], v[12:15], 0\n"
    "ds_read_b128 v[36:39], v193 offset:2048\n"
    "v_mfma_f32_16x16x32_bf16 a[100:103], v[68:71], v[12:15], 0\n"
    "v_mfma_f32_16x16x32_bf16 a[128:131], v[64:67], v[16:19], 0\n"
    "ds_read_b128 v[40:43], v193 offset:4096\n"
    "v_mfma_f32_16x16x32_bf16 a[132:135], v[68:71], v[16:19], 0\n"
    "v_mfma_f32_16x16x32_bf16 a[160:163], v[64:67], v[20:23], 0\n"
    "ds_read_b128 v[44:47], v193 offset:6144\n"
    "v_mfma_f32_16x16x32_bf16 a[164:167], v[68:71], v[20:23], 0\n"
    "buffer_load_dwordx4 v133, s[44:47], s49 offen offset:1024 nt lds\n"
    "v_mfma_f32_16x16x32_bf16 a[192:195], v[64:67], v[24:27], 0\n"
    "ds_read_b128 v[48:51], v193 offset:8192\n"
    "v_mfma_f32_16x16x32_bf16 a[196:199], v[68:71], v[24:27], 0\n"
    "v_mfma_f32_16x16x32_bf16 a[224:227], v[64:67], v[28:31], 0\n"
    "ds_read_b128 v[52:55], v193 offset:10240\n"
    "v_mfma_f32_16x16x32_bf16 a[228:231], v[68:71], v[28:31], 0\n"
    "v_mfma_f32_16x16x32_bf16 a[232:235], v[72:75], v[28:31], 0\n"
    "ds_read_b128 v[56:59], v193 offset:12288\n"
    "v_mfma_f32_16x16x32_bf16 a[236:239], v[76:79], v[28:31], 0\n"
    "v_mfma_f32_16x16x32_bf16 a[200:203], v[72:75], v[24:27], 0\n"
    "ds_read_b128 v[60:63], v193 offset:14336\n"
    "v_mfma_f32_16x16x32_bf16 a[204:207], v[76:79], v[24:27], 0\n"
    "buffer_load_dwordx4 v134, s[44:47], s49 offen offset:2048 nt lds\n"
    "v_mfma_f32_16x16x32_bf16 a[168:171], v[72:75], v[20:23], 0\n"
    "ds_read_b64_tr_b16 v[100:101], v141 offset:32768\n"
    "v_mfma_f32_16x16x32_bf16 a[172:175], v[76:79], v[20:23], 0\n"
    "v_mfma_f32_16x16x32_bf16 a[136:139], v[72:75], v[16:19], 0\n"
    "ds_read_b64_tr_b16 v[102:103], v141 offset:33280\n"
    "v_mfma_f32_16x16x32_bf16 a[140:143], v[76:79], v[16:19], 0\n"
    "v_mfma_f32_16x16x32_bf16 a[104:107], v[72:75], v[12:15], 0\n"
    "ds_read_b64_tr_b16 v[104:105], v142 offset:32768\n"
    "v_mfma_f32_16x16x32_bf16 a[108:111], v[76:79], v[12:15], 0\n"
    "v_mfma_f32_16x16x32_bf16 a[72:75], v[72:75], v[8:11], 0\n"
    "ds_read_b64_tr_b16 v[106:107], v142 offset:33280\n"
    "v_mfma_f32_16x16x32_bf16 a[76:79], v[76:79], v[8:11], 0\n"
    "buffer_load_dwordx4 v135, s[44:47], s49 offen offset:3072 nt lds\n"
    "v_mfma_f32_16x16x32_bf16 a[40:43], v[72:75], v[4:7], 0\n"
    "ds_read_b64_tr_b16 v[108:109], v143 offset:32768\n"
    "v_mfma_f32_16x16x32_bf16 a[44:47], v[76:79], v[4:7], 0\n"
    "s_add_u32 s49, s49, 0x200000\n"
    "v_mfma_f32_16x16x32_bf16 a[8:11], v[72:75], v[0:3], 0\n"
    "ds_read_b64_tr_b16 v[110:111], v143 offset:33280\n"
    "v_mfma_f32_16x16x32_bf16 a[12:15], v[76:79], v[0:3], 0\n"
    "s_and_b32 s49, s49, 0xfffffff\n"
    "v_mfma_f32_16x16x32_bf16 a[16:19], v[80:83], v[0:3], 0\n"
    "ds_read_b64_tr_b16 v[112:113], v140 offset:36864\n"
    "v_mfma_f32_16x16x32_bf16 a[20:23], v[84:87], v[0:3], 0\n"
    "s_add_u32 s48, s78, 131328\n"
    "v_mfma_f32_16x16x32_bf16 a[48:51], v[80:83], v[4:7], 0\n"
    "ds_read_b64_tr_b16 v[114:115], v140 offset:37376\n"
    "s_add_u32 m0, s50, 32768\n"
    "v_mfma_f32_16x16x32_bf16 a[52:55], v[84:87], v[4:7], 0\n"
    "buffer_load_dwordx4 v154, s[40:43], s48 offen offset:0 lds\n"
    "v_mfma_f32_16x16x32_bf16 a[80:83], v[80:83], v[8:11], 0\n"
    "ds_read_b64_tr_b16 v[116:117], v141 offset:36864\n"
    "v_mfma_f32_16x16x32_bf16 a[84:87], v[84:87], v[8:11], 0\n"
    "v_mfma_f32_16x16x32_bf16 a[112:115], v[80:83], v[12:15], 0\n"
    "ds_read_b64_tr_b16 v[118:119], v141 offset:37376\n"
    "v_mfma_f32_16x16x32_bf16 a[116:119], v[84:87], v[12:15], 0\n"
    "v_mfma_f32_16x16x32_bf16 a[144:147], v[80:83], v[16:19], 0\n"
    "ds_read_b64_tr_b16 v[120:121], v142 offset:36864\n"
    "v_mfma_f32_16x16x32_bf16 a[148:151], v[84:87], v[16:19], 0\n"
    "v_mfma_f32_16x16x32_bf16 a[176:179], v[80:83], v[20:23], 0\n"
    "ds_read_b64_tr_b16 v[122:123], v142 offset:37376\n"
    "v_mfma_f32_16x16x32_bf16 a[180:183], v[84:87], v[20:23], 0\n"
    "buffer_load_dwordx4 v155, s[40:43], s48 offen offset:1024 lds\n"
    "v_mfma_f32_16x16x32_bf16 a[208:211], v[80:83], v[24:27], 0\n"
    "ds_read_b64_tr_b16 v[124:125], v143 offset:36864\n"
    "v_mfma_f32_16x16x32_bf16 a[212:215], v[84:87], v[24:27], 0\n"
    "v_mfma_f32_16x16x32_bf16 a[240:243], v[80:83], v[28:31], 0\n"
    "ds_read_b64_tr_b16 v[126:127], v143 offset:37376\n"
    "v_mfma_f32_16x16x32_bf16 a[244:247], v[84:87], v[28:31], 0\n"
    "v_mfma_f32_16x16x32_bf16 a[248:251], v[88:91], v[28:31], 0\n"
    "v_mfma_f32_16x16x32_bf16 a[252:255], v[92:95], v[28:31], 0\n"
    "v_mfma_f32_16x16x32_bf16 a[216:219], v[88:91], v[24:27], 0\n"
    "v_mfma_f32_16x16x32_bf16 a[220:223], v[92:95], v[24:27], 0\n"
    "buffer_load_dwordx4 v156, s[40:43], s48 offen offset:2048 lds\n"
    "v_mfma_f32_16x16x32_bf16 a[184:187], v[88:91], v[20:23], 0\n"
    "v_mfma_f32_16x16x32_bf16 a[188:191], v[92:95], v[20:23], 0\n"
    "v_mfma_f32_16x16x32_bf16 a[152:155], v[88:91], v[16:19], 0\n"
    "v_mfma_f32_16x16x32_bf16 a[156:159], v[92:95], v[16:19], 0\n"
    "v_mfma_f32_16x16x32_bf16 a[120:123], v[88:91], v[12:15], 0\n"
    "v_mfma_f32_16x16x32_bf16 a[124:127], v[92:95], v[12:15], 0\n"
    "v_mfma_f32_16x16x32_bf16 a[88:91], v[88:91], v[8:11], 0\n"
    "v_mfma_f32_16x16x32_bf16 a[92:95], v[92:95], v[8:11], 0\n"
    "buffer_load_dwordx4 v157, s[40:43], s48 offen offset:3072 lds\n"
    "v_mfma_f32_16x16x32_bf16 a[56:59], v[88:91], v[4:7], 0\n"
    "v_mfma_f32_16x16x32_bf16 a[60:63], v[92:95], v[4:7], 0\n"
    "v_mfma_f32_16x16x32_bf16 a[24:27], v[88:91], v[0:3], 0\n"
    "v_mfma_f32_16x16x32_bf16 a[28:31], v[92:95], v[0:3], 0\n"
    "s_waitcnt vmcnt(21) lgkmcnt(0)\n"
    "s_barrier\n"
    "v_mfma_f32_16x16x32_bf16 a[0:3], v[96:99], v[32:35], a[0:3]\n"
    "ds_read_b64_tr_b16 v[64:65], v144 offset:0\n"
    "v_mfma_f32_16x16x32_bf16 a[4:7], v[100:103], v[32:35], a[4:7]\n"
    "v_mfma_f32_16x16x32_bf16 a[32:35], v[96:99], v[36:39], a[32:35]\n"
    "ds_read_b64_tr_b16 v[66:67], v144 offset:512\n"
    "s_add_u32 m0, s50, 49152\n"
    "buffer_load_dwordx4 v[164:167], v129, s[40:43], s76 offen offset:1024\n"
    "v_mfma_f32_16x16x32_bf16 a[36:39], v[100:103], v[36:39], a[36:39]\n"
    "buffer_load_dwordx4 v132, s[44:47], s49 offen offset:0 nt lds\n"
    "v_mfma_f32_16x16x32_bf16 a[64:67], v[96:99], v[40:43], a[64:67]\n"
    "ds_read_b128 v[0:3], v194 offset:0\n"
    "v_mfma_f32_16x16x32_bf16 a[68:71], v[100:103], v[40:43], a[68:71]\n"
    "v_mfma_f32_16x16x32_bf16 a[96:99], v[96:99], v[44:47], a[96:99]\n"
    "ds_read_b128 v[4:7], v194 offset:2048\n"
    "v_mfma_f32_16x16x32_bf16 a[100:103], v[100:103], v[44:47], a[100:103]\n"
    "v_mfma_f32_16x16x32_bf16 a[128:131], v[96:99], v[48:51], a[128:131]\n"
    "ds_read_b128 v[8:11], v194 offset:4096\n"
    "v_mfma_f32_16x16x32_bf16 a[132:135], v[100:103], v[48:51], a[132:135]\n"
    "v_mfma_f32_16x16x32_bf16 a[160:163], v[96:99], v[52:55], a[160:163]\n"
    "ds_read_b128 v[12:15], v194 offset:6144\n"
    "v_mfma_f32_16x16x32_bf16 a[164:167], v[100:103], v[52:55], a[164:167]\n"
    "buffer_load_dwordx4 v133, s[44:47], s49 offen offset:1024 nt lds\n"
    "v_mfma_f32_16x16x32_bf16 a[192:195], v[96:99], v[56:59], a[192:195]\n"
    "ds_read_b128 v[16:19], v194 offset:8192\n"
    "v_mfma_f32_16x16x32_bf16 a[196:199], v[100:103], v[56:59], a[196:199]\n"
    "v_mfma_f32_16x16x32_bf16 a[224:227], v[96:99], v[60:63], a[224:227]\n"
    "ds_read_b128 v[20:23], v194 offset:10240\n"
    "v_mfma_f32_16x16x32_bf16 a[228:231], v[100:103], v[60:63], a[228:231]\n"
    "v_mfma_f32_16x16x32_bf16 a[232:235], v[104:107], v[60:63], a[232:235]\n"
    "ds_read_b128 v[24:27], v194 offset:12288\n"
    "v_mfma_f32_16x16x32_bf16 a[236:239], v[108:111], v[60:63], a[236:239]\n"
    "v_mfma_f32_16x16x32_bf16 a[200:203], v[104:107], v[56:59], a[200:203]\n"
    "ds_read_b128 v[28:31], v194 offset:14336\n"
    "v_mfma_f32_16x16x32_bf16 a[204:207], v[108:111], v[56:59], a[204:207]\n"
    "buffer_load_dwordx4 v134, s[44:47], s49 offen offset:2048 nt lds\n"
    "v_mfma_f32_16x16x32_bf16 a[168:171], v[104:107], v[52:55], a[168:171]\n"
    "ds_read_b64_tr_b16 v[68:69], v145 offset:0\n"
    "v_mfma_f32_16x16x32_bf16 a[172:175], v[108:111], v[52:55], a[172:175]\n"
    "v_mfma_f32_16x16x32_bf16 a[136:139], v[104:107], v[48:51], a[136:139]\n"
    "ds_read_b64_tr_b16 v[70:71], v145 offset:512\n"
    "v_mfma_f32_16x16x32_bf16 a[140:143], v[108:111], v[48:51], a[140:143]\n"
    "v_mfma_f32_16x16x32_bf16 a[104:107], v[104:107], v[44:47], a[104:107]\n"
    "ds_read_b64_tr_b16 v[72:73], v146 offset:0\n"
    "v_mfma_f32_16x16x32_bf16 a[108:111], v[108:111], v[44:47], a[108:111]\n"
    "v_mfma_f32_16x16x32_bf16 a[72:75], v[104:107], v[40:43], a[72:75]\n"
    "ds_read_b64_tr_b16 v[74:75], v146 offset:512\n"
    "v_mfma_f32_16x16x32_bf16 a[76:79], v[108:111], v[40:43], a[76:79]\n"
    "buffer_load_dwordx4 v135, s[44:47], s49 offen offset:3072 nt lds\n"
    "v_mfma_f32_16x16x32_bf16 a[40:43], v[104:107], v[36:39], a[40:43]\n"
    "ds_read_b64_tr_b16 v[76:77], v147 offset:0\n"
    "v_mfma_f32_16x16x32_bf16 a[44:47], v[108:111], v[36:39], a[44:47]\n"
    "s_add_u32 s49, s49, 0x200000\n"
    "v_mfma_f32_16x16x32_bf16 a[8:11], v[104:107], v[32:35], a[8:11]\n"
    "ds_read_b64_tr_b16 v[78:79], v147 offset:512\n"
    "v_mfma_f32_16x16x32_bf16 a[12:15], v[108:111], v[32:35], a[12:15]\n"
    "s_and_b32 s49, s49, 0xfffffff\n"
    "v_mfma_f32_16x16x32_bf16 a[16:19], v[112:115], v[32:35], a[16:19]\n"
    "ds_read_b64_tr_b16 v[80:81], v144 offset:4096\n"
    "v_mfma_f32_16x16x32_bf16 a[20:23], v[116:119], v[32:35], a[20:23]\n"
    "s_add_u32 s48, s78, 384\n"
    "v_mfma_f32_16x16x32_bf16 a[48:51], v[112:115], v[36:39], a[48:51]\n"
    "ds_read_b64_tr_b16 v[82:83], v144 offset:4608\n"
    "s_add_u32 m0, s50, 65536\n"
    "v_mfma_f32_16x16x32_bf16 a[52:55], v[116:119], v[36:39], a[52:55]\n"
    "buffer_load_dwordx4 v154, s[40:43], s48 offen offset:0 lds\n"
    "v_mfma_f32_16x16x32_bf16 a[80:83], v[112:115], v[40:43], a[80:83]\n"
    "ds_read_b64_tr_b16 v[84:85], v145 offset:4096\n"
    "v_mfma_f32_16x16x32_bf16 a[84:87], v[116:119], v[40:43], a[84:87]\n"
    "v_mfma_f32_16x16x32_bf16 a[112:115], v[112:115], v[44:47], a[112:115]\n"
    "ds_read_b64_tr_b16 v[86:87], v145 offset:4608\n"
    "v_mfma_f32_16x16x32_bf16 a[116:119], v[116:119], v[44:47], a[116:119]\n"
    "v_mfma_f32_16x16x32_bf16 a[144:147], v[112:115], v[48:51], a[144:147]\n"
    "ds_read_b64_tr_b16 v[88:89], v146 offset:4096\n"
    "v_mfma_f32_16x16x32_bf16 a[148:151], v[116:119], v[48:51], a[148:151]\n"
    "v_mfma_f32_16x16x32_bf16 a[176:179], v[112:115], v[52:55], a[176:179]\n"
    "ds_read_b64_tr_b16 v[90:91], v146 offset:4608\n"
    "v_mfma_f32_16x16x32_bf16 a[180:183], v[116:119], v[52:55], a[180:183]\n"
    "buffer_load_dwordx4 v155, s[40:43], s48 offen offset:1024 lds\n"
    "v_mfma_f32_16x16x32_bf16 a[208:211], v[112:115], v[56:59], a[208:211]\n"
    "ds_read_b64_tr_b16 v[92:93], v147 offset:4096\n"
    "v_mfma_f32_16x16x32_bf16 a[212:215], v[116:119], v[56:59], a[212:215]\n"
    "v_mfma_f32_16x16x32_bf16 a[240:243], v[112:115], v[60:63], a[240:243]\n"
    "ds_read_b64_tr_b16 v[94:95], v147 offset:4608\n"
    "v_mfma_f32_16x16x32_bf16 a[244:247], v[116:119], v[60:63], a[244:247]\n"
    "v_mfma_f32_16x16x32_bf16 a[248:251], v[120:123], v[60:63], a[248:251]\n"
    "v_mfma_f32_16x16x32_bf16 a[252:255], v[124:127], v[60:63], a[252:255]\n"
    "v_mfma_f32_16x16x32_bf16 a[216:219], v[120:123], v[56:59], a[216:219]\n"
    "v_mfma_f32_16x16x32_bf16 a[220:223], v[124:127], v[56:59], a[220:223]\n"
    "buffer_load_dwordx4 v156, s[40:43], s48 offen offset:2048 lds\n"
    "v_mfma_f32_16x16x32_bf16 a[184:187], v[120:123], v[52:55], a[184:187]\n"
    "v_mfma_f32_16x16x32_bf16 a[188:191], v[124:127], v[52:55], a[188:191]\n"
    "v_mfma_f32_16x16x32_bf16 a[152:155], v[120:123], v[48:51], a[152:155]\n"
    "s_memrealtime s[28:29]\n"
    "v_mfma_f32_16x16x32_bf16 a[156:159], v[124:127], v[48:51], a[156:159]\n"
    "v_mfma_f32_16x16x32_bf16 a[120:123], v[120:123], v[44:47], a[120:123]\n"
    "v_mfma_f32_16x16x32_bf16 a[124:127], v[124:127], v[44:47], a[124:127]\n"
    "v_mfma_f32_16x16x32_bf16 a[88:91], v[120:123], v[40:43], a[88:91]\n"
    "v_mfma_f32_16x16x32_bf16 a[92:95], v[124:127], v[40:43], a[92:95]\n"
    "buffer_load_dwordx4 v157, s[40:43], s48 offen offset:3072 lds\n"
    "v_mfma_f32_16x16x32_bf16 a[56:59], v[120:123], v[36:39], a[56:59]\n"
    "v_mfma_f32_16x16x32_bf16 a[60:63], v[124:127], v[36:39], a[60:63]\n"
    "v_mfma_f32_16x16x32_bf16 a[24:27], v[120:123], v[32:35], a[24:27]\n"
    "v_mfma_f32_16x16x32_bf16 a[28:31], v[124:127], v[32:35], a[28:31]\n"
    "s_waitcnt vmcnt(26) lgkmcnt(0)\n"
    "s_barrier\n"
    "v_mfma_f32_16x16x32_bf16 a[0:3], v[64:67], v[0:3], a[0:3]\n"
    "ds_read_b64_tr_b16 v[96:97], v144 offset:32768\n"
    "v_mfma_f32_16x16x32_bf16 a[4:7], v[68:71], v[0:3], a[4:7]\n"
    "v_mfma_f32_16x16x32_bf16 a[32:35], v[64:67], v[4:7], a[32:35]\n"
    "ds_read_b64_tr_b16 v[98:99], v144 offset:33280\n"
    "s_add_u32 m0, s50, 81920\n"
    "buffer_load_dwordx4 v[168:171], v130, s[40:43], s76 offen offset:2048\n"
    "v_mfma_f32_16x16x32_bf16 a[36:39], v[68:71], v[4:7], a[36:39]\n"
    "buffer_load_dwordx4 v132, s[44:47], s49 offen offset:0 nt lds\n"
    "v_mfma_f32_16x16x32_bf16 a[64:67], v[64:67], v[8:11], a[64:67]\n"
    "ds_read_b128 v[32:35], v195 offset:0\n"
    "v_mfma_f32_16x16x32_bf16 a[68:71], v[68:71], v[8:11], a[68:71]\n"
    "s_lshr_b32 s54, s24, 4\n"
    "s_add_u32 s33, s33, s54\n"
    "v_mfma_f32_16x16x32_bf16 a[96:99], v[64:67], v[12:15], a[96:99]\n"
    "ds_read_b128 v[36:39], v195 offset:2048\n"
    "s_add_u32 s32, s32, s54\n"
    "v_mfma_f32_16x16x32_bf16 a[100:103], v[68:71], v[12:15], a[100:103]\n"
    "s_sub_u32 s54, s33, s28\n"
    "s_max_i32 s54, s54, -250\n"
    "v_mfma_f32_16x16x32_bf16 a[128:131], v[64:67], v[16:19], a[128:131]\n"
    "ds_read_b128 v[40:43], v195 offset:4096\n"
    "v_mfma_f32_16x16x32_bf16 a[132:135], v[68:71], v[16:19], a[132:135]\n"
    "s_add_u32 s33, s28, s54\n"
    "s_cmp_gt_i32 s54, 0\n"
    "v_mfma_f32_16x16x32_bf16 a[160:163], v[64:67], v[20:23], a[160:163]\n"
    "ds_read_b128 v[44:47], v195 offset:6144\n"
    "s_cbranch_scc1 PACEI02_%=\n"
    "PACEBI02_%=:\n"
    "v_mfma_f32_16x16x32_bf16 a[164:167], v[68:71], v[20:23], a[164:167]\n"
    "buffer_load_dwordx4 v133, s[44:47], s49 offen offset:1024 nt lds\n"
    "v_mfma_f32_16x16x32_bf16 a[192:195], v[64:67], v[24:27], a[192:195]\n"
    "ds_read_b128 v[48:51], v195 offset:8192\n"
    "v_mfma_f32_16x16x32_bf16 a[196:199], v[68:71], v[24:27], a[196:199]\n"
    "s_sub_u32 s24, s24, 16\n"
    "s_max_i32 s24, s24, s35\n"
    "v_mfma_f32_16x16x32_bf16 a[224:227], v[64:67], v[28:31], a[224:227]\n"
    "ds_read_b128 v[52:55], v195 offset:10240\n"
    "v_mfma_f32_16x16x32_bf16 a[228:231], v[68:71], v[28:31], a[228:231]\n"
    "v_mfma_f32_16x16x32_bf16 a[232:235], v[72:75], v[28:31], a[232:235]\n"
    "ds_read_b128 v[56:59], v195 offset:12288\n"
    "v_mfma_f32_16x16x32_bf16 a[236:239], v[76:79], v[28:31], a[236:239]\n"
    "v_mfma_f32_16x16x32_bf16 a[200:203], v[72:75], v[24:27], a[200:203]\n"
    "ds_read_b128 v[60:63], v195 offset:14336\n"
    "v_mfma_f32_16x16x32_bf16 a[204:207], v[76:79], v[24:27], a[204:207]\n"
    "buffer_load_dwordx4 v134, s[44:47], s49 offen offset:2048 nt lds\n"
    "v_mfma_f32_16x16x32_bf16 a[168:171], v[72:75], v[20:23], a[168:171]\n"
    "ds_read_b64_tr_b16 v[100:101], v145 offset:32768\n"
    "v_mfma_f32_16x16x32_bf16 a[172:175], v[76:79], v[20:23], a[172:175]\n"
    "v_mfma_f32_16x16x32_bf16 a[136:139], v[72:75], v[16:19], a[136:139]\n"
    "ds_read_b64_tr_b16 v[102:103], v145 offset:33280\n"
    "v_mfma_f32_16x16x32_bf16 a[140:143], v[76:79], v[16:19], a[140:143]\n"
    "v_mfma_f32_16x16x32_bf16 a[104:107], v[72:75], v[12:15], a[104:107]\n"
    "ds_read_b64_tr_b16 v[104:105], v146 offset:32768\n"
    "v_mfma_f32_16x16x32_bf16 a[108:111], v[76:79], v[12:15], a[108:111]\n"
    "v_mfma_f32_16x16x32_bf16 a[72:75], v[72:75], v[8:11], a[72:75]\n"
    "ds_read_b64_tr_b16 v[106:107], v146 offset:33280\n"
    "v_mfma_f32_16x16x32_bf16 a[76:79], v[76:79], v[8:11], a[76:79]\n"
    "buffer_load_dwordx4 v135, s[44:47], s49 offen offset:3072 nt lds\n"
    "v_mfma_f32_16x16x32_bf16 a[40:43], v[72:75], v[4:7], a[40:43]\n"
    "ds_read_b64_tr_b16 v[108:109], v147 offset:32768\n"
    "v_mfma_f32_16x16x32_bf16 a[44:47], v[76:79], v[4:7], a[44:47]\n"
    "s_add_u32 s49, s49, 0x200000\n"
    "v_mfma_f32_16x16x32_bf16 a[8:11], v[72:75], v[0:3], a[8:11]\n"
    "ds_read_b64_tr_b16 v[110:111], v147 offset:33280\n"
    "v_mfma_f32_16x16x32_bf16 a[12:15], v[76:79], v[0:3], a[12:15]\n"
    "s_and_b32 s49, s49, 0xfffffff\n"
    "v_mfma_f32_16x16x32_bf16 a[16:19], v[80:83], v[0:3], a[16:19]\n"
    "ds_read_b64_tr_b16 v[112:113], v144 offset:36864\n"
    "v_mfma_f32_16x16x32_bf16 a[20:23], v[84:87], v[0:3], a[20:23]\n"
    "s_add_u32 s48, s78, 131456\n"
    "v_mfma_f32_16x16x32_bf16 a[48:51], v[80:83], v[4:7], a[48:51]\n"
    "ds_read_b64_tr_b16 v[114:115], v144 offset:37376\n"
    "s_add_u32 m0, s50, 98304\n"
    "v_mfma_f32_16x16x32_bf16 a[52:55], v[84:87], v[4:7], a[52:55]\n"
    "buffer_load_dwordx4 v154, s[40:43], s48 offen offset:0 lds\n"
    "v_mfma_f32_16x16x32_bf16 a[80:83], v[80:83], v[8:11], a[80:83]\n"
    "ds_read_b64_tr_b16 v[116:117], v145 offset:36864\n"
    "v_mfma_f32_16x16x32_bf16 a[84:87], v[84:87], v[8:11], a[84:87]\n"
    "v_mfma_f32_16x16x32_bf16 a[112:115], v[80:83], v[12:15], a[112:115]\n"
    "ds_read_b64_tr_b16 v[118:119], v145 offset:37376\n"
    "v_mfma_f32_16x16x32_bf16 a[116:119], v[84:87], v[12:15], a[116:119]\n"
    "v_mfma_f32_16x16x32_bf16 a[144:147], v[80:83], v[16:19], a[144:147]\n"
    "ds_read_b64_tr_b16 v[120:121], v146 offset:36864\n"
    "v_mfma_f32_16x16x32_bf16 a[148:151], v[84:87], v[16:19], a[148:151]\n"
    "v_mfma_f32_16x16x32_bf16 a[176:179], v[80:83], v[20:23], a[176:179]\n"
    "ds_read_b64_tr_b16 v[122:123], v146 offset:37376\n"
    "v_mfma_f32_16x16x32_bf16 a[180:183], v[84:87], v[20:23], a[180:183]\n"
    "buffer_load_dwordx4 v155, s[40:43], s48 offen offset:1024 lds\n"
    "v_mfma_f32_16x16x32_bf16 a[208:211], v[80:83], v[24:27], a[208:211]\n"
    "ds_read_b64_tr_b16 v[124:125], v147 offset:36864\n"
    "v_mfma_f32_16x16x32_bf16 a[212:215], v[84:87], v[24:27], a[212:215]\n"
    "v_mfma_f32_16x16x32_bf16 a[240:243], v[80:83], v[28:31], a[240:243]\n"
    "ds_read_b64_tr_b16 v[126:127], v147 offset:37376\n"
    "v_mfma_f32_16x16x32_bf16 a[244:247], v[84:87], v[28:31], a[244:247]\n"
    "v_mfma_f32_16x16x32_bf16 a[248:251], v[88:91], v[28:31], a[248:251]\n"
    "v_mfma_f32_16x16x32_bf16 a[252:255], v[92:95], v[28:31], a[252:255]\n"
    "v_mfma_f32_16x16x32_bf16 a[216:219], v[88:91], v[24:27], a[216:219]\n"
    "v_mfma_f32_16x16x32_bf16 a[220:223], v[92:95], v[24:27], a[220:223]\n"
    "buffer_load_dwordx4 v156, s[40:43], s48 offen offset:2048 lds\n"
    "v_mfma_f32_16x16x32_bf16 a[184:187], v[88:91], v[20:23], a[184:187]\n"
    "v_mfma_f32_16x16x32_bf16 a[188:191], v[92:95], v[20:23], a[188:191]\n"
    "v_mfma_f32_16x16x32_bf16 a[152:155], v[88:91], v[16:19], a[152:155]\n"
    "v_mfma_f32_16x16x32_bf16 a[156:159], v[92:95], v[16:19], a[156:159]\n"
    "v_mfma_f32_16x16x32_bf16 a[120:123], v[88:91], v[12:15], a[120:123]\n"
    "v_mfma_f32_16x16x32_bf16 a[124:127], v[92:95], v[12:15], a[124:127]\n"
    "v_mfma_f32_16x16x32_bf16 a[88:91], v[88:91], v[8:11], a[88:91]\n"
    "v_mfma_f32_16x16x32_bf16 a[92:95], v[92:95], v[8:11], a[92:95]\n"
    "buffer_load_dwordx4 v157, s[40:43], s48 offen offset:3072 lds\n"
    "v_mfma_f32_16x16x32_bf16 a[56:59], v[88:91], v[4:7], a[56:59]\n"
    "v_mfma_f32_16x16x32_bf16 a[60:63], v[92:95], v[4:7], a[60:63]\n"
    "v_mfma_f32_16x16x32_bf16 a[24:27], v[88:91], v[0:3], a[24:27]\n"
    "v_mfma_f32_16x16x32_bf16 a[28:31], v[92:95], v[0:3], a[28:31]\n"
    "s_waitcnt vmcnt(18) lgkmcnt(0)\n"
    "s_barrier\n"
    "v_mfma_f32_16x16x32_bf16 a[0:3], v[96:99], v[32:35], a[0:3]\n"
    "ds_read_b64_tr_b16 v[64:65], v140 offset:0\n"
    "s_cmp_eq_u32 s73, 0\n"
    "s_cselect_b64 exec, 1, 0\n"
    "global_store_dword v148, v152, s[64:65] sc0 sc1\n"
    "s_mov_b64 exec, -1\n"
    "v_mfma_f32_16x16x32_bf16 a[4:7], v[100:103], v[32:35], a[4:7]\n"
    "v_mfma_f32_16x16x32_bf16 a[32:35], v[96:99], v[36:39], a[32:35]\n"
    "ds_read_b64_tr_b16 v[66:67], v140 offset:512\n"
    "s_add_u32 m0, s50, 114688\n"
    "buffer_load_dwordx4 v[172:175], v131, s[40:43], s76 offen offset:3072\n"
    "v_mfma_f32_16x16x32_bf16 a[36:39], v[100:103], v[36:39], a[36:39]\n"
    "buffer_load_dwordx4 v132, s[44:47], s49 offen offset:0 nt lds\n"
    "v_mfma_f32_16x16x32_bf16 a[64:67], v[96:99], v[40:43], a[64:67]\n"
    "ds_read_b128 v[0:3], v190 offset:0\n"
    "v_mfma_f32_16x16x32_bf16 a[68:71], v[100:103], v[40:43], a[68:71]\n"
    "v_mfma_f32_16x16x32_bf16 a[96:99], v[96:99], v[44:47], a[96:99]\n"
    "ds_read_b128 v[4:7], v190 offset:2048\n"
    "v_mfma_f32_16x16x32_bf16 a[100:103], v[100:103], v[44:47], a[100:103]\n"
    "v_mfma_f32_16x16x32_bf16 a[128:131], v[96:99], v[48:51], a[128:131]\n"
    "ds_read_b128 v[8:11], v190 offset:4096\n"
    "v_mfma_f32_16x16x32_bf16 a[132:135], v[100:103], v[48:51], a[132:135]\n"
    "v_mfma_f32_16x16x32_bf16 a[160:163], v[96:99], v[52:55], a[160:163]\n"
    "ds_read_b128 v[12:15], v190 offset:6144\n"
    "v_mfma_f32_16x16x32_bf16 a[164:167], v[100:103], v[52:55], a[164:167]\n"
    "buffer_load_dwordx4 v133, s[44:47], s49 offen offset:1024 nt lds\n"
    "v_mfma_f32_16x16x32_bf16 a[192:195], v[96:99], v[56:59], a[192:195]\n"
    "ds_read_b128 v[16:19], v190 offset:8192\n"
    "v_mfma_f32_16x16x32_bf16 a[196:199], v[100:103], v[56:59], a[196:199]\n"
    "v_mfma_f32_16x16x32_bf16 a[224:227], v[96:99], v[60:63], a[224:227]\n"
    "ds_read_b128 v[20:23], v190 offset:10240\n"
    "v_mfma_f32_16x16x32_bf16 a[228:231], v[100:103], v[60:63], a[228:231]\n"
    "v_mfma_f32_16x16x32_bf16 a[232:235], v[104:107], v[60:63], a[232:235]\n"
    "ds_read_b128 v[24:27], v190 offset:12288\n"
    "v_mfma_f32_16x16x32_bf16 a[236:239], v[108:111], v[60:63], a[236:239]\n"
    "v_mfma_f32_16x16x32_bf16 a[200:203], v[104:107], v[56:59], a[200:203]\n"
    "ds_read_b128 v[28:31], v190 offset:14336\n"
    "v_mfma_f32_16x16x32_bf16 a[204:207], v[108:111], v[56:59], a[204:207]\n"
    "buffer_load_dwordx4 v134, s[44:47], s49 offen offset:2048 nt lds\n"
    "v_mfma_f32_16x16x32_bf16 a[168:171], v[104:107], v[52:55], a[168:171]\n"
    "ds_read_b64_tr_b16 v[68:69], v141 offset:0\n"
    "v_mfma_f32_16x16x32_bf16 a[172:175], v[108:111], v[52:55], a[172:175]\n"
    "v_mfma_f32_16x16x32_bf16 a[136:139], v[104:107], v[48:51], a[136:139]\n"
    "ds_read_b64_tr_b16 v[70:71], v141 offset:512\n"
    "v_mfma_f32_16x16x32_bf16 a[140:143], v[108:111], v[48:51], a[140:143]\n"
    "v_mfma_f32_16x16x32_bf16 a[104:107], v[104:107], v[44:47], a[104:107]\n"
    "ds_read_b64_tr_b16 v[72:73], v142 offset:0\n"
    "v_mfma_f32_16x16x32_bf16 a[108:111], v[108:111], v[44:47], a[108:111]\n"
    "v_mfma_f32_16x16x32_bf16 a[72:75], v[104:107], v[40:43], a[72:75]\n"
    "ds_read_b64_tr_b16 v[74:75], v142 offset:512\n"
    "v_mfma_f32_16x16x32_bf16 a[76:79], v[108:111], v[40:43], a[76:79]\n"
    "buffer_load_dwordx4 v135, s[44:47], s49 offen offset:3072 nt lds\n"
    "v_mfma_f32_16x16x32_bf16 a[40:43], v[104:107], v[36:39], a[40:43]\n"
    "ds_read_b64_tr_b16 v[76:77], v143 offset:0\n"
    "v_mfma_f32_16x16x32_bf16 a[44:47], v[108:111], v[36:39], a[44:47]\n"
    "s_add_u32 s49, s49, 0x200000\n"
    "v_mfma_f32_16x16x32_bf16 a[8:11], v[104:107], v[32:35], a[8:11]\n"
    "ds_read_b64_tr_b16 v[78:79], v143 offset:512\n"
    "v_mfma_f32_16x16x32_bf16 a[12:15], v[108:111], v[32:35], a[12:15]\n"
    "s_and_b32 s49, s49, 0xfffffff\n"
    "v_mfma_f32_16x16x32_bf16 a[16:19], v[112:115], v[32:35], a[16:19]\n"
    "ds_read_b64_tr_b16 v[80:81], v140 offset:4096\n"
    "v_mfma_f32_16x16x32_bf16 a[20:23], v[116:119], v[32:35], a[20:23]\n"
    "s_add_u32 s48, s78, 512\n"
    "v_mfma_f32_16x16x32_bf16 a[48:51], v[112:115], v[36:39], a[48:51]\n"
    "ds_read_b64_tr_b16 v[82:83], v140 offset:4608\n"
    "s_add_u32 m0, s50, 131072\n"
    "v_mfma_f32_16x16x32_bf16 a[52:55], v[116:119], v[36:39], a[52:55]\n"
    "buffer_load_dwordx4 v154, s[40:43], s48 offen offset:0 lds\n"
    "v_mfma_f32_16x16x32_bf16 a[80:83], v[112:115], v[40:43], a[80:83]\n"
    "ds_read_b64_tr_b16 v[84:85], v141 offset:4096\n"
    "v_mfma_f32_16x16x32_bf16 a[84:87], v[116:119], v[40:43], a[84:87]\n"
    "v_mfma_f32_16x16x32_bf16 a[112:115], v[112:115], v[44:47], a[112:115]\n"
    "ds_read_b64_tr_b16 v[86:87], v141 offset:4608\n"
    "v_mfma_f32_16x16x32_bf16 a[116:119], v[116:119], v[44:47], a[116:119]\n"
    "v_mfma_f32_16x16x32_bf16 a[144:147], v[112:115], v[48:51], a[144:147]\n"
    "ds_read_b64_tr_b16 v[88:89], v142 offset:4096\n"
    "v_mfma_f32_16x16x32_bf16 a[148:151], v[116:119], v[48:51], a[148:151]\n"
    "v_mfma_f32_16x16x32_bf16 a[176:179], v[112:115], v[52:55], a[176:179]\n"
    "ds_read_b64_tr_b16 v[90:91], v142 offset:4608\n"
    "v_mfma_f32_16x16x32_bf16 a[180:183], v[116:119], v[52:55], a[180:183]\n"
    "buffer_load_dwordx4 v155, s[40:43], s48 offen offset:1024 lds\n"
    "v_mfma_f32_16x16x32_bf16 a[208:211], v[112:115], v[56:59], a[208:211]\n"
    "ds_read_b64_tr_b16 v[92:93], v143 offset:4096\n"
    "v_mfma_f32_16x16x32_bf16 a[212:215], v[116:119], v[56:59], a[212:215]\n"
    "v_mfma_f32_16x16x32_bf16 a[240:243], v[112:115], v[60:63], a[240:243]\n"
    "ds_read_b64_tr_b16 v[94:95], v143 offset:4608\n"
    "v_mfma_f32_16x16x32_bf16 a[244:247], v[116:119], v[60:63], a[244:247]\n"
    "v_mfma_f32_16x16x32_bf16 a[248:251], v[120:123], v[60:63], a[248:251]\n"
    "v_mfma_f32_16x16x32_bf16 a[252:255], v[124:127], v[60:63], a[252:255]\n"
    "v_mfma_f32_16x16x32_bf16 a[216:219], v[120:123], v[56:59], a[216:219]\n"
    "v_mfma_f32_16x16x32_bf16 a[220:223], v[124:127], v[56:59], a[220:223]\n"
    "buffer_load_dwordx4 v156, s[40:43], s48 offen offset:2048 lds\n"
    "v_mfma_f32_16x16x32_bf16 a[184:187], v[120:123], v[52:55], a[184:187]\n"
    "v_mfma_f32_16x16x32_bf16 a[188:191], v[124:127], v[52:55], a[188:191]\n"
    "v_mfma_f32_16x16x32_bf16 a[152:155], v[120:123], v[48:51], a[152:155]\n"
    "s_memrealtime s[28:29]\n"
    "v_mfma_f32_16x16x32_bf16 a[156:159], v[124:127], v[48:51], a[156:159]\n"
    "v_mfma_f32_16x16x32_bf16 a[120:123], v[120:123], v[44:47], a[120:123]\n"
    "v_mfma_f32_16x16x32_bf16 a[124:127], v[124:127], v[44:47], a[124:127]\n"
    "v_mfma_f32_16x16x32_bf16 a[88:91], v[120:123], v[40:43], a[88:91]\n"
    "v_mfma_f32_16x16x32_bf16 a[92:95], v[124:127], v[40:43], a[92:95]\n"
    "buffer_load_dwordx4 v157, s[40:43], s48 offen offset:3072 lds\n"
    "v_mfma_f32_16x16x32_bf16 a[56:59], v[120:123], v[36:39], a[56:59]\n"
    "v_mfma_f32_16x16x32_bf16 a[60:63], v[124:127], v[36:39], a[60:63]\n"
    "v_mfma_f32_16x16x32_bf16 a[24:27], v[120:123], v[32:35], a[24:27]\n"
    "v_mfma_f32_16x16x32_bf16 a[28:31], v[124:127], v[32:35], a[28:31]\n"
    "s_waitcnt vmcnt(18) lgkmcnt(0)\n"
    "s_barrier\n"
    "v_mfma_f32_16x16x32_bf16 a[0:3], v[64:67], v[0:3], a[0:3]\n"
    "ds_read_b64_tr_b16 v[96:97], v140 offset:32768\n"
    "v_mfma_f32_16x16x32_bf16 a[4:7], v[68:71], v[0:3], a[4:7]\n"
    "s_cmp_eq_u32 s66, 1\n"
    "s_cselect_b64 s[30:31], -1, 1\n"
    "s_cselect_b64 s[26:27], s[60:61], s[62:63]\n"
    "v_mfma_f32_16x16x32_bf16 a[32:35], v[64:67], v[4:7], a[32:35]\n"
    "ds_read_b64_tr_b16 v[98:99], v140 offset:33280\n"
    "s_add_u32 m0, s50, 16384\n"
    "s_mov_b64 exec, s[30:31]\n"
    "global_store_dwordx4 v138, v[160:163], s[26:27] offset:0 sc0 sc1\n"
    "s_mov_b64 exec, -1\n"
    "v_mfma_f32_16x16x32_bf16 a[36:39], v[68:71], v[4:7], a[36:39]\n"
    "buffer_load_dwordx4 v132, s[44:47], s49 offen offset:0 nt lds\n"
    "v_mfma_f32_16x16x32_bf16 a[64:67], v[64:67], v[8:11], a[64:67]\n"
    "ds_read_b128 v[32:35], v191 offset:0\n"
    "v_mfma_f32_16x16x32_bf16 a[68:71], v[68:71], v[8:11], a[68:71]\n"
    "s_lshr_b32 s54, s24, 4\n"
    "s_add_u32 s33, s33, s54\n"
    "v_mfma_f32_16x16x32_bf16 a[96:99], v[64:67], v[12:15], a[96:99]\n"
    "ds_read_b128 v[36:39], v191 offset:2048\n"
    "s_add_u32 s32, s32, s54\n"
    "v_mfma_f32_16x16x32_bf16 a[100:103], v[68:71], v[12:15], a[100:103]\n"
    "s_sub_u32 s54, s33, s28\n"
    "s_max_i32 s54, s54, -250\n"
    "v_mfma_f32_16x16x32_bf16 a[128:131], v[64:67], v[16:19], a[128:131]\n"
    "ds_read_b128 v[40:43], v191 offset:4096\n"
    "v_mfma_f32_16x16x32_bf16 a[132:135], v[68:71], v[16:19], a[132:135]\n"
    "s_add_u32 s33, s28, s54\n"
    "s_cmp_gt_i32 s54, 0\n"
    "v_mfma_f32_16x16x32_bf16 a[160:163], v[64:67], v[20:23], a[160:163]\n"
    "ds_read_b128 v[44:47], v191 offset:6144\n"
    "s_cbranch_scc1 PACEI10_%=\n"
    "PACEBI10_%=:\n"
    "v_mfma_f32_16x16x32_bf16 a[164:167], v[68:71], v[20:23], a[164:167]\n"
    "buffer_load_dwordx4 v133, s[44:47], s49 offen offset:1024 nt lds\n"
    "v_mfma_f32_16x16x32_bf16 a[192:195], v[64:67], v[24:27], a[192:195]\n"
    "ds_read_b128 v[48:51], v191 offset:8192\n"
    "v_mfma_f32_16x16x32_bf16 a[196:199], v[68:71], v[24:27], a[196:199]\n"
    "s_sub_u32 s24, s24, 16\n"
    "s_max_i32 s24, s24, s35\n"
    "v_mfma_f32_16x16x32_bf16 a[224:227], v[64:67], v[28:31], a[224:227]\n"
    "ds_read_b128 v[52:55], v191 offset:10240\n"
    "v_mfma_f32_16x16x32_bf16 a[228:231], v[68:71], v[28:31], a[228:231]\n"
    "v_mfma_f32_16x16x32_bf16 a[232:235], v[72:75], v[28:31], a[232:235]\n"
    "ds_read_b128 v[56:59], v191 offset:12288\n"
    "v_mfma_f32_16x16x32_bf16 a[236:239], v[76:79], v[28:31], a[236:239]\n"
    "v_mfma_f32_16x16x32_bf16 a[200:203], v[72:75], v[24:27], a[200:203]\n"
    "ds_read_b128 v[60:63], v191 offset:14336\n"
    "v_mfma_f32_16x16x32_bf16 a[204:207], v[76:79], v[24:27], a[204:207]\n"
    "buffer_load_dwordx4 v134, s[44:47], s49 offen offset:2048 nt lds\n"
    "v_mfma_f32_16x16x32_bf16 a[168:171], v[72:75], v[20:23], a[168:171]\n"
    "ds_read_b64_tr_b16 v[100:101], v141 offset:32768\n"
    "v_mfma_f32_16x16x32_bf16 a[172:175], v[76:79], v[20:23], a[172:175]\n"
    "v_mfma_f32_16x16x32_bf16 a[136:139], v[72:75], v[16:19], a[136:139]\n"
    "ds_read_b64_tr_b16 v[102:103], v141 offset:33280\n"
    "v_mfma_f32_16x16x32_bf16 a[140:143], v[76:79], v[16:19], a[140:143]\n"
    "v_mfma_f32_16x16x32_bf16 a[104:107], v[72:75], v[12:15], a[104:107]\n"
    "ds_read_b64_tr_b16 v[104:105], v142 offset:32768\n"
    "v_mfma_f32_16x16x32_bf16 a[108:111], v[76:79], v[12:15], a[108:111]\n"
    "v_mfma_f32_16x16x32_bf16 a[72:75], v[72:75], v[8:11], a[72:75]\n"
    "ds_read_b64_tr_b16 v[106:107], v142 offset:33280\n"
    "v_mfma_f32_16x16x32_bf16 a[76:79], v[76:79], v[8:11], a[76:79]\n"
    "buffer_load_dwordx4 v135, s[44:47], s49 offen offset:3072 nt lds\n"
    "v_mfma_f32_16x16x32_bf16 a[40:43], v[72:75], v[4:7], a[40:43]\n"
    "ds_read_b64_tr_b16 v[108:109], v143 offset:32768\n"
    "v_mfma_f32_16x16x32_bf16 a[44:47], v[76:79], v[4:7], a[44:47]\n"
    "s_add_u32 s49, s49, 0x200000\n"
    "v_mfma_f32_16x16x32_bf16 a[8:11], v[72:75], v[0:3], a[8:11]\n"
    "ds_read_b64_tr_b16 v[110:111], v143 offset:33280\n"
    "v_mfma_f32_16x16x32_bf16 a[12:15], v[76:79], v[0:3], a[12:15]\n"
    "s_and_b32 s49, s49, 0xfffffff\n"
    "v_mfma_f32_16x16x32_bf16 a[16:19], v[80:83], v[0:3], a[16:19]\n"
    "ds_read_b64_tr_b16 v[112:113], v140 offset:36864\n"
    "v_mfma_f32_16x16x32_bf16 a[20:23], v[84:87], v[0:3], a[20:23]\n"
    "s_add_u32 s48, s78, 131584\n"
    "v_mfma_f32_16x16x32_bf16 a[48:51], v[80:83], v[4:7], a[48:51]\n"
    "ds_read_b64_tr_b16 v[114:115], v140 offset:37376\n"
    "s_add_u32 m0, s50, 147456\n"
    "v_mfma_f32_16x16x32_bf16 a[52:55], v[84:87], v[4:7], a[52:55]\n"
    "buffer_load_dwordx4 v154, s[40:43], s48 offen offset:0 lds\n"
    "v_mfma_f32_16x16x32_bf16 a[80:83], v[80:83], v[8:11], a[80:83]\n"
    "ds_read_b64_tr_b16 v[116:117], v141 offset:36864\n"
    "v_mfma_f32_16x16x32_bf16 a[84:87], v[84:87], v[8:11], a[84:87]\n"
    "v_mfma_f32_16x16x32_bf16 a[112:115], v[80:83], v[12:15], a[112:115]\n"
    "ds_read_b64_tr_b16 v[118:119], v141 offset:37376\n"
    "v_mfma_f32_16x16x32_bf16 a[116:119], v[84:87], v[12:15], a[116:119]\n"
    "v_mfma_f32_16x16x32_bf16 a[144:147], v[80:83], v[16:19], a[144:147]\n"
    "ds_read_b64_tr_b16 v[120:121], v142 offset:36864\n"
    "v_mfma_f32_16x16x32_bf16 a[148:151], v[84:87], v[16:19], a[148:151]\n"
    "v_mfma_f32_16x16x32_bf16 a[176:179], v[80:83], v[20:23], a[176:179]\n"
    "ds_read_b64_tr_b16 v[122:123], v142 offset:37376\n"
    "v_mfma_f32_16x16x32_bf16 a[180:183], v[84:87], v[20:23], a[180:183]\n"
    "buffer_load_dwordx4 v155, s[40:43], s48 offen offset:1024 lds\n"
    "v_mfma_f32_16x16x32_bf16 a[208:211], v[80:83], v[24:27], a[208:211]\n"
    "ds_read_b64_tr_b16 v[124:125], v143 offset:36864\n"
    "v_mfma_f32_16x16x32_bf16 a[212:215], v[84:87], v[24:27], a[212:215]\n"
    "v_mfma_f32_16x16x32_bf16 a[240:243], v[80:83], v[28:31], a[240:243]\n"
    "ds_read_b64_tr_b16 v[126:127], v143 offset:37376\n"
    "v_mfma_f32_16x16x32_bf16 a[244:247], v[84:87], v[28:31], a[244:247]\n"
    "v_mfma_f32_16x16x32_bf16 a[248:251], v[88:91], v[28:31], a[248:251]\n"
    "v_mfma_f32_16x16x32_bf16 a[252:255], v[92:95], v[28:31], a[252:255]\n"
    "v_mfma_f32_16x16x32_bf16 a[216:219], v[88:91], v[24:27], a[216:219]\n"
    "v_mfma_f32_16x16x32_bf16 a[220:223], v[92:95], v[24:27], a[220:223]\n"
    "buffer_load_dwordx4 v156, s[40:43], s48 offen offset:2048 lds\n"
    "v_mfma_f32_16x16x32_bf16 a[184:187], v[88:91], v[20:23], a[184:187]\n"
    "v_mfma_f32_16x16x32_bf16 a[188:191], v[92:95], v[20:23], a[188:191]\n"
    "v_mfma_f32_16x16x32_bf16 a[152:155], v[88:91], v[16:19], a[152:155]\n"
    "v_mfma_f32_16x16x32_bf16 a[156:159], v[92:95], v[16:19], a[156:159]\n"
    "v_mfma_f32_16x16x32_bf16 a[120:123], v[88:91], v[12:15], a[120:123]\n"
    "v_mfma_f32_16x16x32_bf16 a[124:127], v[92:95], v[12:15], a[124:127]\n"
    "v_mfma_f32_16x16x32_bf16 a[88:91], v[88:91], v[8:11], a[88:91]\n"
    "v_mfma_f32_16x16x32_bf16 a[92:95], v[92:95], v[8:11], a[92:95]\n"
    "buffer_load_dwordx4 v157, s[40:43], s48 offen offset:3072 lds\n"
    "v_mfma_f32_16x16x32_bf16 a[56:59], v[88:91], v[4:7], a[56:59]\n"
    "v_mfma_f32_16x16x32_bf16 a[60:63], v[92:95], v[4:7], a[60:63]\n"
    "v_mfma_f32_16x16x32_bf16 a[24:27], v[88:91], v[0:3], a[24:27]\n"
    "v_mfma_f32_16x16x32_bf16 a[28:31], v[92:95], v[0:3], a[28:31]\n"
    "s_waitcnt vmcnt(18) lgkmcnt(0)\n"
    "s_barrier\n"
    "v_mfma_f32_16x16x32_bf16 a[0:3], v[96:99], v[32:35], a[0:3]\n"
    "ds_read_b64_tr_b16 v[64:65], v144 offset:0\n"
    "v_mfma_f32_16x16x32_bf16 a[4:7], v[100:103], v[32:35], a[4:7]\n"
    "v_mfma_f32_16x16x32_bf16 a[32:35], v[96:99], v[36:39], a[32:35]\n"
    "ds_read_b64_tr_b16 v[66:67], v144 offset:512\n"
    "s_add_u32 m0, s50, 49152\n"
    "s_mov_b64 exec, s[30:31]\n"
    "global_store_dwordx4 v138, v[164:167], s[26:27] offset:1024 sc0 sc1\n"
    "s_mov_b64 exec, -1\n"
    "v_mfma_f32_16x16x32_bf16 a[36:39], v[100:103], v[36:39], a[36:39]\n"
    "buffer_load_dwordx4 v132, s[44:47], s49 offen offset:0 nt lds\n"
    "v_mfma_f32_16x16x32_bf16 a[64:67], v[96:99], v[40:43], a[64:67]\n"
    "ds_read_b128 v[0:3], v192 offset:0\n"
    "v_mfma_f32_16x16x32_bf16 a[68:71], v[100:103], v[40:43], a[68:71]\n"
    "v_mfma_f32_16x16x32_bf16 a[96:99], v[96:99], v[44:47], a[96:99]\n"
    "ds_read_b128 v[4:7], v192 offset:2048\n"
    "v_mfma_f32_16x16x32_bf16 a[100:103], v[100:103], v[44:47], a[100:103]\n"
    "v_mfma_f32_16x16x32_bf16 a[128:131], v[96:99], v[48:51], a[128:131]\n"
    "ds_read_b128 v[8:11], v192 offset:4096\n"
    "v_mfma_f32_16x16x32_bf16 a[132:135], v[100:103], v[48:51], a[132:135]\n"
    "v_mfma_f32_16x16x32_bf16 a[160:163], v[96:99], v[52:55], a[160:163]\n"
    "ds_read_b128 v[12:15], v192 offset:6144\n"
    "v_mfma_f32_16x16x32_bf16 a[164:167], v[100:103], v[52:55], a[164:167]\n"
    "buffer_load_dwordx4 v133, s[44:47], s49 offen offset:1024 nt lds\n"
    "v_mfma_f32_16x16x32_bf16 a[192:195], v[96:99], v[56:59], a[192:195]\n"
    "ds_read_b128 v[16:19], v192 offset:8192\n"
    "v_mfma_f32_16x16x32_bf16 a[196:199], v[100:103], v[56:59], a[196:199]\n"
    "v_mfma_f32_16x16x32_bf16 a[224:227], v[96:99], v[60:63], a[224:227]\n"
    "ds_read_b128 v[20:23], v192 offset:10240\n"
    "v_mfma_f32_16x16x32_bf16 a[228:231], v[100:103], v[60:63], a[228:231]\n"
    "v_mfma_f32_16x16x32_bf16 a[232:235], v[104:107], v[60:63], a[232:235]\n"
    "ds_read_b128 v[24:27], v192 offset:12288\n"
    "v_mfma_f32_16x16x32_bf16 a[236:239], v[108:111], v[60:63], a[236:239]\n"
    "v_mfma_f32_16x16x32_bf16 a[200:203], v[104:107], v[56:59], a[200:203]\n"
    "ds_read_b128 v[28:31], v192 offset:14336\n"
    "v_mfma_f32_16x16x32_bf16 a[204:207], v[108:111], v[56:59], a[204:207]\n"
    "buffer_load_dwordx4 v134, s[44:47], s49 offen offset:2048 nt lds\n"
    "v_mfma_f32_16x16x32_bf16 a[168:171], v[104:107], v[52:55], a[168:171]\n"
    "ds_read_b64_tr_b16 v[68:69], v145 offset:0\n"
    "v_mfma_f32_16x16x32_bf16 a[172:175], v[108:111], v[52:55], a[172:175]\n"
    "v_mfma_f32_16x16x32_bf16 a[136:139], v[104:107], v[48:51], a[136:139]\n"
    "ds_read_b64_tr_b16 v[70:71], v145 offset:512\n"
    "v_mfma_f32_16x16x32_bf16 a[140:143], v[108:111], v[48:51], a[140:143]\n"
    "v_mfma_f32_16x16x32_bf16 a[104:107], v[104:107], v[44:47], a[104:107]\n"
    "ds_read_b64_tr_b16 v[72:73], v146 offset:0\n"
    "v_mfma_f32_16x16x32_bf16 a[108:111], v[108:111], v[44:47], a[108:111]\n"
    "v_mfma_f32_16x16x32_bf16 a[72:75], v[104:107], v[40:43], a[72:75]\n"
    "ds_read_b64_tr_b16 v[74:75], v146 offset:512\n"
    "v_mfma_f32_16x16x32_bf16 a[76:79], v[108:111], v[40:43], a[76:79]\n"
    "buffer_load_dwordx4 v135, s[44:47], s49 offen offset:3072 nt lds\n"
    "v_mfma_f32_16x16x32_bf16 a[40:43], v[104:107], v[36:39], a[40:43]\n"
    "ds_read_b64_tr_b16 v[76:77], v147 offset:0\n"
    "v_mfma_f32_16x16x32_bf16 a[44:47], v[108:111], v[36:39], a[44:47]\n"
    "s_add_u32 s49, s49, 0x200000\n"
    "v_mfma_f32_16x16x32_bf16 a[8:11], v[104:107], v[32:35], a[8:11]\n"
    "ds_read_b64_tr_b16 v[78:79], v147 offset:512\n"
    "v_mfma_f32_16x16x32_bf16 a[12:15], v[108:111], v[32:35], a[12:15]\n"
    "s_and_b32 s49, s49, 0xfffffff\n"
    "v_mfma_f32_16x16x32_bf16 a[16:19], v[112:115], v[32:35], a[16:19]\n"
    "ds_read_b64_tr_b16 v[80:81], v144 offset:4096\n"
    "v_mfma_f32_16x16x32_bf16 a[20:23], v[116:119], v[32:35], a[20:23]\n"
    "s_add_u32 s48, s78, 640\n"
    "v_mfma_f32_16x16x32_bf16 a[48:51], v[112:115], v[36:39], a[48:51]\n"
    "ds_read_b64_tr_b16 v[82:83], v144 offset:4608\n"
    "s_add_u32 m0, s50, 0\n"
    "v_mfma_f32_16x16x32_bf16 a[52:55], v[116:119], v[36:39], a[52:55]\n"
    "buffer_load_dwordx4 v154, s[40:43], s48 offen offset:0 lds\n"
    "v_mfma_f32_16x16x32_bf16 a[80:83], v[112:115], v[40:43], a[80:83]\n"
    "ds_read_b64_tr_b16 v[84:85], v145 offset:4096\n"
    "v_mfma_f32_16x16x32_bf16 a[84:87], v[116:119], v[40:43], a[84:87]\n"
    "v_mfma_f32_16x16x32_bf16 a[112:115], v[112:115], v[44:47], a[112:115]\n"
    "ds_read_b64_tr_b16 v[86:87], v145 offset:4608\n"
    "v_mfma_f32_16x16x32_bf16 a[116:119], v[116:119], v[44:47], a[116:119]\n"
    "v_mfma_f32_16x16x32_bf16 a[144:147], v[112:115], v[48:51], a[144:147]\n"
    "ds_read_b64_tr_b16 v[88:89], v146 offset:4096\n"
    "v_mfma_f32_16x16x32_bf16 a[148:151], v[116:119], v[48:51], a[148:151]\n"
    "v_mfma_f32_16x16x32_bf16 a[176:179], v[112:115], v[52:55], a[176:179]\n"
    "ds_read_b64_tr_b16 v[90:91], v146 offset:4608\n"
    "v_mfma_f32_16x16x32_bf16 a[180:183], v[116:119], v[52:55], a[180:183]\n"
    "buffer_load_dwordx4 v155, s[40:43], s48 offen offset:1024 lds\n"
    "v_mfma_f32_16x16x32_bf16 a[208:211], v[112:115], v[56:59], a[208:211]\n"
    "ds_read_b64_tr_b16 v[92:93], v147 offset:4096\n"
    "v_mfma_f32_16x16x32_bf16 a[212:215], v[116:119], v[56:59], a[212:215]\n"
    "v_mfma_f32_16x16x32_bf16 a[240:243], v[112:115], v[60:63], a[240:243]\n"
    "ds_read_b64_tr_b16 v[94:95], v147 offset:4608\n"
    "v_mfma_f32_16x16x32_bf16 a[244:247], v[116:119], v[60:63], a[244:247]\n"
    "v_mfma_f32_16x16x32_bf16 a[248:251], v[120:123], v[60:63], a[248:251]\n"
    "v_mfma_f32_16x16x32_bf16 a[252:255], v[124:127], v[60:63], a[252:255]\n"
    "v_mfma_f32_16x16x32_bf16 a[216:219], v[120:123], v[56:59], a[216:219]\n"
    "v_mfma_f32_16x16x32_bf16 a[220:223], v[124:127], v[56:59], a[220:223]\n"
    "buffer_load_dwordx4 v156, s[40:43], s48 offen offset:2048 lds\n"
    "v_mfma_f32_16x16x32_bf16 a[184:187], v[120:123], v[52:55], a[184:187]\n"
    "v_mfma_f32_16x16x32_bf16 a[188:191], v[124:127], v[52:55], a[188:191]\n"
    "v_mfma_f32_16x16x32_bf16 a[152:155], v[120:123], v[48:51], a[152:155]\n"
    "s_memrealtime s[28:29]\n"
    "v_mfma_f32_16x16x32_bf16 a[156:159], v[124:127], v[48:51], a[156:159]\n"
    "v_mfma_f32_16x16x32_bf16 a[120:123], v[120:123], v[44:47], a[120:123]\n"
    "v_mfma_f32_16x16x32_bf16 a[124:127], v[124:127], v[44:47], a[124:127]\n"
    "v_mfma_f32_16x16x32_bf16 a[88:91], v[120:123], v[40:43], a[88:91]\n"
    "v_mfma_f32_16x16x32_bf16 a[92:95], v[124:127], v[40:43], a[92:95]\n"
    "buffer_load_dwordx4 v157, s[40:43], s48 offen offset:3072 lds\n"
    "v_mfma_f32_16x16x32_bf16 a[56:59], v[120:123], v[36:39], a[56:59]\n"
    "v_mfma_f32_16x16x32_bf16 a[60:63], v[124:127], v[36:39], a[60:63]\n"
    "v_mfma_f32_16x16x32_bf16 a[24:27], v[120:123], v[32:35], a[24:27]\n"
    "v_mfma_f32_16x16x32_bf16 a[28:31], v[124:127], v[32:35], a[28:31]\n"
    "s_waitcnt vmcnt(18) lgkmcnt(0)\n"
    "s_barrier\n"
    "v_mfma_f32_16x16x32_bf16 a[0:3], v[64:67], v[0:3], a[0:3]\n"
    "ds_read_b64_tr_b16 v[96:97], v144 offset:32768\n"
    "v_mfma_f32_16x16x32_bf16 a[4:7], v[68:71], v[0:3], a[4:7]\n"
    "v_mfma_f32_16x16x32_bf16 a[32:35], v[64:67], v[4:7], a[32:35]\n"
    "ds_read_b64_tr_b16 v[98:99], v144 offset:33280\n"
    "s_add_u32 m0, s50, 81920\n"
    "s_mov_b64 exec, s[30:31]\n"
    "global_store_dwordx4 v138, v[168:171], s[26:27] offset:2048 sc0 sc1\n"
    "s_mov_b64 exec, -1\n"
    "v_mfma_f32_16x16x32_bf16 a[36:39], v[68:71], v[4:7], a[36:39]\n"
    "buffer_load_dwordx4 v132, s[44:47], s49 offen offset:0 nt lds\n"
    "v_mfma_f32_16x16x32_bf16 a[64:67], v[64:67], v[8:11], a[64:67]\n"
    "ds_read_b128 v[32:35], v193 offset:0\n"
    "v_mfma_f32_16x16x32_bf16 a[68:71], v[68:71], v[8:11], a[68:71]\n"
    "s_lshr_b32 s54, s24, 4\n"
    "s_add_u32 s33, s33, s54\n"
    "v_mfma_f32_16x16x32_bf16 a[96:99], v[64:67], v[12:15], a[96:99]\n"
    "ds_read_b128 v[36:39], v193 offset:2048\n"
    "s_add_u32 s32, s32, s54\n"
    "v_mfma_f32_16x16x32_bf16 a[100:103], v[68:71], v[12:15], a[100:103]\n"
    "s_sub_u32 s54, s33, s28\n"
    "s_max_i32 s54, s54, -250\n"
    "v_mfma_f32_16x16x32_bf16 a[128:131], v[64:67], v[16:19], a[128:131]\n"
    "ds_read_b128 v[40:43], v193 offset:4096\n"
    "v_mfma_f32_16x16x32_bf16 a[132:135], v[68:71], v[16:19], a[132:135]\n"
    "s_add_u32 s33, s28, s54\n"
    "s_cmp_gt_i32 s54, 0\n"
    "v_mfma_f32_16x16x32_bf16 a[160:163], v[64:67], v[20:23], a[160:163]\n"
    "ds_read_b128 v[44:47], v193 offset:6144\n"
    "s_cbranch_scc1 PACEI12_%=\n"
    "PACEBI12_%=:\n"
    "v_mfma_f32_16x16x32_bf16 a[164:167], v[68:71], v[20:23], a[164:167]\n"
    "buffer_load_dwordx4 v133, s[44:47], s49 offen offset:1024 nt lds\n"
    "v_mfma_f32_16x16x32_bf16 a[192:195], v[64:67], v[24:27], a[192:195]\n"
    "ds_read_b128 v[48:51], v193 offset:8192\n"
    "v_mfma_f32_16x16x32_bf16 a[196:199], v[68:71], v[24:27], a[196:199]\n"
    "s_sub_u32 s24, s24, 16\n"
    "s_max_i32 s24, s24, s35\n"
    "v_mfma_f32_16x16x32_bf16 a[224:227], v[64:67], v[28:31], a[224:227]\n"
    "ds_read_b128 v[52:55], v193 offset:10240\n"
    "v_mfma_f32_16x16x32_bf16 a[228:231], v[68:71], v[28:31], a[228:231]\n"
    "v_mfma_f32_16x16x32_bf16 a[232:235], v[72:75], v[28:31], a[232:235]\n"
    "ds_read_b128 v[56:59], v193 offset:12288\n"
    "v_mfma_f32_16x16x32_bf16 a[236:239], v[76:79], v[28:31], a[236:239]\n"
    "v_mfma_f32_16x16x32_bf16 a[200:203], v[72:75], v[24:27], a[200:203]\n"
    "ds_read_b128 v[60:63], v193 offset:14336\n"
    "v_mfma_f32_16x16x32_bf16 a[204:207], v[76:79], v[24:27], a[204:207]\n"
    "buffer_load_dwordx4 v134, s[44:47], s49 offen offset:2048 nt lds\n"
    "v_mfma_f32_16x16x32_bf16 a[168:171], v[72:75], v[20:23], a[168:171]\n"
    "ds_read_b64_tr_b16 v[100:101], v145 offset:32768\n"
    "v_mfma_f32_16x16x32_bf16 a[172:175], v[76:79], v[20:23], a[172:175]\n"
    "v_mfma_f32_16x16x32_bf16 a[136:139], v[72:75], v[16:19], a[136:139]\n"
    "ds_read_b64_tr_b16 v[102:103], v145 offset:33280\n"
    "v_mfma_f32_16x16x32_bf16 a[140:143], v[76:79], v[16:19], a[140:143]\n"
    "v_mfma_f32_16x16x32_bf16 a[104:107], v[72:75], v[12:15], a[104:107]\n"
    "ds_read_b64_tr_b16 v[104:105], v146 offset:32768\n"
    "v_mfma_f32_16x16x32_bf16 a[108:111], v[76:79], v[12:15], a[108:111]\n"
    "v_mfma_f32_16x16x32_bf16 a[72:75], v[72:75], v[8:11], a[72:75]\n"
    "ds_read_b64_tr_b16 v[106:107], v146 offset:33280\n"
    "v_mfma_f32_16x16x32_bf16 a[76:79], v[76:79], v[8:11], a[76:79]\n"
    "buffer_load_dwordx4 v135, s[44:47], s49 offen offset:3072 nt lds\n"
    "v_mfma_f32_16x16x32_bf16 a[40:43], v[72:75], v[4:7], a[40:43]\n"
    "ds_read_b64_tr_b16 v[108:109], v147 offset:32768\n"
    "v_mfma_f32_16x16x32_bf16 a[44:47], v[76:79], v[4:7], a[44:47]\n"
    "s_add_u32 s49, s49, 0x200000\n"
    "v_mfma_f32_16x16x32_bf16 a[8:11], v[72:75], v[0:3], a[8:11]\n"
    "ds_read_b64_tr_b16 v[110:111], v147 offset:33280\n"
    "v_mfma_f32_16x16x32_bf16 a[12:15], v[76:79], v[0:3], a[12:15]\n"
    "s_and_b32 s49, s49, 0xfffffff\n"
    "v_mfma_f32_16x16x32_bf16 a[16:19], v[80:83], v[0:3], a[16:19]\n"
    "ds_read_b64_tr_b16 v[112:113], v144 offset:36864\n"
    "v_mfma_f32_16x16x32_bf16 a[20:23], v[84:87], v[0:3], a[20:23]\n"
    "s_add_u32 s48, s78, 131712\n"
    "v_mfma_f32_16x16x32_bf16 a[48:51], v[80:83], v[4:7], a[48:51]\n"
    "ds_read_b64_tr_b16 v[114:115], v144 offset:37376\n"
    "s_add_u32 m0, s50, 32768\n"
    "v_mfma_f32_16x16x32_bf16 a[52:55], v[84:87], v[4:7], a[52:55]\n"
    "buffer_load_dwordx4 v154, s[40:43], s48 offen offset:0 lds\n"
    "v_mfma_f32_16x16x32_bf16 a[80:83], v[80:83], v[8:11], a[80:83]\n"
    "ds_read_b64_tr_b16 v[116:117], v145 offset:36864\n"
    "v_mfma_f32_16x16x32_bf16 a[84:87], v[84:87], v[8:11], a[84:87]\n"
    "v_mfma_f32_16x16x32_bf16 a[112:115], v[80:83], v[12:15], a[112:115]\n"
    "ds_read_b64_tr_b16 v[118:119], v145 offset:37376\n"
    "v_mfma_f32_16x16x32_bf16 a[116:119], v[84:87], v[12:15], a[116:119]\n"
    "v_mfma_f32_16x16x32_bf16 a[144:147], v[80:83], v[16:19], a[144:147]\n"
    "ds_read_b64_tr_b16 v[120:121], v146 offset:36864\n"
    "v_mfma_f32_16x16x32_bf16 a[148:151], v[84:87], v[16:19], a[148:151]\n"
    "v_mfma_f32_16x16x32_bf16 a[176:179], v[80:83], v[20:23], a[176:179]\n"
    "ds_read_b64_tr_b16 v[122:123], v146 offset:37376\n"
    "v_mfma_f32_16x16x32_bf16 a[180:183], v[84:87], v[20:23], a[180:183]\n"
    "buffer_load_dwordx4 v155, s[40:43], s48 offen offset:1024 lds\n"
    "v_mfma_f32_16x16x32_bf16 a[208:211], v[80:83], v[24:27], a[208:211]\n"
    "ds_read_b64_tr_b16 v[124:125], v147 offset:36864\n"
    "v_mfma_f32_16x16x32_bf16 a[212:215], v[84:87], v[24:27], a[212:215]\n"
    "v_mfma_f32_16x16x32_bf16 a[240:243], v[80:83], v[28:31], a[240:243]\n"
    "ds_read_b64_tr_b16 v[126:127], v147 offset:37376\n"
    "v_mfma_f32_16x16x32_bf16 a[244:247], v[84:87], v[28:31], a[244:247]\n"
    "v_mfma_f32_16x16x32_bf16 a[248:251], v[88:91], v[28:31], a[248:251]\n"
    "v_mfma_f32_16x16x32_bf16 a[252:255], v[92:95], v[28:31], a[252:255]\n"
    "v_mfma_f32_16x16x32_bf16 a[216:219], v[88:91], v[24:27], a[216:219]\n"
    "v_mfma_f32_16x16x32_bf16 a[220:223], v[92:95], v[24:27], a[220:223]\n"
    "buffer_load_dwordx4 v156, s[40:43], s48 offen offset:2048 lds\n"
    "v_mfma_f32_16x16x32_bf16 a[184:187], v[88:91], v[20:23], a[184:187]\n"
    "v_mfma_f32_16x16x32_bf16 a[188:191], v[92:95], v[20:23], a[188:191]\n"
    "v_mfma_f32_16x16x32_bf16 a[152:155], v[88:91], v[16:19], a[152:155]\n"
    "v_mfma_f32_16x16x32_bf16 a[156:159], v[92:95], v[16:19], a[156:159]\n"
    "v_mfma_f32_16x16x32_bf16 a[120:123], v[88:91], v[12:15], a[120:123]\n"
    "v_mfma_f32_16x16x32_bf16 a[124:127], v[92:95], v[12:15], a[124:127]\n"
    "v_mfma_f32_16x16x32_bf16 a[88:91], v[88:91], v[8:11], a[88:91]\n"
    "v_mfma_f32_16x16x32_bf16 a[92:95], v[92:95], v[8:11], a[92:95]\n"
    "buffer_load_dwordx4 v157, s[40:43], s48 offen offset:3072 lds\n"
    "v_mfma_f32_16x16x32_bf16 a[56:59], v[88:91], v[4:7], a[56:59]\n"
    "v_mfma_f32_16x16x32_bf16 a[60:63], v[92:95], v[4:7], a[60:63]\n"
    "v_mfma_f32_16x16x32_bf16 a[24:27], v[88:91], v[0:3], a[24:27]\n"
    "v_mfma_f32_16x16x32_bf16 a[28:31], v[92:95], v[0:3], a[28:31]\n"
    "s_waitcnt vmcnt(18) lgkmcnt(0)\n"
    "s_barrier\n"
    "v_mfma_f32_16x16x32_bf16 a[0:3], v[96:99], v[32:35], a[0:3]\n"
    "ds_read_b64_tr_b16 v[64:65], v140 offset:0\n"
    "v_mfma_f32_16x16x32_bf16 a[4:7], v[100:103], v[32:35], a[4:7]\n"
    "v_mfma_f32_16x16x32_bf16 a[32:35], v[96:99], v[36:39], a[32:35]\n"
    "ds_read_b64_tr_b16 v[66:67], v140 offset:512\n"
    "s_add_u32 m0, s50, 114688\n"
    "s_mov_b64 exec, s[30:31]\n"
    "global_store_dwordx4 v138, v[172:175], s[26:27] offset:3072 sc0 sc1\n"
    "s_mov_b64 exec, -1\n"
    "v_mfma_f32_16x16x32_bf16 a[36:39], v[100:103], v[36:39], a[36:39]\n"
    "buffer_load_dwordx4 v132, s[44:47], s49 offen offset:0 nt lds\n"
    "v_mfma_f32_16x16x32_bf16 a[64:67], v[96:99], v[40:43], a[64:67]\n"
    "ds_read_b128 v[0:3], v194 offset:0\n"
    "v_mfma_f32_16x16x32_bf16 a[68:71], v[100:103], v[40:43], a[68:71]\n"
    "v_mfma_f32_16x16x32_bf16 a[96:99], v[96:99], v[44:47], a[96:99]\n"
    "ds_read_b128 v[4:7], v194 offset:2048\n"
    "v_mfma_f32_16x16x32_bf16 a[100:103], v[100:103], v[44:47], a[100:103]\n"
    "v_mfma_f32_16x16x32_bf16 a[128:131], v[96:99], v[48:51], a[128:131]\n"
    "ds_read_b128 v[8:11], v194 offset:4096\n"
    "v_mfma_f32_16x16x32_bf16 a[132:135], v[100:103], v[48:51], a[132:135]\n"
    "v_mfma_f32_16x16x32_bf16 a[160:163], v[96:99], v[52:55], a[160:163]\n"
    "ds_read_b128 v[12:15], v194 offset:6144\n"
    "v_mfma_f32_16x16x32_bf16 a[164:167], v[100:103], v[52:55], a[164:167]\n"
    "buffer_load_dwordx4 v133, s[44:47], s49 offen offset:1024 nt lds\n"
    "v_mfma_f32_16x16x32_bf16 a[192:195], v[96:99], v[56:59], a[192:195]\n"
    "ds_read_b128 v[16:19], v194 offset:8192\n"
    "v_mfma_f32_16x16x32_bf16 a[196:199], v[100:103], v[56:59], a[196:199]\n"
    "v_mfma_f32_16x16x32_bf16 a[224:227], v[96:99], v[60:63], a[224:227]\n"
    "ds_read_b128 v[20:23], v194 offset:10240\n"
    "v_mfma_f32_16x16x32_bf16 a[228:231], v[100:103], v[60:63], a[228:231]\n"
    "v_mfma_f32_16x16x32_bf16 a[232:235], v[104:107], v[60:63], a[232:235]\n"
    "ds_read_b128 v[24:27], v194 offset:12288\n"
    "v_mfma_f32_16x16x32_bf16 a[236:239], v[108:111], v[60:63], a[236:239]\n"
    "v_mfma_f32_16x16x32_bf16 a[200:203], v[104:107], v[56:59], a[200:203]\n"
    "ds_read_b128 v[28:31], v194 offset:14336\n"
    "v_mfma_f32_16x16x32_bf16 a[204:207], v[108:111], v[56:59], a[204:207]\n"
    "buffer_load_dwordx4 v134, s[44:47], s49 offen offset:2048 nt lds\n"
    "v_mfma_f32_16x16x32_bf16 a[168:171], v[104:107], v[52:55], a[168:171]\n"
    "ds_read_b64_tr_b16 v[68:69], v141 offset:0\n"
    "v_mfma_f32_16x16x32_bf16 a[172:175], v[108:111], v[52:55], a[172:175]\n"
    "v_mfma_f32_16x16x32_bf16 a[136:139], v[104:107], v[48:51], a[136:139]\n"
    "ds_read_b64_tr_b16 v[70:71], v141 offset:512\n"
    "v_mfma_f32_16x16x32_bf16 a[140:143], v[108:111], v[48:51], a[140:143]\n"
    "v_mfma_f32_16x16x32_bf16 a[104:107], v[104:107], v[44:47], a[104:107]\n"
    "ds_read_b64_tr_b16 v[72:73], v142 offset:0\n"
    "v_mfma_f32_16x16x32_bf16 a[108:111], v[108:111], v[44:47], a[108:111]\n"
    "v_mfma_f32_16x16x32_bf16 a[72:75], v[104:107], v[40:43], a[72:75]\n"
    "ds_read_b64_tr_b16 v[74:75], v142 offset:512\n"
    "v_mfma_f32_16x16x32_bf16 a[76:79], v[108:111], v[40:43], a[76:79]\n"
    "buffer_load_dwordx4 v135, s[44:47], s49 offen offset:3072 nt lds\n"
    "v_mfma_f32_16x16x32_bf16 a[40:43], v[104:107], v[36:39], a[40:43]\n"
    "ds_read_b64_tr_b16 v[76:77], v143 offset:0\n"
    "v_mfma_f32_16x16x32_bf16 a[44:47], v[108:111], v[36:39], a[44:47]\n"
    "s_add_u32 s49, s49, 0x200000\n"
    "v_mfma_f32_16x16x32_bf16 a[8:11], v[104:107], v[32:35], a[8:11]\n"
    "ds_read_b64_tr_b16 v[78:79], v143 offset:512\n"
    "v_mfma_f32_16x16x32_bf16 a[12:15], v[108:111], v[32:35], a[12:15]\n"
    "s_and_b32 s49, s49, 0xfffffff\n"
    "v_mfma_f32_16x16x32_bf16 a[16:19], v[112:115], v[32:35], a[16:19]\n"
    "ds_read_b64_tr_b16 v[80:81], v140 offset:4096\n"
    "v_mfma_f32_16x16x32_bf16 a[20:23], v[116:119], v[32:35], a[20:23]\n"
    "s_add_u32 s48, s78, 768\n"
    "v_mfma_f32_16x16x32_bf16 a[48:51], v[112:115], v[36:39], a[48:51]\n"
    "ds_read_b64_tr_b16 v[82:83], v140 offset:4608\n"
    "s_add_u32 m0, s50, 65536\n"
    "v_mfma_f32_16x16x32_bf16 a[52:55], v[116:119], v[36:39], a[52:55]\n"
    "buffer_load_dwordx4 v154, s[40:43], s48 offen offset:0 lds\n"
    "v_mfma_f32_16x16x32_bf16 a[80:83], v[112:115], v[40:43], a[80:83]\n"
    "ds_read_b64_tr_b16 v[84:85], v141 offset:4096\n"
    "v_mfma_f32_16x16x32_bf16 a[84:87], v[116:119], v[40:43], a[84:87]\n"
    "v_mfma_f32_16x16x32_bf16 a[112:115], v[112:115], v[44:47], a[112:115]\n"
    "ds_read_b64_tr_b16 v[86:87], v141 offset:4608\n"
    "v_mfma_f32_16x16x32_bf16 a[116:119], v[116:119], v[44:47], a[116:119]\n"
    "v_mfma_f32_16x16x32_bf16 a[144:147], v[112:115], v[48:51], a[144:147]\n"
    "ds_read_b64_tr_b16 v[88:89], v142 offset:4096\n"
    "v_mfma_f32_16x16x32_bf16 a[148:151], v[116:119], v[48:51], a[148:151]\n"
    "v_mfma_f32_16x16x32_bf16 a[176:179], v[112:115], v[52:55], a[176:179]\n"
    "ds_read_b64_tr_b16 v[90:91], v142 offset:4608\n"
    "v_mfma_f32_16x16x32_bf16 a[180:183], v[116:119], v[52:55], a[180:183]\n"
    "buffer_load_dwordx4 v155, s[40:43], s48 offen offset:1024 lds\n"
    "v_mfma_f32_16x16x32_bf16 a[208:211], v[112:115], v[56:59], a[208:211]\n"
    "ds_read_b64_tr_b16 v[92:93], v143 offset:4096\n"
    "v_mfma_f32_16x16x32_bf16 a[212:215], v[116:119], v[56:59], a[212:215]\n"
    "v_mfma_f32_16x16x32_bf16 a[240:243], v[112:115], v[60:63], a[240:243]\n"
    "ds_read_b64_tr_b16 v[94:95], v143 offset:4608\n"
    "v_mfma_f32_16x16x32_bf16 a[244:247], v[116:119], v[60:63], a[244:247]\n"
    "v_mfma_f32_16x16x32_bf16 a[248:251], v[120:123], v[60:63], a[248:251]\n"
    "v_mfma_f32_16x16x32_bf16 a[252:255], v[124:127], v[60:63], a[252:255]\n"
    "v_mfma_f32_16x16x32_bf16 a[216:219], v[120:123], v[56:59], a[216:219]\n"
    "v_mfma_f32_16x16x32_bf16 a[220:223], v[124:127], v[56:59], a[220:223]\n"
    "buffer_load_dwordx4 v156, s[40:43], s48 offen offset:2048 lds\n"
    "v_mfma_f32_16x16x32_bf16 a[184:187], v[120:123], v[52:55], a[184:187]\n"
    "v_mfma_f32_16x16x32_bf16 a[188:191], v[124:127], v[52:55], a[188:191]\n"
    "v_mfma_f32_16x16x32_bf16 a[152:155], v[120:123], v[48:51], a[152:155]\n"
    "s_memrealtime s[28:29]\n"
    "v_mfma_f32_16x16x32_bf16 a[156:159], v[124:127], v[48:51], a[156:159]\n"
    "v_mfma_f32_16x16x32_bf16 a[120:123], v[120:123], v[44:47], a[120:123]\n"
    "v_mfma_f32_16x16x32_bf16 a[124:127], v[124:127], v[44:47], a[124:127]\n"
    "v_mfma_f32_16x16x32_bf16 a[88:91], v[120:123], v[40:43], a[88:91]\n"
    "v_mfma_f32_16x16x32_bf16 a[92:95], v[124:127], v[40:43], a[92:95]\n"
    "buffer_load_dwordx4 v157, s[40:43], s48 offen offset:3072 lds\n"
    "v_mfma_f32_16x16x32_bf16 a[56:59], v[120:123], v[36:39], a[56:59]\n"
    "v_mfma_f32_16x16x32_bf16 a[60:63], v[124:127], v[36:39], a[60:63]\n"
    "v_mfma_f32_16x16x32_bf16 a[24:27], v[120:123], v[32:35], a[24:27]\n"
    "v_mfma_f32_16x16x32_bf16 a[28:31], v[124:127], v[32:35], a[28:31]\n"
    "s_waitcnt vmcnt(18) lgkmcnt(0)\n"
    "s_barrier\n"
    "v_mfma_f32_16x16x32_bf16 a[0:3], v[64:67], v[0:3], a[0:3]\n"
    "ds_read_b64_tr_b16 v[96:97], v140 offset:32768\n"
    "v_mfma_f32_16x16x32_bf16 a[4:7], v[68:71], v[0:3], a[4:7]\n"
    "s_cmp_eq_u32 s66, 2\n"
    "s_cselect_b64 s[30:31], -1, 1\n"
    "s_cselect_b64 s[26:27], s[60:61], s[62:63]\n"
    "v_mfma_f32_16x16x32_bf16 a[32:35], v[64:67], v[4:7], a[32:35]\n"
    "ds_read_b64_tr_b16 v[98:99], v140 offset:33280\n"
    "s_add_u32 m0, s50, 16384\n"
    "s_mov_b64 exec, s[30:31]\n"
    "global_store_dwordx4 v138, v[160:163], s[26:27] offset:0 sc0 sc1\n"
    "s_mov_b64 exec, -1\n"
    "v_mfma_f32_16x16x32_bf16 a[36:39], v[68:71], v[4:7], a[36:39]\n"
    "buffer_load_dwordx4 v132, s[44:47], s49 offen offset:0 nt lds\n"
    "v_mfma_f32_16x16x32_bf16 a[64:67], v[64:67], v[8:11], a[64:67]\n"
    "ds_read_b128 v[32:35], v195 offset:0\n"
    "v_mfma_f32_16x16x32_bf16 a[68:71], v[68:71], v[8:11], a[68:71]\n"
    "s_lshr_b32 s54, s24, 4\n"
    "s_add_u32 s33, s33, s54\n"
    "v_mfma_f32_16x16x32_bf16 a[96:99], v[64:67], v[12:15], a[96:99]\n"
    "ds_read_b128 v[36:39], v195 offset:2048\n"
    "s_add_u32 s32, s32, s54\n"
    "v_mfma_f32_16x16x32_bf16 a[100:103], v[68:71], v[12:15], a[100:103]\n"
    "s_sub_u32 s54, s33, s28\n"
    "s_max_i32 s54, s54, -250\n"
    "v_mfma_f32_16x16x32_bf16 a[128:131], v[64:67], v[16:19], a[128:131]\n"
    "ds_read_b128 v[40:43], v195 offset:4096\n"
    "v_mfma_f32_16x16x32_bf16 a[132:135], v[68:71], v[16:19], a[132:135]\n"
    "s_add_u32 s33, s28, s54\n"
    "s_cmp_gt_i32 s54, 0\n"
    "v_mfma_f32_16x16x32_bf16 a[160:163], v[64:67], v[20:23], a[160:163]\n"
    "ds_read_b128 v[44:47], v195 offset:6144\n"
    "s_cbranch_scc1 PACEI20_%=\n"
    "PACEBI20_%=:\n"
    "v_mfma_f32_16x16x32_bf16 a[164:167], v[68:71], v[20:23], a[164:167]\n"
    "buffer_load_dwordx4 v133, s[44:47], s49 offen offset:1024 nt lds\n"
    "v_mfma_f32_16x16x32_bf16 a[192:195], v[64:67], v[24:27], a[192:195]\n"
    "ds_read_b128 v[48:51], v195 offset:8192\n"
    "v_mfma_f32_16x16x32_bf16 a[196:199], v[68:71], v[24:27], a[196:199]\n"
    "s_sub_u32 s24, s24, 16\n"
    "s_max_i32 s24, s24, s35\n"
    "v_mfma_f32_16x16x32_bf16 a[224:227], v[64:67], v[28:31], a[224:227]\n"
    "ds_read_b128 v[52:55], v195 offset:10240\n"
    "v_mfma_f32_16x16x32_bf16 a[228:231], v[68:71], v[28:31], a[228:231]\n"
    "v_mfma_f32_16x16x32_bf16 a[232:235], v[72:75], v[28:31], a[232:235]\n"
    "ds_read_b128 v[56:59], v195 offset:12288\n"
    "v_mfma_f32_16x16x32_bf16 a[236:239], v[76:79], v[28:31], a[236:239]\n"
    "v_mfma_f32_16x16x32_bf16 a[200:203], v[72:75], v[24:27], a[200:203]\n"
    "ds_read_b128 v[60:63], v195 offset:14336\n"
    "v_mfma_f32_16x16x32_bf16 a[204:207], v[76:79], v[24:27], a[204:207]\n"
    "buffer_load_dwordx4 v134, s[44:47], s49 offen offset:2048 nt lds\n"
    "v_mfma_f32_16x16x32_bf16 a[168:171], v[72:75], v[20:23], a[168:171]\n"
    "ds_read_b64_tr_b16 v[100:101], v141 offset:32768\n"
    "v_mfma_f32_16x16x32_bf16 a[172:175], v[76:79], v[20:23], a[172:175]\n"
    "v_mfma_f32_16x16x32_bf16 a[136:139], v[72:75], v[16:19], a[136:139]\n"
    "ds_read_b64_tr_b16 v[102:103], v141 offset:33280\n"
    "v_mfma_f32_16x16x32_bf16 a[140:143], v[76:79], v[16:19], a[140:143]\n"
    "v_mfma_f32_16x16x32_bf16 a[104:107], v[72:75], v[12:15], a[104:107]\n"
    "ds_read_b64_tr_b16 v[104:105], v142 offset:32768\n"
    "v_mfma_f32_16x16x32_bf16 a[108:111], v[76:79], v[12:15], a[108:111]\n"
    "v_mfma_f32_16x16x32_bf16 a[72:75], v[72:75], v[8:11], a[72:75]\n"
    "ds_read_b64_tr_b16 v[106:107], v142 offset:33280\n"
    "v_mfma_f32_16x16x32_bf16 a[76:79], v[76:79], v[8:11], a[76:79]\n"
    "buffer_load_dwordx4 v135, s[44:47], s49 offen offset:3072 nt lds\n"
    "v_mfma_f32_16x16x32_bf16 a[40:43], v[72:75], v[4:7], a[40:43]\n"
    "ds_read_b64_tr_b16 v[108:109], v143 offset:32768\n"
    "v_mfma_f32_16x16x32_bf16 a[44:47], v[76:79], v[4:7], a[44:47]\n"
    "s_add_u32 s49, s49, 0x200000\n"
    "v_mfma_f32_16x16x32_bf16 a[8:11], v[72:75], v[0:3], a[8:11]\n"
    "ds_read_b64_tr_b16 v[110:111], v143 offset:33280\n"
    "v_mfma_f32_16x16x32_bf16 a[12:15], v[76:79], v[0:3], a[12:15]\n"
    "s_and_b32 s49, s49, 0xfffffff\n"
    "v_mfma_f32_16x16x32_bf16 a[16:19], v[80:83], v[0:3], a[16:19]\n"
    "ds_read_b64_tr_b16 v[112:113], v140 offset:36864\n"
    "v_mfma_f32_16x16x32_bf16 a[20:23], v[84:87], v[0:3], a[20:23]\n"
    "s_add_u32 s48, s78, 131840\n"
    "v_mfma_f32_16x16x32_bf16 a[48:51], v[80:83], v[4:7], a[48:51]\n"
    "ds_read_b64_tr_b16 v[114:115], v140 offset:37376\n"
    "s_add_u32 m0, s50, 98304\n"
    "v_mfma_f32_16x16x32_bf16 a[52:55], v[84:87], v[4:7], a[52:55]\n"
    "buffer_load_dwordx4 v154, s[40:43], s48 offen offset:0 lds\n"
    "v_mfma_f32_16x16x32_bf16 a[80:83], v[80:83], v[8:11], a[80:83]\n"
    "ds_read_b64_tr_b16 v[116:117], v141 offset:36864\n"
    "v_mfma_f32_16x16x32_bf16 a[84:87], v[84:87], v[8:11], a[84:87]\n"
    "v_mfma_f32_16x16x32_bf16 a[112:115], v[80:83], v[12:15], a[112:115]\n"
    "ds_read_b64_tr_b16 v[118:119], v141 offset:37376\n"
    "v_mfma_f32_16x16x32_bf16 a[116:119], v[84:87], v[12:15], a[116:119]\n"
    "v_mfma_f32_16x16x32_bf16 a[144:147], v[80:83], v[16:19], a[144:147]\n"
    "ds_read_b64_tr_b16 v[120:121], v142 offset:36864\n"
    "v_mfma_f32_16x16x32_bf16 a[148:151], v[84:87], v[16:19], a[148:151]\n"
    "v_mfma_f32_16x16x32_bf16 a[176:179], v[80:83], v[20:23], a[176:179]\n"
    "ds_read_b64_tr_b16 v[122:123], v142 offset:37376\n"
    "v_mfma_f32_16x16x32_bf16 a[180:183], v[84:87], v[20:23], a[180:183]\n"
    "buffer_load_dwordx4 v155, s[40:43], s48 offen offset:1024 lds\n"
    "v_mfma_f32_16x16x32_bf16 a[208:211], v[80:83], v[24:27], a[208:211]\n"
    "ds_read_b64_tr_b16 v[124:125], v143 offset:36864\n"
    "v_mfma_f32_16x16x32_bf16 a[212:215], v[84:87], v[24:27], a[212:215]\n"
    "v_mfma_f32_16x16x32_bf16 a[240:243], v[80:83], v[28:31], a[240:243]\n"
    "ds_read_b64_tr_b16 v[126:127], v143 offset:37376\n"
    "v_mfma_f32_16x16x32_bf16 a[244:247], v[84:87], v[28:31], a[244:247]\n"
    "v_mfma_f32_16x16x32_bf16 a[248:251], v[88:91], v[28:31], a[248:251]\n"
    "v_mfma_f32_16x16x32_bf16 a[252:255], v[92:95], v[28:31], a[252:255]\n"
    "v_mfma_f32_16x16x32_bf16 a[216:219], v[88:91], v[24:27], a[216:219]\n"
    "v_mfma_f32_16x16x32_bf16 a[220:223], v[92:95], v[24:27], a[220:223]\n"
    "buffer_load_dwordx4 v156, s[40:43], s48 offen offset:2048 lds\n"
    "v_mfma_f32_16x16x32_bf16 a[184:187], v[88:91], v[20:23], a[184:187]\n"
    "v_mfma_f32_16x16x32_bf16 a[188:191], v[92:95], v[20:23], a[188:191]\n"
    "v_mfma_f32_16x16x32_bf16 a[152:155], v[88:91], v[16:19], a[152:155]\n"
    "v_mfma_f32_16x16x32_bf16 a[156:159], v[92:95], v[16:19], a[156:159]\n"
    "v_mfma_f32_16x16x32_bf16 a[120:123], v[88:91], v[12:15], a[120:123]\n"
    "v_mfma_f32_16x16x32_bf16 a[124:127], v[92:95], v[12:15], a[124:127]\n"
    "v_mfma_f32_16x16x32_bf16 a[88:91], v[88:91], v[8:11], a[88:91]\n"
    "v_mfma_f32_16x16x32_bf16 a[92:95], v[92:95], v[8:11], a[92:95]\n"
    "buffer_load_dwordx4 v157, s[40:43], s48 offen offset:3072 lds\n"
    "v_mfma_f32_16x16x32_bf16 a[56:59], v[88:91], v[4:7], a[56:59]\n"
    "v_mfma_f32_16x16x32_bf16 a[60:63], v[92:95], v[4:7], a[60:63]\n"
    "v_mfma_f32_16x16x32_bf16 a[24:27], v[88:91], v[0:3], a[24:27]\n"
    "v_mfma_f32_16x16x32_bf16 a[28:31], v[92:95], v[0:3], a[28:31]\n"
    "s_waitcnt vmcnt(18) lgkmcnt(0)\n"
    "s_barrier\n"
    "v_mfma_f32_16x16x32_bf16 a[0:3], v[96:99], v[32:35], a[0:3]\n"
    "ds_read_b64_tr_b16 v[64:65], v144 offset:0\n"
    "v_mfma_f32_16x16x32_bf16 a[4:7], v[100:103], v[32:35], a[4:7]\n"
    "s_mov_b64 exec, s[70:71]\n"
    "global_load_dwordx4 v[176:179], v139, s[74:75] sc0 sc1\n"
    "s_mov_b64 exec, -1\n"
    "v_mfma_f32_16x16x32_bf16 a[32:35], v[96:99], v[36:39], a[32:35]\n"
    "ds_read_b64_tr_b16 v[66:67], v144 offset:512\n"
    "s_add_u32 m0, s50, 49152\n"
    "s_mov_b64 exec, s[30:31]\n"
    "global_store_dwordx4 v138, v[164:167], s[26:27] offset:1024 sc0 sc1\n"
    "s_mov_b64 exec, -1\n"
    "v_mfma_f32_16x16x32_bf16 a[36:39], v[100:103], v[36:39], a[36:39]\n"
    "buffer_load_dwordx4 v132, s[44:47], s49 offen offset:0 nt lds\n"
    "v_mfma_f32_16x16x32_bf16 a[64:67], v[96:99], v[40:43], a[64:67]\n"
    "ds_read_b128 v[0:3], v190 offset:0\n"
    "v_mfma_f32_16x16x32_bf16 a[68:71], v[100:103], v[40:43], a[68:71]\n"
    "v_mfma_f32_16x16x32_bf16 a[96:99], v[96:99], v[44:47], a[96:99]\n"
    "ds_read_b128 v[4:7], v190 offset:2048\n"
    "v_mfma_f32_16x16x32_bf16 a[100:103], v[100:103], v[44:47], a[100:103]\n"
    "v_mfma_f32_16x16x32_bf16 a[128:131], v[96:99], v[48:51], a[128:131]\n"
    "ds_read_b128 v[8:11], v190 offset:4096\n"
    "v_mfma_f32_16x16x32_bf16 a[132:135], v[100:103], v[48:51], a[132:135]\n"
    "v_mfma_f32_16x16x32_bf16 a[160:163], v[96:99], v[52:55], a[160:163]\n"
    "ds_read_b128 v[12:15], v190 offset:6144\n"
    "v_mfma_f32_16x16x32_bf16 a[164:167], v[100:103], v[52:55], a[164:167]\n"
    "buffer_load_dwordx4 v133, s[44:47], s49 offen offset:1024 nt lds\n"
    "v_mfma_f32_16x16x32_bf16 a[192:195], v[96:99], v[56:59], a[192:195]\n"
    "ds_read_b128 v[16:19], v190 offset:8192\n"
    "v_mfma_f32_16x16x32_bf16 a[196:199], v[100:103], v[56:59], a[196:199]\n"
    "v_mfma_f32_16x16x32_bf16 a[224:227], v[96:99], v[60:63], a[224:227]\n"
    "ds_read_b128 v[20:23], v190 offset:10240\n"
    "v_mfma_f32_16x16x32_bf16 a[228:231], v[100:103], v[60:63], a[228:231]\n"
    "v_mfma_f32_16x16x32_bf16 a[232:235], v[104:107], v[60:63], a[232:235]\n"
    "ds_read_b128 v[24:27], v190 offset:12288\n"
    "v_mfma_f32_16x16x32_bf16 a[236:239], v[108:111], v[60:63], a[236:239]\n"
    "v_mfma_f32_16x16x32_bf16 a[200:203], v[104:107], v[56:59], a[200:203]\n"
    "ds_read_b128 v[28:31], v190 offset:14336\n"
    "v_mfma_f32_16x16x32_bf16 a[204:207], v[108:111], v[56:59], a[204:207]\n"
    "buffer_load_dwordx4 v134, s[44:47], s49 offen offset:2048 nt lds\n"
    "v_mfma_f32_16x16x32_bf16 a[168:171], v[104:107], v[52:55], a[168:171]\n"
    "ds_read_b64_tr_b16 v[68:69], v145 offset:0\n"
    "v_mfma_f32_16x16x32_bf16 a[172:175], v[108:111], v[52:55], a[172:175]\n"
    "v_mfma_f32_16x16x32_bf16 a[136:139], v[104:107], v[48:51], a[136:139]\n"
    "ds_read_b64_tr_b16 v[70:71], v145 offset:512\n"
    "v_mfma_f32_16x16x32_bf16 a[140:143], v[108:111], v[48:51], a[140:143]\n"
    "v_mfma_f32_16x16x32_bf16 a[104:107], v[104:107], v[44:47], a[104:107]\n"
    "ds_read_b64_tr_b16 v[72:73], v146 offset:0\n"
    "v_mfma_f32_16x16x32_bf16 a[108:111], v[108:111], v[44:47], a[108:111]\n"
    "v_mfma_f32_16x16x32_bf16 a[72:75], v[104:107], v[40:43], a[72:75]\n"
    "ds_read_b64_tr_b16 v[74:75], v146 offset:512\n"
    "v_mfma_f32_16x16x32_bf16 a[76:79], v[108:111], v[40:43], a[76:79]\n"
    "buffer_load_dwordx4 v135, s[44:47], s49 offen offset:3072 nt lds\n"
    "v_mfma_f32_16x16x32_bf16 a[40:43], v[104:107], v[36:39], a[40:43]\n"
    "ds_read_b64_tr_b16 v[76:77], v147 offset:0\n"
    "v_mfma_f32_16x16x32_bf16 a[44:47], v[108:111], v[36:39], a[44:47]\n"
    "s_add_u32 s49, s49, 0x200000\n"
    "v_mfma_f32_16x16x32_bf16 a[8:11], v[104:107], v[32:35], a[8:11]\n"
    "ds_read_b64_tr_b16 v[78:79], v147 offset:512\n"
    "v_mfma_f32_16x16x32_bf16 a[12:15], v[108:111], v[32:35], a[12:15]\n"
    "s_and_b32 s49, s49, 0xfffffff\n"
    "v_mfma_f32_16x16x32_bf16 a[16:19], v[112:115], v[32:35], a[16:19]\n"
    "ds_read_b64_tr_b16 v[80:81], v144 offset:4096\n"
    "v_mfma_f32_16x16x32_bf16 a[20:23], v[116:119], v[32:35], a[20:23]\n"
    "s_add_u32 s48, s78, 896\n"
    "v_mfma_f32_16x16x32_bf16 a[48:51], v[112:115], v[36:39], a[48:51]\n"
    "ds_read_b64_tr_b16 v[82:83], v144 offset:4608\n"
    "s_add_u32 m0, s50, 131072\n"
    "v_mfma_f32_16x16x32_bf16 a[52:55], v[116:119], v[36:39], a[52:55]\n"
    "buffer_load_dwordx4 v154, s[40:43], s48 offen offset:0 lds\n"
    "v_mfma_f32_16x16x32_bf16 a[80:83], v[112:115], v[40:43], a[80:83]\n"
    "ds_read_b64_tr_b16 v[84:85], v145 offset:4096\n"
    "v_mfma_f32_16x16x32_bf16 a[84:87], v[116:119], v[40:43], a[84:87]\n"
    "v_mfma_f32_16x16x32_bf16 a[112:115], v[112:115], v[44:47], a[112:115]\n"
    "ds_read_b64_tr_b16 v[86:87], v145 offset:4608\n"
    "v_mfma_f32_16x16x32_bf16 a[116:119], v[116:119], v[44:47], a[116:119]\n"
    "v_mfma_f32_16x16x32_bf16 a[144:147], v[112:115], v[48:51], a[144:147]\n"
    "ds_read_b64_tr_b16 v[88:89], v146 offset:4096\n"
    "v_mfma_f32_16x16x32_bf16 a[148:151], v[116:119], v[48:51], a[148:151]\n"
    "v_mfma_f32_16x16x32_bf16 a[176:179], v[112:115], v[52:55], a[176:179]\n"
    "ds_read_b64_tr_b16 v[90:91], v146 offset:4608\n"
    "v_mfma_f32_16x16x32_bf16 a[180:183], v[116:119], v[52:55], a[180:183]\n"
    "buffer_load_dwordx4 v155, s[40:43], s48 offen offset:1024 lds\n"
    "v_mfma_f32_16x16x32_bf16 a[208:211], v[112:115], v[56:59], a[208:211]\n"
    "ds_read_b64_tr_b16 v[92:93], v147 offset:4096\n"
    "v_mfma_f32_16x16x32_bf16 a[212:215], v[116:119], v[56:59], a[212:215]\n"
    "v_mfma_f32_16x16x32_bf16 a[240:243], v[112:115], v[60:63], a[240:243]\n"
    "ds_read_b64_tr_b16 v[94:95], v147 offset:4608\n"
    "v_mfma_f32_16x16x32_bf16 a[244:247], v[116:119], v[60:63], a[244:247]\n"
    "v_mfma_f32_16x16x32_bf16 a[248:251], v[120:123], v[60:63], a[248:251]\n"
    "v_mfma_f32_16x16x32_bf16 a[252:255], v[124:127], v[60:63], a[252:255]\n"
    "v_mfma_f32_16x16x32_bf16 a[216:219], v[120:123], v[56:59], a[216:219]\n"
    "v_mfma_f32_16x16x32_bf16 a[220:223], v[124:127], v[56:59], a[220:223]\n"
    "buffer_load_dwordx4 v156, s[40:43], s48 offen offset:2048 lds\n"
    "v_mfma_f32_16x16x32_bf16 a[184:187], v[120:123], v[52:55], a[184:187]\n"
    "v_mfma_f32_16x16x32_bf16 a[188:191], v[124:127], v[52:55], a[188:191]\n"
    "v_mfma_f32_16x16x32_bf16 a[152:155], v[120:123], v[48:51], a[152:155]\n"
    "s_memrealtime s[28:29]\n"
    "v_mfma_f32_16x16x32_bf16 a[156:159], v[124:127], v[48:51], a[156:159]\n"
    "v_mfma_f32_16x16x32_bf16 a[120:123], v[120:123], v[44:47], a[120:123]\n"
    "v_mfma_f32_16x16x32_bf16 a[124:127], v[124:127], v[44:47], a[124:127]\n"
    "v_mfma_f32_16x16x32_bf16 a[88:91], v[120:123], v[40:43], a[88:91]\n"
    "v_mfma_f32_16x16x32_bf16 a[92:95], v[124:127], v[40:43], a[92:95]\n"
    "buffer_load_dwordx4 v157, s[40:43], s48 offen offset:3072 lds\n"
    "v_mfma_f32_16x16x32_bf16 a[56:59], v[120:123], v[36:39], a[56:59]\n"
    "v_mfma_f32_16x16x32_bf16 a[60:63], v[124:127], v[36:39], a[60:63]\n"
    "v_mfma_f32_16x16x32_bf16 a[24:27], v[120:123], v[32:35], a[24:27]\n"
    "v_mfma_f32_16x16x32_bf16 a[28:31], v[124:127], v[32:35], a[28:31]\n"
    "s_waitcnt vmcnt(18) lgkmcnt(0)\n"
    "s_barrier\n"
    "v_mfma_f32_16x16x32_bf16 a[0:3], v[64:67], v[0:3], a[0:3]\n"
    "ds_read_b64_tr_b16 v[96:97], v144 offset:32768\n"
    "v_mfma_f32_16x16x32_bf16 a[4:7], v[68:71], v[0:3], a[4:7]\n"
    "v_mfma_f32_16x16x32_bf16 a[32:35], v[64:67], v[4:7], a[32:35]\n"
    "ds_read_b64_tr_b16 v[98:99], v144 offset:33280\n"
    "s_add_u32 m0, s50, 81920\n"
    "s_mov_b64 exec, s[30:31]\n"
    "global_store_dwordx4 v138, v[168:171], s[26:27] offset:2048 sc0 sc1\n"
    "s_mov_b64 exec, -1\n"
    "v_mfma_f32_16x16x32_bf16 a[36:39], v[68:71], v[4:7], a[36:39]\n"
    "buffer_load_dwordx4 v132, s[44:47], s49 offen offset:0 nt lds\n"
    "v_mfma_f32_16x16x32_bf16 a[64:67], v[64:67], v[8:11], a[64:67]\n"
    "ds_read_b128 v[32:35], v191 offset:0\n"
    "v_mfma_f32_16x16x32_bf16 a[68:71], v[68:71], v[8:11], a[68:71]\n"
    "s_lshr_b32 s54, s24, 4\n"
    "s_add_u32 s33, s33, s54\n"
    "v_mfma_f32_16x16x32_bf16 a[96:99], v[64:67], v[12:15], a[96:99]\n"
    "ds_read_b128 v[36:39], v191 offset:2048\n"
    "s_add_u32 s32, s32, s54\n"
    "v_mfma_f32_16x16x32_bf16 a[100:103], v[68:71], v[12:15], a[100:103]\n"
    "s_cmp_eq_u32 s73, 2\n"
    "s_cselect_b64 exec, 1, 0\n"
    "global_store_dword v148, v152, s[64:65] sc0 sc1\n"
    "s_mov_b64 exec, -1\n"
    "s_sub_u32 s54, s33, s28\n"
    "s_max_i32 s54, s54, -250\n"
    "v_mfma_f32_16x16x32_bf16 a[128:131], v[64:67], v[16:19], a[128:131]\n"
    "ds_read_b128 v[40:43], v191 offset:4096\n"
    "v_mfma_f32_16x16x32_bf16 a[132:135], v[68:71], v[16:19], a[132:135]\n"
    "s_add_u32 s33, s28, s54\n"
    "s_cmp_gt_i32 s54, 0\n"
    "v_mfma_f32_16x16x32_bf16 a[160:163], v[64:67], v[20:23], a[160:163]\n"
    "ds_read_b128 v[44:47], v191 offset:6144\n"
    "s_cbranch_scc1 PACEI22_%=\n"
    "PACEBI22_%=:\n"
    "v_mfma_f32_16x16x32_bf16 a[164:167], v[68:71], v[20:23], a[164:167]\n"
    "buffer_load_dwordx4 v133, s[44:47], s49 offen offset:1024 nt lds\n"
    "v_mfma_f32_16x16x32_bf16 a[192:195], v[64:67], v[24:27], a[192:195]\n"
    "ds_read_b128 v[48:51], v191 offset:8192\n"
    "v_mfma_f32_16x16x32_bf16 a[196:199], v[68:71], v[24:27], a[196:199]\n"
    "s_sub_u32 s24, s24, 16\n"
    "s_max_i32 s24, s24, s35\n"
    "v_mfma_f32_16x16x32_bf16 a[224:227], v[64:67], v[28:31], a[224:227]\n"
    "ds_read_b128 v[52:55], v191 offset:10240\n"
    "v_mfma_f32_16x16x32_bf16 a[228:231], v[68:71], v[28:31], a[228:231]\n"
    "v_mfma_f32_16x16x32_bf16 a[232:235], v[72:75], v[28:31], a[232:235]\n"
    "ds_read_b128 v[56:59], v191 offset:12288\n"
    "v_mfma_f32_16x16x32_bf16 a[236:239], v[76:79], v[28:31], a[236:239]\n"
    "v_mfma_f32_16x16x32_bf16 a[200:203], v[72:75], v[24:27], a[200:203]\n"
    "ds_read_b128 v[60:63], v191 offset:14336\n"
    "v_mfma_f32_16x16x32_bf16 a[204:207], v[76:79], v[24:27], a[204:207]\n"
    "buffer_load_dwordx4 v134, s[44:47], s49 offen offset:2048 nt lds\n"
    "v_mfma_f32_16x16x32_bf16 a[168:171], v[72:75], v[20:23], a[168:171]\n"
    "ds_read_b64_tr_b16 v[100:101], v145 offset:32768\n"
    "v_mfma_f32_16x16x32_bf16 a[172:175], v[76:79], v[20:23], a[172:175]\n"
    "v_mfma_f32_16x16x32_bf16 a[136:139], v[72:75], v[16:19], a[136:139]\n"
    "ds_read_b64_tr_b16 v[102:103], v145 offset:33280\n"
    "v_mfma_f32_16x16x32_bf16 a[140:143], v[76:79], v[16:19], a[140:143]\n"
    "v_mfma_f32_16x16x32_bf16 a[104:107], v[72:75], v[12:15], a[104:107]\n"
    "ds_read_b64_tr_b16 v[104:105], v146 offset:32768\n"
    "v_mfma_f32_16x16x32_bf16 a[108:111], v[76:79], v[12:15], a[108:111]\n"
    "v_mfma_f32_16x16x32_bf16 a[72:75], v[72:75], v[8:11], a[72:75]\n"
    "ds_read_b64_tr_b16 v[106:107], v146 offset:33280\n"
    "v_mfma_f32_16x16x32_bf16 a[76:79], v[76:79], v[8:11], a[76:79]\n"
    "buffer_load_dwordx4 v135, s[44:47], s49 offen offset:3072 nt lds\n"
    "v_mfma_f32_16x16x32_bf16 a[40:43], v[72:75], v[4:7], a[40:43]\n"
    "ds_read_b64_tr_b16 v[108:109], v147 offset:32768\n"
    "v_mfma_f32_16x16x32_bf16 a[44:47], v[76:79], v[4:7], a[44:47]\n"
    "s_add_u32 s49, s49, 0x200000\n"
    "v_mfma_f32_16x16x32_bf16 a[8:11], v[72:75], v[0:3], a[8:11]\n"
    "ds_read_b64_tr_b16 v[110:111], v147 offset:33280\n"
    "v_mfma_f32_16x16x32_bf16 a[12:15], v[76:79], v[0:3], a[12:15]\n"
    "s_and_b32 s49, s49, 0xfffffff\n"
    "v_mfma_f32_16x16x32_bf16 a[16:19], v[80:83], v[0:3], a[16:19]\n"
    "ds_read_b64_tr_b16 v[112:113], v144 offset:36864\n"
    "v_mfma_f32_16x16x32_bf16 a[20:23], v[84:87], v[0:3], a[20:23]\n"
    "s_add_u32 s48, s78, 131968\n"
    "v_mfma_f32_16x16x32_bf16 a[48:51], v[80:83], v[4:7], a[48:51]\n"
    "ds_read_b64_tr_b16 v[114:115], v144 offset:37376\n"
    "s_add_u32 m0, s50, 147456\n"
    "v_mfma_f32_16x16x32_bf16 a[52:55], v[84:87], v[4:7], a[52:55]\n"
    "buffer_load_dwordx4 v154, s[40:43], s48 offen offset:0 lds\n"
    "v_mfma_f32_16x16x32_bf16 a[80:83], v[80:83], v[8:11], a[80:83]\n"
    "ds_read_b64_tr_b16 v[116:117], v145 offset:36864\n"
    "v_mfma_f32_16x16x32_bf16 a[84:87], v[84:87], v[8:11], a[84:87]\n"
    "v_mfma_f32_16x16x32_bf16 a[112:115], v[80:83], v[12:15], a[112:115]\n"
    "ds_read_b64_tr_b16 v[118:119], v145 offset:37376\n"
    "v_mfma_f32_16x16x32_bf16 a[116:119], v[84:87], v[12:15], a[116:119]\n"
    "v_mfma_f32_16x16x32_bf16 a[144:147], v[80:83], v[16:19], a[144:147]\n"
    "ds_read_b64_tr_b16 v[120:121], v146 offset:36864\n"
    "v_mfma_f32_16x16x32_bf16 a[148:151], v[84:87], v[16:19], a[148:151]\n"
    "v_mfma_f32_16x16x32_bf16 a[176:179], v[80:83], v[20:23], a[176:179]\n"
    "ds_read_b64_tr_b16 v[122:123], v146 offset:37376\n"
    "v_mfma_f32_16x16x32_bf16 a[180:183], v[84:87], v[20:23], a[180:183]\n"
    "buffer_load_dwordx4 v155, s[40:43], s48 offen offset:1024 lds\n"
    "v_mfma_f32_16x16x32_bf16 a[208:211], v[80:83], v[24:27], a[208:211]\n"
    "ds_read_b64_tr_b16 v[124:125], v147 offset:36864\n"
    "v_mfma_f32_16x16x32_bf16 a[212:215], v[84:87], v[24:27], a[212:215]\n"
    "v_mfma_f32_16x16x32_bf16 a[240:243], v[80:83], v[28:31], a[240:243]\n"
    "ds_read_b64_tr_b16 v[126:127], v147 offset:37376\n"
    "v_mfma_f32_16x16x32_bf16 a[244:247], v[84:87], v[28:31], a[244:247]\n"
    "v_mfma_f32_16x16x32_bf16 a[248:251], v[88:91], v[28:31], a[248:251]\n"
    "v_mfma_f32_16x16x32_bf16 a[252:255], v[92:95], v[28:31], a[252:255]\n"
    "v_mfma_f32_16x16x32_bf16 a[216:219], v[88:91], v[24:27], a[216:219]\n"
    "v_mfma_f32_16x16x32_bf16 a[220:223], v[92:95], v[24:27], a[220:223]\n"
    "buffer_load_dwordx4 v156, s[40:43], s48 offen offset:2048 lds\n"
    "v_mfma_f32_16x16x32_bf16 a[184:187], v[88:91], v[20:23], a[184:187]\n"
    "v_mfma_f32_16x16x32_bf16 a[188:191], v[92:95], v[20:23], a[188:191]\n"
    "v_mfma_f32_16x16x32_bf16 a[152:155], v[88:91], v[16:19], a[152:155]\n"
    "v_mfma_f32_16x16x32_bf16 a[156:159], v[92:95], v[16:19], a[156:159]\n"
    "v_mfma_f32_16x16x32_bf16 a[120:123], v[88:91], v[12:15], a[120:123]\n"
    "v_mfma_f32_16x16x32_bf16 a[124:127], v[92:95], v[12:15], a[124:127]\n"
    "v_mfma_f32_16x16x32_bf16 a[88:91], v[88:91], v[8:11], a[88:91]\n"
    "v_mfma_f32_16x16x32_bf16 a[92:95], v[92:95], v[8:11], a[92:95]\n"
    "buffer_load_dwordx4 v157, s[40:43], s48 offen offset:3072 lds\n"
    "v_mfma_f32_16x16x32_bf16 a[56:59], v[88:91], v[4:7], a[56:59]\n"
    "v_mfma_f32_16x16x32_bf16 a[60:63], v[92:95], v[4:7], a[60:63]\n"
    "v_mfma_f32_16x16x32_bf16 a[24:27], v[88:91], v[0:3], a[24:27]\n"
    "v_mfma_f32_16x16x32_bf16 a[28:31], v[92:95], v[0:3], a[28:31]\n"
    "s_waitcnt vmcnt(18) lgkmcnt(0)\n"
    "s_barrier\n"
    "v_mfma_f32_16x16x32_bf16 a[0:3], v[96:99], v[32:35], a[0:3]\n"
    "ds_read_b64_tr_b16 v[64:65], v140 offset:0\n"
    "v_mfma_f32_16x16x32_bf16 a[4:7], v[100:103], v[32:35], a[4:7]\n"
    "v_min_u32 v176, v176, v177\n"
    "v_mfma_f32_16x16x32_bf16 a[32:35], v[96:99], v[36:39], a[32:35]\n"
    "ds_read_b64_tr_b16 v[66:67], v140 offset:512\n"
    "s_add_u32 m0, s50, 114688\n"
    "s_mov_b64 exec, s[30:31]\n"
    "global_store_dwordx4 v138, v[172:175], s[26:27] offset:3072 sc0 sc1\n"
    "s_mov_b64 exec, -1\n"
    "v_mfma_f32_16x16x32_bf16 a[36:39], v[100:103], v[36:39], a[36:39]\n"
    "buffer_load_dwordx4 v132, s[44:47], s49 offen offset:0 nt lds\n"
    "v_mfma_f32_16x16x32_bf16 a[64:67], v[96:99], v[40:43], a[64:67]\n"
    "ds_read_b128 v[0:3], v192 offset:0\n"
    "v_mfma_f32_16x16x32_bf16 a[68:71], v[100:103], v[40:43], a[68:71]\n"
    "v_min3_u32 v176, v176, v178, v179\n"
    "v_mfma_f32_16x16x32_bf16 a[96:99], v[96:99], v[44:47], a[96:99]\n"
    "ds_read_b128 v[4:7], v192 offset:2048\n"
    "v_mfma_f32_16x16x32_bf16 a[100:103], v[100:103], v[44:47], a[100:103]\n"
    "v_cmp_le_u32 vcc, s68, v176\n"
    "v_mfma_f32_16x16x32_bf16 a[128:131], v[96:99], v[48:51], a[128:131]\n"
    "ds_read_b128 v[8:11], v192 offset:4096\n"
    "v_mfma_f32_16x16x32_bf16 a[132:135], v[100:103], v[48:51], a[132:135]\n"
    "v_mfma_f32_16x16x32_bf16 a[160:163], v[96:99], v[52:55], a[160:163]\n"
    "ds_read_b128 v[12:15], v192 offset:6144\n"
    "v_mfma_f32_16x16x32_bf16 a[164:167], v[100:103], v[52:55], a[164:167]\n"
    "buffer_load_dwordx4 v133, s[44:47], s49 offen offset:1024 nt lds\n"
    "v_mfma_f32_16x16x32_bf16 a[192:195], v[96:99], v[56:59], a[192:195]\n"
    "ds_read_b128 v[16:19], v192 offset:8192\n"
    "v_mfma_f32_16x16x32_bf16 a[196:199], v[100:103], v[56:59], a[196:199]\n"
    "s_and_b32 s54, vcc_lo, s70\n"
    "s_or_b32 s69, s69, s54\n"
    "v_mfma_f32_16x16x32_bf16 a[224:227], v[96:99], v[60:63], a[224:227]\n"
    "ds_read_b128 v[20:23], v192 offset:10240\n"
    "v_mfma_f32_16x16x32_bf16 a[228:231], v[100:103], v[60:63], a[228:231]\n"
    "s_andn2_b32 s70, s70, s54\n"
    "s_bitcmp1_b32 s69, s72\n"
    "v_mfma_f32_16x16x32_bf16 a[232:235], v[104:107], v[60:63], a[232:235]\n"
    "ds_read_b128 v[24:27], v192 offset:12288\n"
    "v_mfma_f32_16x16x32_bf16 a[236:239], v[108:111], v[60:63], a[236:239]\n"
    "s_cbranch_scc0 SLOWI2_%=\n"
    "SLOWBACKI2_%=:\n"
    "s_add_u32 s72, s72, 1\n"
    "v_mfma_f32_16x16x32_bf16 a[200:203], v[104:107], v[56:59], a[200:203]\n"
    "ds_read_b128 v[28:31], v192 offset:14336\n"
    "v_mfma_f32_16x16x32_bf16 a[204:207], v[108:111], v[56:59], a[204:207]\n"
    "buffer_load_dwordx4 v134, s[44:47], s49 offen offset:2048 nt lds\n"
    "v_mfma_f32_16x16x32_bf16 a[168:171], v[104:107], v[52:55], a[168:171]\n"
    "ds_read_b64_tr_b16 v[68:69], v141 offset:0\n"
    "v_mfma_f32_16x16x32_bf16 a[172:175], v[108:111], v[52:55], a[172:175]\n"
    "v_mfma_f32_16x16x32_bf16 a[136:139], v[104:107], v[48:51], a[136:139]\n"
    "ds_read_b64_tr_b16 v[70:71], v141 offset:512\n"
    "v_mfma_f32_16x16x32_bf16 a[140:143], v[108:111], v[48:51], a[140:143]\n"
    "v_mfma_f32_16x16x32_bf16 a[104:107], v[104:107], v[44:47], a[104:107]\n"
    "ds_read_b64_tr_b16 v[72:73], v142 offset:0\n"
    "v_mfma_f32_16x16x32_bf16 a[108:111], v[108:111], v[44:47], a[108:111]\n"
    "v_mfma_f32_16x16x32_bf16 a[72:75], v[104:107], v[40:43], a[72:75]\n"
    "ds_read_b64_tr_b16 v[74:75], v142 offset:512\n"
    "v_mfma_f32_16x16x32_bf16 a[76:79], v[108:111], v[40:43], a[76:79]\n"
    "buffer_load_dwordx4 v135, s[44:47], s49 offen offset:3072 nt lds\n"
    "v_mfma_f32_16x16x32_bf16 a[40:43], v[104:107], v[36:39], a[40:43]\n"
    "ds_read_b64_tr_b16 v[76:77], v143 offset:0\n"
    "v_mfma_f32_16x16x32_bf16 a[44:47], v[108:111], v[36:39], a[44:47]\n"
    "s_add_u32 s49, s49, 0x200000\n"
    "v_mfma_f32_16x16x32_bf16 a[8:11], v[104:107], v[32:35], a[8:11]\n"
    "ds_read_b64_tr_b16 v[78:79], v143 offset:512\n"
    "v_mfma_f32_16x16x32_bf16 a[12:15], v[108:111], v[32:35], a[12:15]\n"
    "s_and_b32 s49, s49, 0xfffffff\n"
    "v_mfma_f32_16x16x32_bf16 a[16:19], v[112:115], v[32:35], a[16:19]\n"
    "ds_read_b64_tr_b16 v[80:81], v140 offset:4096\n"
    "v_mfma_f32_16x16x32_bf16 a[20:23], v[116:119], v[32:35], a[20:23]\n"
    "v_mfma_f32_16x16x32_bf16 a[48:51], v[112:115], v[36:39], a[48:51]\n"
    "ds_read_b64_tr_b16 v[82:83], v140 offset:4608\n"
    "v_mfma_f32_16x16x32_bf16 a[52:55], v[116:119], v[36:39], a[52:55]\n"
    "v_mfma_f32_16x16x32_bf16 a[80:83], v[112:115], v[40:43], a[80:83]\n"
    "ds_read_b64_tr_b16 v[84:85], v141 offset:4096\n"
    "v_mfma_f32_16x16x32_bf16 a[84:87], v[116:119], v[40:43], a[84:87]\n"
    "v_mfma_f32_16x16x32_bf16 a[112:115], v[112:115], v[44:47], a[112:115]\n"
    "ds_read_b64_tr_b16 v[86:87], v141 offset:4608\n"
    "v_mfma_f32_16x16x32_bf16 a[116:119], v[116:119], v[44:47], a[116:119]\n"
    "v_mfma_f32_16x16x32_bf16 a[144:147], v[112:115], v[48:51], a[144:147]\n"
    "ds_read_b64_tr_b16 v[88:89], v142 offset:4096\n"
    "v_mfma_f32_16x16x32_bf16 a[148:151], v[116:119], v[48:51], a[148:151]\n"
    "v_mfma_f32_16x16x32_bf16 a[176:179], v[112:115], v[52:55], a[176:179]\n"
    "ds_read_b64_tr_b16 v[90:91], v142 offset:4608\n"
    "v_mfma_f32_16x16x32_bf16 a[180:183], v[116:119], v[52:55], a[180:183]\n"
    "v_mfma_f32_16x16x32_bf16 a[208:211], v[112:115], v[56:59], a[208:211]\n"
    "ds_read_b64_tr_b16 v[92:93], v143 offset:4096\n"
    "v_mfma_f32_16x16x32_bf16 a[212:215], v[116:119], v[56:59], a[212:215]\n"
    "v_mfma_f32_16x16x32_bf16 a[240:243], v[112:115], v[60:63], a[240:243]\n"
    "ds_read_b64_tr_b16 v[94:95], v143 offset:4608\n"
    "v_mfma_f32_16x16x32_bf16 a[244:247], v[116:119], v[60:63], a[244:247]\n"
    "v_mfma_f32_16x16x32_bf16 a[248:251], v[120:123], v[60:63], a[248:251]\n"
    "v_mfma_f32_16x16x32_bf16 a[252:255], v[124:127], v[60:63], a[252:255]\n"
    "v_mfma_f32_16x16x32_bf16 a[216:219], v[120:123], v[56:59], a[216:219]\n"
    "v_mfma_f32_16x16x32_bf16 a[220:223], v[124:127], v[56:59], a[220:223]\n"
    "v_mfma_f32_16x16x32_bf16 a[184:187], v[120:123], v[52:55], a[184:187]\n"
    "v_mfma_f32_16x16x32_bf16 a[188:191], v[124:127], v[52:55], a[188:191]\n"
    "v_mfma_f32_16x16x32_bf16 a[152:155], v[120:123], v[48:51], a[152:155]\n"
    "s_memrealtime s[28:29]\n"
    "v_mfma_f32_16x16x32_bf16 a[156:159], v[124:127], v[48:51], a[156:159]\n"
    "v_mfma_f32_16x16x32_bf16 a[120:123], v[120:123], v[44:47], a[120:123]\n"
    "v_mfma_f32_16x16x32_bf16 a[124:127], v[124:127], v[44:47], a[124:127]\n"
    "v_mfma_f32_16x16x32_bf16 a[88:91], v[120:123], v[40:43], a[88:91]\n"
    "v_mfma_f32_16x16x32_bf16 a[92:95], v[124:127], v[40:43], a[92:95]\n"
    "v_mfma_f32_16x16x32_bf16 a[56:59], v[120:123], v[36:39], a[56:59]\n"
    "v_mfma_f32_16x16x32_bf16 a[60:63], v[124:127], v[36:39], a[60:63]\n"
    "v_mfma_f32_16x16x32_bf16 a[24:27], v[120:123], v[32:35], a[24:27]\n"
    "v_mfma_f32_16x16x32_bf16 a[28:31], v[124:127], v[32:35], a[28:31]\n"
    "s_waitcnt vmcnt(14) lgkmcnt(0)\n"
    "s_barrier\n"
    "v_mfma_f32_16x16x32_bf16 a[0:3], v[64:67], v[0:3], a[0:3]\n"
    "ds_read_b64_tr_b16 v[96:97], v140 offset:32768\n"
    "v_mfma_f32_16x16x32_bf16 a[4:7], v[68:71], v[0:3], a[4:7]\n"
    "s_cmp_eq_u32 s66, 3\n"
    "s_cselect_b64 s[30:31], -1, 1\n"
    "s_cselect_b64 s[26:27], s[60:61], s[62:63]\n"
    "v_mfma_f32_16x16x32_bf16 a[32:35], v[64:67], v[4:7], a[32:35]\n"
    "ds_read_b64_tr_b16 v[98:99], v140 offset:33280\n"
    "s_add_u32 m0, s50, 16384\n"
    "s_mov_b64 exec, s[30:31]\n"
    "global_store_dwordx4 v138, v[160:163], s[26:27] offset:0 sc0 sc1\n"
    "s_mov_b64 exec, -1\n"
    "v_mfma_f32_16x16x32_bf16 a[36:39], v[68:71], v[4:7], a[36:39]\n"
    "buffer_load_dwordx4 v132, s[44:47], s49 offen offset:0 nt lds\n"
    "v_mfma_f32_16x16x32_bf16 a[64:67], v[64:67], v[8:11], a[64:67]\n"
    "ds_read_b128 v[32:35], v193 offset:0\n"
    "v_mfma_f32_16x16x32_bf16 a[68:71], v[68:71], v[8:11], a[68:71]\n"
    "s_lshr_b32 s54, s24, 4\n"
    "s_add_u32 s33, s33, s54\n"
    "v_mfma_f32_16x16x32_bf16 a[96:99], v[64:67], v[12:15], a[96:99]\n"
    "ds_read_b128 v[36:39], v193 offset:2048\n"
    "s_add_u32 s32, s32, s54\n"
    "v_mfma_f32_16x16x32_bf16 a[100:103], v[68:71], v[12:15], a[100:103]\n"
    "s_sub_u32 s54, s33, s28\n"
    "s_max_i32 s54, s54, -250\n"
    "v_mfma_f32_16x16x32_bf16 a[128:131], v[64:67], v[16:19], a[128:131]\n"
    "ds_read_b128 v[40:43], v193 offset:4096\n"
    "v_mfma_f32_16x16x32_bf16 a[132:135], v[68:71], v[16:19], a[132:135]\n"
    "s_add_u32 s33, s28, s54\n"
    "s_cmp_gt_i32 s54, 0\n"
    "v_mfma_f32_16x16x32_bf16 a[160:163], v[64:67], v[20:23], a[160:163]\n"
    "ds_read_b128 v[44:47], v193 offset:6144\n"
    "s_cbranch_scc1 PACEI30_%=\n"
    "PACEBI30_%=:\n"
    "v_mfma_f32_16x16x32_bf16 a[164:167], v[68:71], v[20:23], a[164:167]\n"
    "buffer_load_dwordx4 v133, s[44:47], s49 offen offset:1024 nt lds\n"
    "v_mfma_f32_16x16x32_bf16 a[192:195], v[64:67], v[24:27], a[192:195]\n"
    "ds_read_b128 v[48:51], v193 offset:8192\n"
    "v_mfma_f32_16x16x32_bf16 a[196:199], v[68:71], v[24:27], a[196:199]\n"
    "s_sub_u32 s24, s24, 16\n"
    "s_max_i32 s24, s24, s35\n"
    "v_mfma_f32_16x16x32_bf16 a[224:227], v[64:67], v[28:31], a[224:227]\n"
    "ds_read_b128 v[52:55], v193 offset:10240\n"
    "v_mfma_f32_16x16x32_bf16 a[228:231], v[68:71], v[28:31], a[228:231]\n"
    "v_mfma_f32_16x16x32_bf16 a[232:235], v[72:75], v[28:31], a[232:235]\n"
    "ds_read_b128 v[56:59], v193 offset:12288\n"
    "v_mfma_f32_16x16x32_bf16 a[236:239], v[76:79], v[28:31], a[236:239]\n"
    "v_mfma_f32_16x16x32_bf16 a[200:203], v[72:75], v[24:27], a[200:203]\n"
    "ds_read_b128 v[60:63], v193 offset:14336\n"
    "v_mfma_f32_16x16x32_bf16 a[204:207], v[76:79], v[24:27], a[204:207]\n"
    "buffer_load_dwordx4 v134, s[44:47], s49 offen offset:2048 nt lds\n"
    "v_mfma_f32_16x16x32_bf16 a[168:171], v[72:75], v[20:23], a[168:171]\n"
    "ds_read_b64_tr_b16 v[100:101], v141 offset:32768\n"
    "v_mfma_f32_16x16x32_bf16 a[172:175], v[76:79], v[20:23], a[172:175]\n"
    "v_mfma_f32_16x16x32_bf16 a[136:139], v[72:75], v[16:19], a[136:139]\n"
    "ds_read_b64_tr_b16 v[102:103], v141 offset:33280\n"
    "v_mfma_f32_16x16x32_bf16 a[140:143], v[76:79], v[16:19], a[140:143]\n"
    "v_mfma_f32_16x16x32_bf16 a[104:107], v[72:75], v[12:15], a[104:107]\n"
    "ds_read_b64_tr_b16 v[104:105], v142 offset:32768\n"
    "v_mfma_f32_16x16x32_bf16 a[108:111], v[76:79], v[12:15], a[108:111]\n"
    "v_mfma_f32_16x16x32_bf16 a[72:75], v[72:75], v[8:11], a[72:75]\n"
    "ds_read_b64_tr_b16 v[106:107], v142 offset:33280\n"
    "v_mfma_f32_16x16x32_bf16 a[76:79], v[76:79], v[8:11], a[76:79]\n"
    "buffer_load_dwordx4 v135, s[44:47], s49 offen offset:3072 nt lds\n"
    "v_mfma_f32_16x16x32_bf16 a[40:43], v[72:75], v[4:7], a[40:43]\n"
    "ds_read_b64_tr_b16 v[108:109], v143 offset:32768\n"
    "v_mfma_f32_16x16x32_bf16 a[44:47], v[76:79], v[4:7], a[44:47]\n"
    "s_add_u32 s53, s53, 0x24924925\n"
    "s_cselect_b32 s55, s79, s80\n"
    "v_mfma_f32_16x16x32_bf16 a[8:11], v[72:75], v[0:3], a[8:11]\n"
    "ds_read_b64_tr_b16 v[110:111], v143 offset:33280\n"
    "v_mfma_f32_16x16x32_bf16 a[12:15], v[76:79], v[0:3], a[12:15]\n"
    "s_add_u32 s49, s49, s55\n"
    "v_mfma_f32_16x16x32_bf16 a[16:19], v[80:83], v[0:3], a[16:19]\n"
    "ds_read_b64_tr_b16 v[112:113], v140 offset:36864\n"
    "v_mfma_f32_16x16x32_bf16 a[20:23], v[84:87], v[0:3], a[20:23]\n"
    "s_and_b32 s49, s49, 0xfffffff\n"
    "v_mfma_f32_16x16x32_bf16 a[48:51], v[80:83], v[4:7], a[48:51]\n"
    "ds_read_b64_tr_b16 v[114:115], v140 offset:37376\n"
    "s_add_u32 m0, s50, 0\n"
    "v_mfma_f32_16x16x32_bf16 a[52:55], v[84:87], v[4:7], a[52:55]\n"
    "buffer_load_dwordx4 v138, s[56:59], s52 offen offset:0 sc0 sc1 lds\n"
    "v_mfma_f32_16x16x32_bf16 a[80:83], v[80:83], v[8:11], a[80:83]\n"
    "ds_read_b64_tr_b16 v[116:117], v141 offset:36864\n"
    "v_mfma_f32_16x16x32_bf16 a[84:87], v[84:87], v[8:11], a[84:87]\n"
    "v_mfma_f32_16x16x32_bf16 a[112:115], v[80:83], v[12:15], a[112:115]\n"
    "ds_read_b64_tr_b16 v[118:119], v141 offset:37376\n"
    "v_mfma_f32_16x16x32_bf16 a[116:119], v[84:87], v[12:15], a[116:119]\n"
    "v_mfma_f32_16x16x32_bf16 a[144:147], v[80:83], v[16:19], a[144:147]\n"
    "ds_read_b64_tr_b16 v[120:121], v142 offset:36864\n"
    "v_mfma_f32_16x16x32_bf16 a[148:151], v[84:87], v[16:19], a[148:151]\n"
    "v_mfma_f32_16x16x32_bf16 a[176:179], v[80:83], v[20:23], a[176:179]\n"
    "ds_read_b64_tr_b16 v[122:123], v142 offset:37376\n"
    "v_mfma_f32_16x16x32_bf16 a[180:183], v[84:87], v[20:23], a[180:183]\n"
    "buffer_load_dwordx4 v138, s[56:59], s52 offen offset:1024 sc0 sc1 lds\n"
    "v_mfma_f32_16x16x32_bf16 a[208:211], v[80:83], v[24:27], a[208:211]\n"
    "ds_read_b64_tr_b16 v[124:125], v143 offset:36864\n"
    "v_mfma_f32_16x16x32_bf16 a[212:215], v[84:87], v[24:27], a[212:215]\n"
    "v_mfma_f32_16x16x32_bf16 a[240:243], v[80:83], v[28:31], a[240:243]\n"
    "ds_read_b64_tr_b16 v[126:127], v143 offset:37376\n"
    "v_mfma_f32_16x16x32_bf16 a[244:247], v[84:87], v[28:31], a[244:247]\n"
    "v_mfma_f32_16x16x32_bf16 a[248:251], v[88:91], v[28:31], a[248:251]\n"
    "v_mfma_f32_16x16x32_bf16 a[252:255], v[92:95], v[28:31], a[252:255]\n"
    "v_mfma_f32_16x16x32_bf16 a[216:219], v[88:91], v[24:27], a[216:219]\n"
    "v_mfma_f32_16x16x32_bf16 a[220:223], v[92:95], v[24:27], a[220:223]\n"
    "buffer_load_dwordx4 v138, s[56:59], s52 offen offset:2048 sc0 sc1 lds\n"
    "v_mfma_f32_16x16x32_bf16 a[184:187], v[88:91], v[20:23], a[184:187]\n"
    "v_mfma_f32_16x16x32_bf16 a[188:191], v[92:95], v[20:23], a[188:191]\n"
    "v_mfma_f32_16x16x32_bf16 a[152:155], v[88:91], v[16:19], a[152:155]\n"
    "v_mfma_f32_16x16x32_bf16 a[156:159], v[92:95], v[16:19], a[156:159]\n"
    "v_mfma_f32_16x16x32_bf16 a[120:123], v[88:91], v[12:15], a[120:123]\n"
    "v_mfma_f32_16x16x32_bf16 a[124:127], v[92:95], v[12:15], a[124:127]\n"
    "v_mfma_f32_16x16x32_bf16 a[88:91], v[88:91], v[8:11], a[88:91]\n"
    "v_mfma_f32_16x16x32_bf16 a[92:95], v[92:95], v[8:11], a[92:95]\n"
    "buffer_load_dwordx4 v138, s[56:59], s52 offen offset:3072 sc0 sc1 lds\n"
    "v_mfma_f32_16x16x32_bf16 a[56:59], v[88:91], v[4:7], a[56:59]\n"
    "s_add_u32 s52, s52, 0x4000\n"
    "v_mfma_f32_16x16x32_bf16 a[60:63], v[92:95], v[4:7], a[60:63]\n"
    "v_mfma_f32_16x16x32_bf16 a[24:27], v[88:91], v[0:3], a[24:27]\n"
    "v_mfma_f32_16x16x32_bf16 a[28:31], v[92:95], v[0:3], a[28:31]\n"
    "s_waitcnt vmcnt(14) lgkmcnt(0)\n"
    "s_barrier\n"
    "v_mfma_f32_16x16x32_bf16 a[0:3], v[96:99], v[32:35], a[0:3]\n"
    "ds_read_b64_tr_b16 v[64:65], v144 offset:0\n"
    "v_mfma_f32_16x16x32_bf16 a[4:7], v[100:103], v[32:35], a[4:7]\n"
    "s_mov_b64 exec, s[70:71]\n"
    "global_load_dwordx4 v[176:179], v139, s[74:75] sc0 sc1\n"
    "s_mov_b64 exec, -1\n"
    "v_mfma_f32_16x16x32_bf16 a[32:35], v[96:99], v[36:39], a[32:35]\n"
    "ds_read_b64_tr_b16 v[66:67], v144 offset:512\n"
    "s_add_u32 m0, s50, 49152\n"
    "s_mov_b64 exec, s[30:31]\n"
    "global_store_dwordx4 v138, v[164:167], s[26:27] offset:1024 sc0 sc1\n"
    "s_mov_b64 exec, -1\n"
    "v_mfma_f32_16x16x32_bf16 a[36:39], v[100:103], v[36:39], a[36:39]\n"
    "buffer_load_dwordx4 v132, s[44:47], s49 offen offset:0 nt lds\n"
    "v_mfma_f32_16x16x32_bf16 a[64:67], v[96:99], v[40:43], a[64:67]\n"
    "ds_read_b128 v[0:3], v194 offset:0\n"
    "v_mfma_f32_16x16x32_bf16 a[68:71], v[100:103], v[40:43], a[68:71]\n"
    "v_mfma_f32_16x16x32_bf16 a[96:99], v[96:99], v[44:47], a[96:99]\n"
    "ds_read_b128 v[4:7], v194 offset:2048\n"
    "v_mfma_f32_16x16x32_bf16 a[100:103], v[100:103], v[44:47], a[100:103]\n"
    "v_mfma_f32_16x16x32_bf16 a[128:131], v[96:99], v[48:51], a[128:131]\n"
    "ds_read_b128 v[8:11], v194 offset:4096\n"
    "v_mfma_f32_16x16x32_bf16 a[132:135], v[100:103], v[48:51], a[132:135]\n"
    "v_mfma_f32_16x16x32_bf16 a[160:163], v[96:99], v[52:55], a[160:163]\n"
    "ds_read_b128 v[12:15], v194 offset:6144\n"
    "v_mfma_f32_16x16x32_bf16 a[164:167], v[100:103], v[52:55], a[164:167]\n"
    "buffer_load_dwordx4 v133, s[44:47], s49 offen offset:1024 nt lds\n"
    "v_mfma_f32_16x16x32_bf16 a[192:195], v[96:99], v[56:59], a[192:195]\n"
    "ds_read_b128 v[16:19], v194 offset:8192\n"
    "v_mfma_f32_16x16x32_bf16 a[196:199], v[100:103], v[56:59], a[196:199]\n"
    "v_mfma_f32_16x16x32_bf16 a[224:227], v[96:99], v[60:63], a[224:227]\n"
    "ds_read_b128 v[20:23], v194 offset:10240\n"
    "v_mfma_f32_16x16x32_bf16 a[228:231], v[100:103], v[60:63], a[228:231]\n"
    "v_mfma_f32_16x16x32_bf16 a[232:235], v[104:107], v[60:63], a[232:235]\n"
    "ds_read_b128 v[24:27], v194 offset:12288\n"
    "v_mfma_f32_16x16x32_bf16 a[236:239], v[108:111], v[60:63], a[236:239]\n"
    "v_mfma_f32_16x16x32_bf16 a[200:203], v[104:107], v[56:59], a[200:203]\n"
    "ds_read_b128 v[28:31], v194 offset:14336\n"
    "v_mfma_f32_16x16x32_bf16 a[204:207], v[108:111], v[56:59], a[204:207]\n"
    "buffer_load_dwordx4 v134, s[44:47], s49 offen offset:2048 nt lds\n"
    "v_mfma_f32_16x16x32_bf16 a[168:171], v[104:107], v[52:55], a[168:171]\n"
    "ds_read_b64_tr_b16 v[68:69], v145 offset:0\n"
    "v_mfma_f32_16x16x32_bf16 a[172:175], v[108:111], v[52:55], a[172:175]\n"
    "v_mfma_f32_16x16x32_bf16 a[136:139], v[104:107], v[48:51], a[136:139]\n"
    "ds_read_b64_tr_b16 v[70:71], v145 offset:512\n"
    "v_mfma_f32_16x16x32_bf16 a[140:143], v[108:111], v[48:51], a[140:143]\n"
    "v_mfma_f32_16x16x32_bf16 a[104:107], v[104:107], v[44:47], a[104:107]\n"
    "ds_read_b64_tr_b16 v[72:73], v146 offset:0\n"
    "v_mfma_f32_16x16x32_bf16 a[108:111], v[108:111], v[44:47], a[108:111]\n"
    "v_mfma_f32_16x16x32_bf16 a[72:75], v[104:107], v[40:43], a[72:75]\n"
    "ds_read_b64_tr_b16 v[74:75], v146 offset:512\n"
    "v_mfma_f32_16x16x32_bf16 a[76:79], v[108:111], v[40:43], a[76:79]\n"
    "buffer_load_dwordx4 v135, s[44:47], s49 offen offset:3072 nt lds\n"
    "v_mfma_f32_16x16x32_bf16 a[40:43], v[104:107], v[36:39], a[40:43]\n"
    "ds_read_b64_tr_b16 v[76:77], v147 offset:0\n"
    "v_mfma_f32_16x16x32_bf16 a[44:47], v[108:111], v[36:39], a[44:47]\n"
    "s_add_u32 s53, s53, 0x24924925\n"
    "s_cselect_b32 s55, s79, s80\n"
    "v_mfma_f32_16x16x32_bf16 a[8:11], v[104:107], v[32:35], a[8:11]\n"
    "ds_read_b64_tr_b16 v[78:79], v147 offset:512\n"
    "v_mfma_f32_16x16x32_bf16 a[12:15], v[108:111], v[32:35], a[12:15]\n"
    "s_add_u32 s49, s49, s55\n"
    "v_mfma_f32_16x16x32_bf16 a[16:19], v[112:115], v[32:35], a[16:19]\n"
    "ds_read_b64_tr_b16 v[80:81], v144 offset:4096\n"
    "v_mfma_f32_16x16x32_bf16 a[20:23], v[116:119], v[32:35], a[20:23]\n"
    "s_and_b32 s49, s49, 0xfffffff\n"
    "v_mfma_f32_16x16x32_bf16 a[48:51], v[112:115], v[36:39], a[48:51]\n"
    "ds_read_b64_tr_b16 v[82:83], v144 offset:4608\n"
    "s_add_u32 m0, s50, 32768\n"
    "v_mfma_f32_16x16x32_bf16 a[52:55], v[116:119], v[36:39], a[52:55]\n"
    "buffer_load_dwordx4 v138, s[56:59], s52 offen offset:0 sc0 sc1 lds\n"
    "v_mfma_f32_16x16x32_bf16 a[80:83], v[112:115], v[40:43], a[80:83]\n"
    "ds_read_b64_tr_b16 v[84:85], v145 offset:4096\n"
    "v_mfma_f32_16x16x32_bf16 a[84:87], v[116:119], v[40:43], a[84:87]\n"
    "v_mfma_f32_16x16x32_bf16 a[112:115], v[112:115], v[44:47], a[112:115]\n"
    "ds_read_b64_tr_b16 v[86:87], v145 offset:4608\n"
    "v_mfma_f32_16x16x32_bf16 a[116:119], v[116:119], v[44:47], a[116:119]\n"
    "v_mfma_f32_16x16x32_bf16 a[144:147], v[112:115], v[48:51], a[144:147]\n"
    "ds_read_b64_tr_b16 v[88:89], v146 offset:4096\n"
    "v_mfma_f32_16x16x32_bf16 a[148:151], v[116:119], v[48:51], a[148:151]\n"
    "v_mfma_f32_16x16x32_bf16 a[176:179], v[112:115], v[52:55], a[176:179]\n"
    "ds_read_b64_tr_b16 v[90:91], v146 offset:4608\n"
    "v_mfma_f32_16x16x32_bf16 a[180:183], v[116:119], v[52:55], a[180:183]\n"
    "buffer_load_dwordx4 v138, s[56:59], s52 offen offset:1024 sc0 sc1 lds\n"
    "v_mfma_f32_16x16x32_bf16 a[208:211], v[112:115], v[56:59], a[208:211]\n"
    "ds_read_b64_tr_b16 v[92:93], v147 offset:4096\n"
    "v_mfma_f32_16x16x32_bf16 a[212:215], v[116:119], v[56:59], a[212:215]\n"
    "v_mfma_f32_16x16x32_bf16 a[240:243], v[112:115], v[60:63], a[240:243]\n"
    "ds_read_b64_tr_b16 v[94:95], v147 offset:4608\n"
    "v_mfma_f32_16x16x32_bf16 a[244:247], v[116:119], v[60:63], a[244:247]\n"
    "v_mfma_f32_16x16x32_bf16 a[248:251], v[120:123], v[60:63], a[248:251]\n"
    "v_mfma_f32_16x16x32_bf16 a[252:255], v[124:127], v[60:63], a[252:255]\n"
    "v_mfma_f32_16x16x32_bf16 a[216:219], v[120:123], v[56:59], a[216:219]\n"
    "v_mfma_f32_16x16x32_bf16 a[220:223], v[124:127], v[56:59], a[220:223]\n"
    "buffer_load_dwordx4 v138, s[56:59], s52 offen offset:2048 sc0 sc1 lds\n"
    "v_mfma_f32_16x16x32_bf16 a[184:187], v[120:123], v[52:55], a[184:187]\n"
    "v_mfma_f32_16x16x32_bf16 a[188:191], v[124:127], v[52:55], a[188:191]\n"
    "v_mfma_f32_16x16x32_bf16 a[152:155], v[120:123], v[48:51], a[152:155]\n"
    "s_memrealtime s[28:29]\n"
    "v_mfma_f32_16x16x32_bf16 a[156:159], v[124:127], v[48:51], a[156:159]\n"
    "v_mfma_f32_16x16x32_bf16 a[120:123], v[120:123], v[44:47], a[120:123]\n"
    "v_mfma_f32_16x16x32_bf16 a[124:127], v[124:127], v[44:47], a[124:127]\n"
    "v_mfma_f32_16x16x32_bf16 a[88:91], v[120:123], v[40:43], a[88:91]\n"
    "v_mfma_f32_16x16x32_bf16 a[92:95], v[124:127], v[40:43], a[92:95]\n"
    "buffer_load_dwordx4 v138, s[56:59], s52 offen offset:3072 sc0 sc1 lds\n"
    "v_mfma_f32_16x16x32_bf16 a[56:59], v[120:123], v[36:39], a[56:59]\n"
    "s_add_u32 s52, s52, 0x4000\n"
    "v_mfma_f32_16x16x32_bf16 a[60:63], v[124:127], v[36:39], a[60:63]\n"
    "v_mfma_f32_16x16x32_bf16 a[24:27], v[120:123], v[32:35], a[24:27]\n"
    "v_mfma_f32_16x16x32_bf16 a[28:31], v[124:127], v[32:35], a[28:31]\n"
    "s_waitcnt vmcnt(18) lgkmcnt(0)\n"
    "s_barrier\n"
    "v_mfma_f32_16x16x32_bf16 a[0:3], v[64:67], v[0:3], a[0:3]\n"
    "ds_read_b64_tr_b16 v[96:97], v144 offset:32768\n"
    "v_mfma_f32_16x16x32_bf16 a[4:7], v[68:71], v[0:3], a[4:7]\n"
    "v_mfma_f32_16x16x32_bf16 a[32:35], v[64:67], v[4:7], a[32:35]\n"
    "ds_read_b64_tr_b16 v[98:99], v144 offset:33280\n"
    "s_add_u32 m0, s50, 81920\n"
    "s_mov_b64 exec, s[30:31]\n"
    "global_store_dwordx4 v138, v[168:171], s[26:27] offset:2048 sc0 sc1\n"
    "s_mov_b64 exec, -1\n"
    "v_mfma_f32_16x16x32_bf16 a[36:39], v[68:71], v[4:7], a[36:39]\n"
    "buffer_load_dwordx4 v132, s[44:47], s49 offen offset:0 nt lds\n"
    "v_mfma_f32_16x16x32_bf16 a[64:67], v[64:67], v[8:11], a[64:67]\n"
    "ds_read_b128 v[32:35], v195 offset:0\n"
    "v_mfma_f32_16x16x32_bf16 a[68:71], v[68:71], v[8:11], a[68:71]\n"
    "s_lshr_b32 s54, s24, 4\n"
    "s_add_u32 s33, s33, s54\n"
    "v_mfma_f32_16x16x32_bf16 a[96:99], v[64:67], v[12:15], a[96:99]\n"
    "ds_read_b128 v[36:39], v195 offset:2048\n"
    "s_add_u32 s32, s32, s54\n"
    "v_mfma_f32_16x16x32_bf16 a[100:103], v[68:71], v[12:15], a[100:103]\n"
    "s_cmp_eq_u32 s73, 3\n"
    "s_cselect_b64 exec, 1, 0\n"
    "global_store_dword v148, v152, s[64:65] sc0 sc1\n"
    "s_mov_b64 exec, -1\n"
    "s_sub_u32 s54, s33, s28\n"
    "s_max_i32 s54, s54, -250\n"
    "v_mfma_f32_16x16x32_bf16 a[128:131], v[64:67], v[16:19], a[128:131]\n"
    "ds_read_b128 v[40:43], v195 offset:4096\n"
    "v_mfma_f32_16x16x32_bf16 a[132:135], v[68:71], v[16:19], a[132:135]\n"
    "s_add_u32 s33, s28, s54\n"
    "s_cmp_gt_i32 s54, 0\n"
    "v_mfma_f32_16x16x32_bf16 a[160:163], v[64:67], v[20:23], a[160:163]\n"
    "ds_read_b128 v[44:47], v195 offset:6144\n"
    "s_cbranch_scc1 PACEI32_%=\n"
    "PACEBI32_%=:\n"
    "v_mfma_f32_16x16x32_bf16 a[164:167], v[68:71], v[20:23], a[164:167]\n"
    "buffer_load_dwordx4 v133, s[44:47], s49 offen offset:1024 nt lds\n"
    "v_mfma_f32_16x16x32_bf16 a[192:195], v[64:67], v[24:27], a[192:195]\n"
    "ds_read_b128 v[48:51], v195 offset:8192\n"
    "v_mfma_f32_16x16x32_bf16 a[196:199], v[68:71], v[24:27], a[196:199]\n"
    "s_sub_u32 s24, s24, 16\n"
    "s_max_i32 s24, s24, s35\n"
    "v_mfma_f32_16x16x32_bf16 a[224:227], v[64:67], v[28:31], a[224:227]\n"
    "ds_read_b128 v[52:55], v195 offset:10240\n"
    "v_mfma_f32_16x16x32_bf16 a[228:231], v[68:71], v[28:31], a[228:231]\n"
    "v_mfma_f32_16x16x32_bf16 a[232:235], v[72:75], v[28:31], a[232:235]\n"
    "ds_read_b128 v[56:59], v195 offset:12288\n"
    "v_mfma_f32_16x16x32_bf16 a[236:239], v[76:79], v[28:31], a[236:239]\n"
    "v_mfma_f32_16x16x32_bf16 a[200:203], v[72:75], v[24:27], a[200:203]\n"
    "ds_read_b128 v[60:63], v195 offset:14336\n"
    "v_mfma_f32_16x16x32_bf16 a[204:207], v[76:79], v[24:27], a[204:207]\n"
    "buffer_load_dwordx4 v134, s[44:47], s49 offen offset:2048 nt lds\n"
    "v_mfma_f32_16x16x32_bf16 a[168:171], v[72:75], v[20:23], a[168:171]\n"
    "ds_read_b64_tr_b16 v[100:101], v145 offset:32768\n"
    "v_mfma_f32_16x16x32_bf16 a[172:175], v[76:79], v[20:23], a[172:175]\n"
    "v_mfma_f32_16x16x32_bf16 a[136:139], v[72:75], v[16:19], a[136:139]\n"
    "ds_read_b64_tr_b16 v[102:103], v145 offset:33280\n"
    "v_mfma_f32_16x16x32_bf16 a[140:143], v[76:79], v[16:19], a[140:143]\n"
    "v_mfma_f32_16x16x32_bf16 a[104:107], v[72:75], v[12:15], a[104:107]\n"
    "ds_read_b64_tr_b16 v[104:105], v146 offset:32768\n"
    "v_mfma_f32_16x16x32_bf16 a[108:111], v[76:79], v[12:15], a[108:111]\n"
    "v_mfma_f32_16x16x32_bf16 a[72:75], v[72:75], v[8:11], a[72:75]\n"
    "ds_read_b64_tr_b16 v[106:107], v146 offset:33280\n"
    "v_mfma_f32_16x16x32_bf16 a[76:79], v[76:79], v[8:11], a[76:79]\n"
    "buffer_load_dwordx4 v135, s[44:47], s49 offen offset:3072 nt lds\n"
    "v_mfma_f32_16x16x32_bf16 a[40:43], v[72:75], v[4:7], a[40:43]\n"
    "ds_read_b64_tr_b16 v[108:109], v147 offset:32768\n"
    "v_mfma_f32_16x16x32_bf16 a[44:47], v[76:79], v[4:7], a[44:47]\n"
    "s_add_u32 s53, s53, 0x24924925\n"
    "s_cselect_b32 s55, s79, s80\n"
    "v_mfma_f32_16x16x32_bf16 a[8:11], v[72:75], v[0:3], a[8:11]\n"
    "ds_read_b64_tr_b16 v[110:111], v147 offset:33280\n"
    "v_mfma_f32_16x16x32_bf16 a[12:15], v[76:79], v[0:3], a[12:15]\n"
    "s_add_u32 s49, s49, s55\n"
    "v_mfma_f32_16x16x32_bf16 a[16:19], v[80:83], v[0:3], a[16:19]\n"
    "ds_read_b64_tr_b16 v[112:113], v144 offset:36864\n"
    "v_mfma_f32_16x16x32_bf16 a[20:23], v[84:87], v[0:3], a[20:23]\n"
    "s_and_b32 s49, s49, 0xfffffff\n"
    "v_mfma_f32_16x16x32_bf16 a[48:51], v[80:83], v[4:7], a[48:51]\n"
    "ds_read_b64_tr_b16 v[114:115], v144 offset:37376\n"
    "s_add_u32 m0, s50, 65536\n"
    "v_mfma_f32_16x16x32_bf16 a[52:55], v[84:87], v[4:7], a[52:55]\n"
    "buffer_load_dwordx4 v138, s[56:59], s52 offen offset:0 sc0 sc1 lds\n"
    "v_mfma_f32_16x16x32_bf16 a[80:83], v[80:83], v[8:11], a[80:83]\n"
    "ds_read_b64_tr_b16 v[116:117], v145 offset:36864\n"
    "v_mfma_f32_16x16x32_bf16 a[84:87], v[84:87], v[8:11], a[84:87]\n"
    "v_mfma_f32_16x16x32_bf16 a[112:115], v[80:83], v[12:15], a[112:115]\n"
    "ds_read_b64_tr_b16 v[118:119], v145 offset:37376\n"
    "v_mfma_f32_16x16x32_bf16 a[116:119], v[84:87], v[12:15], a[116:119]\n"
    "v_mfma_f32_16x16x32_bf16 a[144:147], v[80:83], v[16:19], a[144:147]\n"
    "ds_read_b64_tr_b16 v[120:121], v146 offset:36864\n"
    "v_mfma_f32_16x16x32_bf16 a[148:151], v[84:87], v[16:19], a[148:151]\n"
    "v_mfma_f32_16x16x32_bf16 a[176:179], v[80:83], v[20:23], a[176:179]\n"
    "ds_read_b64_tr_b16 v[122:123], v146 offset:37376\n"
    "v_mfma_f32_16x16x32_bf16 a[180:183], v[84:87], v[20:23], a[180:183]\n"
    "buffer_load_dwordx4 v138, s[56:59], s52 offen offset:1024 sc0 sc1 lds\n"
    "v_mfma_f32_16x16x32_bf16 a[208:211], v[80:83], v[24:27], a[208:211]\n"
    "ds_read_b64_tr_b16 v[124:125], v147 offset:36864\n"
    "v_mfma_f32_16x16x32_bf16 a[212:215], v[84:87], v[24:27], a[212:215]\n"
    "v_mfma_f32_16x16x32_bf16 a[240:243], v[80:83], v[28:31], a[240:243]\n"
    "ds_read_b64_tr_b16 v[126:127], v147 offset:37376\n"
    "v_mfma_f32_16x16x32_bf16 a[244:247], v[84:87], v[28:31], a[244:247]\n"
    "v_mfma_f32_16x16x32_bf16 a[248:251], v[88:91], v[28:31], a[248:251]\n"
    "v_mfma_f32_16x16x32_bf16 a[252:255], v[92:95], v[28:31], a[252:255]\n"
    "v_mfma_f32_16x16x32_bf16 a[216:219], v[88:91], v[24:27], a[216:219]\n"
    "v_mfma_f32_16x16x32_bf16 a[220:223], v[92:95], v[24:27], a[220:223]\n"
    "buffer_load_dwordx4 v138, s[56:59], s52 offen offset:2048 sc0 sc1 lds\n"
    "v_mfma_f32_16x16x32_bf16 a[184:187], v[88:91], v[20:23], a[184:187]\n"
    "v_mfma_f32_16x16x32_bf16 a[188:191], v[92:95], v[20:23], a[188:191]\n"
    "v_mfma_f32_16x16x32_bf16 a[152:155], v[88:91], v[16:19], a[152:155]\n"
    "v_mfma_f32_16x16x32_bf16 a[156:159], v[92:95], v[16:19], a[156:159]\n"
    "v_mfma_f32_16x16x32_bf16 a[120:123], v[88:91], v[12:15], a[120:123]\n"
    "v_mfma_f32_16x16x32_bf16 a[124:127], v[92:95], v[12:15], a[124:127]\n"
    "v_mfma_f32_16x16x32_bf16 a[88:91], v[88:91], v[8:11], a[88:91]\n"
    "v_mfma_f32_16x16x32_bf16 a[92:95], v[92:95], v[8:11], a[92:95]\n"
    "buffer_load_dwordx4 v138, s[56:59], s52 offen offset:3072 sc0 sc1 lds\n"
    "v_mfma_f32_16x16x32_bf16 a[56:59], v[88:91], v[4:7], a[56:59]\n"
    "s_add_u32 s52, s52, 0x4000\n"
    "v_mfma_f32_16x16x32_bf16 a[60:63], v[92:95], v[4:7], a[60:63]\n"
    "v_mfma_f32_16x16x32_bf16 a[24:27], v[88:91], v[0:3], a[24:27]\n"
    "v_mfma_f32_16x16x32_bf16 a[28:31], v[92:95], v[0:3], a[28:31]\n"
    "s_waitcnt vmcnt(18) lgkmcnt(0)\n"
    "s_barrier\n"
    "v_mfma_f32_16x16x32_bf16 a[0:3], v[96:99], v[32:35], a[0:3]\n"
    "ds_read_b64_tr_b16 v[64:65], v140 offset:0\n"
    "v_mfma_f32_16x16x32_bf16 a[4:7], v[100:103], v[32:35], a[4:7]\n"
    "v_min_u32 v176, v176, v177\n"
    "v_mfma_f32_16x16x32_bf16 a[32:35], v[96:99], v[36:39], a[32:35]\n"
    "ds_read_b64_tr_b16 v[66:67], v140 offset:512\n"
    "s_add_u32 m0, s50, 114688\n"
    "s_mov_b64 exec, s[30:31]\n"
    "global_store_dwordx4 v138, v[172:175], s[26:27] offset:3072 sc0 sc1\n"
    "s_mov_b64 exec, -1\n"
    "v_mfma_f32_16x16x32_bf16 a[36:39], v[100:103], v[36:39], a[36:39]\n"
    "buffer_load_dwordx4 v132, s[44:47], s49 offen offset:0 nt lds\n"
    "v_mfma_f32_16x16x32_bf16 a[64:67], v[96:99], v[40:43], a[64:67]\n"
    "ds_read_b128 v[0:3], v136 offset:0\n"
    "v_mfma_f32_16x16x32_bf16 a[68:71], v[100:103], v[40:43], a[68:71]\n"
    "v_min3_u32 v176, v176, v178, v179\n"
    "v_mfma_f32_16x16x32_bf16 a[96:99], v[96:99], v[44:47], a[96:99]\n"
    "ds_read_b128 v[4:7], v136 offset:1024\n"
    "v_mfma_f32_16x16x32_bf16 a[100:103], v[100:103], v[44:47], a[100:103]\n"
    "v_cmp_le_u32 vcc, s68, v176\n"
    "v_mfma_f32_16x16x32_bf16 a[128:131], v[96:99], v[48:51], a[128:131]\n"
    "ds_read_b128 v[8:11], v136 offset:2048\n"
    "v_mfma_f32_16x16x32_bf16 a[132:135], v[100:103], v[48:51], a[132:135]\n"
    "v_mfma_f32_16x16x32_bf16 a[160:163], v[96:99], v[52:55], a[160:163]\n"
    "ds_read_b128 v[12:15], v136 offset:3072\n"
    "v_mfma_f32_16x16x32_bf16 a[164:167], v[100:103], v[52:55], a[164:167]\n"
    "buffer_load_dwordx4 v133, s[44:47], s49 offen offset:1024 nt lds\n"
    "v_mfma_f32_16x16x32_bf16 a[192:195], v[96:99], v[56:59], a[192:195]\n"
    "ds_read_b128 v[16:19], v136 offset:4096\n"
    "v_mfma_f32_16x16x32_bf16 a[196:199], v[100:103], v[56:59], a[196:199]\n"
    "s_and_b32 s54, vcc_lo, s70\n"
    "s_or_b32 s69, s69, s54\n"
    "v_mfma_f32_16x16x32_bf16 a[224:227], v[96:99], v[60:63], a[224:227]\n"
    "ds_read_b128 v[20:23], v136 offset:5120\n"
    "v_mfma_f32_16x16x32_bf16 a[228:231], v[100:103], v[60:63], a[228:231]\n"
    "s_andn2_b32 s70, s70, s54\n"
    "s_bitcmp1_b32 s69, s72\n"
    "v_mfma_f32_16x16x32_bf16 a[232:235], v[104:107], v[60:63], a[232:235]\n"
    "ds_read_b128 v[24:27], v136 offset:6144\n"
    "v_mfma_f32_16x16x32_bf16 a[236:239], v[108:111], v[60:63], a[236:239]\n"
    "s_cbranch_scc0 SLOWI3_%=\n"
    "SLOWBACKI3_%=:\n"
    "s_add_u32 s72, s72, 1\n"
    "v_mfma_f32_16x16x32_bf16 a[200:203], v[104:107], v[56:59], a[200:203]\n"
    "ds_read_b128 v[28:31], v136 offset:7168\n"
    "v_mfma_f32_16x16x32_bf16 a[204:207], v[108:111], v[56:59], a[204:207]\n"
    "buffer_load_dwordx4 v134, s[44:47], s49 offen offset:2048 nt lds\n"
    "v_mfma_f32_16x16x32_bf16 a[168:171], v[104:107], v[52:55], a[168:171]\n"
    "ds_read_b64_tr_b16 v[68:69], v141 offset:0\n"
    "v_mfma_f32_16x16x32_bf16 a[172:175], v[108:111], v[52:55], a[172:175]\n"
    "v_mfma_f32_16x16x32_bf16 a[136:139], v[104:107], v[48:51], a[136:139]\n"
    "ds_read_b64_tr_b16 v[70:71], v141 offset:512\n"
    "v_mfma_f32_16x16x32_bf16 a[140:143], v[108:111], v[48:51], a[140:143]\n"
    "v_mfma_f32_16x16x32_bf16 a[104:107], v[104:107], v[44:47], a[104:107]\n"
    "ds_read_b64_tr_b16 v[72:73], v142 offset:0\n"
    "v_mfma_f32_16x16x32_bf16 a[108:111], v[108:111], v[44:47], a[108:111]\n"
    "v_mfma_f32_16x16x32_bf16 a[72:75], v[104:107], v[40:43], a[72:75]\n"
    "ds_read_b64_tr_b16 v[74:75], v142 offset:512\n"
    "v_mfma_f32_16x16x32_bf16 a[76:79], v[108:111], v[40:43], a[76:79]\n"
    "buffer_load_dwordx4 v135, s[44:47], s49 offen offset:3072 nt lds\n"
    "v_mfma_f32_16x16x32_bf16 a[40:43], v[104:107], v[36:39], a[40:43]\n"
    "ds_read_b64_tr_b16 v[76:77], v143 offset:0\n"
    "v_mfma_f32_16x16x32_bf16 a[44:47], v[108:111], v[36:39], a[44:47]\n"
    "s_add_u32 s53, s53, 0x24924925\n"
    "s_cselect_b32 s55, s79, s80\n"
    "v_mfma_f32_16x16x32_bf16 a[8:11], v[104:107], v[32:35], a[8:11]\n"
    "ds_read_b64_tr_b16 v[78:79], v143 offset:512\n"
    "v_mfma_f32_16x16x32_bf16 a[12:15], v[108:111], v[32:35], a[12:15]\n"
    "s_add_u32 s49, s49, s55\n"
    "v_mfma_f32_16x16x32_bf16 a[16:19], v[112:115], v[32:35], a[16:19]\n"
    "ds_read_b64_tr_b16 v[80:81], v140 offset:4096\n"
    "v_mfma_f32_16x16x32_bf16 a[20:23], v[116:119], v[32:35], a[20:23]\n"
    "s_and_b32 s49, s49, 0xfffffff\n"
    "v_mfma_f32_16x16x32_bf16 a[48:51], v[112:115], v[36:39], a[48:51]\n"
    "ds_read_b64_tr_b16 v[82:83], v140 offset:4608\n"
    "s_add_u32 m0, s50, 98304\n"
    "v_mfma_f32_16x16x32_bf16 a[52:55], v[116:119], v[36:39], a[52:55]\n"
    "buffer_load_dwordx4 v138, s[56:59], s52 offen offset:0 sc0 sc1 lds\n"
    "v_mfma_f32_16x16x32_bf16 a[80:83], v[112:115], v[40:43], a[80:83]\n"
    "ds_read_b64_tr_b16 v[84:85], v141 offset:4096\n"
    "v_mfma_f32_16x16x32_bf16 a[84:87], v[116:119], v[40:43], a[84:87]\n"
    "v_mfma_f32_16x16x32_bf16 a[112:115], v[112:115], v[44:47], a[112:115]\n"
    "ds_read_b64_tr_b16 v[86:87], v141 offset:4608\n"
    "v_mfma_f32_16x16x32_bf16 a[116:119], v[116:119], v[44:47], a[116:119]\n"
    "v_mfma_f32_16x16x32_bf16 a[144:147], v[112:115], v[48:51], a[144:147]\n"
    "ds_read_b64_tr_b16 v[88:89], v142 offset:4096\n"
    "v_mfma_f32_16x16x32_bf16 a[148:151], v[116:119], v[48:51], a[148:151]\n"
    "s_mov_b32 s51, 5\n"
    "v_mfma_f32_16x16x32_bf16 a[176:179], v[112:115], v[52:55], a[176:179]\n"
    "ds_read_b64_tr_b16 v[90:91], v142 offset:4608\n"
    "v_mfma_f32_16x16x32_bf16 a[180:183], v[116:119], v[52:55], a[180:183]\n"
    "buffer_load_dwordx4 v138, s[56:59], s52 offen offset:1024 sc0 sc1 lds\n"
    "v_mfma_f32_16x16x32_bf16 a[208:211], v[112:115], v[56:59], a[208:211]\n"
    "ds_read_b64_tr_b16 v[92:93], v143 offset:4096\n"
    "v_mfma_f32_16x16x32_bf16 a[212:215], v[116:119], v[56:59], a[212:215]\n"
    "v_mfma_f32_16x16x32_bf16 a[240:243], v[112:115], v[60:63], a[240:243]\n"
    "ds_read_b64_tr_b16 v[94:95], v143 offset:4608\n"
    "v_mfma_f32_16x16x32_bf16 a[244:247], v[116:119], v[60:63], a[244:247]\n"
    "v_mfma_f32_16x16x32_bf16 a[248:251], v[120:123], v[60:63], a[248:251]\n"
    "v_mfma_f32_16x16x32_bf16 a[252:255], v[124:127], v[60:63], a[252:255]\n"
    "v_mfma_f32_16x16x32_bf16 a[216:219], v[120:123], v[56:59], a[216:219]\n"
    "v_mfma_f32_16x16x32_bf16 a[220:223], v[124:127], v[56:59], a[220:223]\n"
    "buffer_load_dwordx4 v138, s[56:59], s52 offen offset:2048 sc0 sc1 lds\n"
    "v_mfma_f32_16x16x32_bf16 a[184:187], v[120:123], v[52:55], a[184:187]\n"
    "v_mfma_f32_16x16x32_bf16 a[188:191], v[124:127], v[52:55], a[188:191]\n"
    "v_mfma_f32_16x16x32_bf16 a[152:155], v[120:123], v[48:51], a[152:155]\n"
    "s_memrealtime s[28:29]\n"
    "v_mfma_f32_16x16x32_bf16 a[156:159], v[124:127], v[48:51], a[156:159]\n"
    "v_mfma_f32_16x16x32_bf16 a[120:123], v[120:123], v[44:47], a[120:123]\n"
    "v_mfma_f32_16x16x32_bf16 a[124:127], v[124:127], v[44:47], a[124:127]\n"
    "v_mfma_f32_16x16x32_bf16 a[88:91], v[120:123], v[40:43], a[88:91]\n"
    "v_mfma_f32_16x16x32_bf16 a[92:95], v[124:127], v[40:43], a[92:95]\n"
    "buffer_load_dwordx4 v138, s[56:59], s52 offen offset:3072 sc0 sc1 lds\n"
    "v_mfma_f32_16x16x32_bf16 a[56:59], v[120:123], v[36:39], a[56:59]\n"
    "s_add_u32 s52, s52, 0x4000\n"
    "v_mfma_f32_16x16x32_bf16 a[60:63], v[124:127], v[36:39], a[60:63]\n"
    "v_mfma_f32_16x16x32_bf16 a[24:27], v[120:123], v[32:35], a[24:27]\n"
    "v_mfma_f32_16x16x32_bf16 a[28:31], v[124:127], v[32:35], a[28:31]\n"
    "s_waitcnt vmcnt(18) lgkmcnt(0)\n"
    "s_barrier\n"
    "LOOPA_%=:\n"
    "v_mfma_f32_16x16x32_bf16 a[0:3], v[64:67], v[0:3], a[0:3]\n"
    "ds_read_b64_tr_b16 v[96:97], v140 offset:32768\n"
    "s_add_u32 s67, s67, 1\n"
    "v_mfma_f32_16x16x32_bf16 a[4:7], v[68:71], v[0:3], a[4:7]\n"
    "s_cmp_eq_u32 s66, s67\n"
    "s_cselect_b64 s[30:31], -1, 1\n"
    "s_cselect_b64 s[26:27], s[60:61], s[62:63]\n"
    "v_mfma_f32_16x16x32_bf16 a[32:35], v[64:67], v[4:7], a[32:35]\n"
    "ds_read_b64_tr_b16 v[98:99], v140 offset:33280\n"
    "s_add_u32 m0, s50, 16384\n"
    "s_mov_b64 exec, s[30:31]\n"
    "global_store_dwordx4 v138, v[160:163], s[26:27] offset:0 sc0 sc1\n"
    "s_mov_b64 exec, -1\n"
    "v_mfma_f32_16x16x32_bf16 a[36:39], v[68:71], v[4:7], a[36:39]\n"
    "buffer_load_dwordx4 v132, s[44:47], s49 offen offset:0 nt lds\n"
    "v_mfma_f32_16x16x32_bf16 a[64:67], v[64:67], v[8:11], a[64:67]\n"
    "ds_read_b128 v[32:35], v136 offset:32768\n"
    "v_mfma_f32_16x16x32_bf16 a[68:71], v[68:71], v[8:11], a[68:71]\n"
    "s_lshr_b32 s54, s24, 4\n"
    "s_add_u32 s33, s33, s54\n"
    "v_mfma_f32_16x16x32_bf16 a[96:99], v[64:67], v[12:15], a[96:99]\n"
    "ds_read_b128 v[36:39], v136 offset:33792\n"
    "s_add_u32 s32, s32, s54\n"
    "v_mfma_f32_16x16x32_bf16 a[100:103], v[68:71], v[12:15], a[100:103]\n"
    "s_sub_u32 s54, s33, s28\n"
    "s_max_i32 s54, s54, -250\n"
    "v_mfma_f32_16x16x32_bf16 a[128:131], v[64:67], v[16:19], a[128:131]\n"
    "ds_read_b128 v[40:43], v136 offset:34816\n"
    "v_mfma_f32_16x16x32_bf16 a[132:135], v[68:71], v[16:19], a[132:135]\n"
    "s_add_u32 s33, s28, s54\n"
    "s_cmp_gt_i32 s54, 0\n"
    "v_mfma_f32_16x16x32_bf16 a[160:163], v[64:67], v[20:23], a[160:163]\n"
    "ds_read_b128 v[44:47], v136 offset:35840\n"
    "s_cbranch_scc1 PACEA0_%=\n"
    "PACEBA0_%=:\n"
    "v_mfma_f32_16x16x32_bf16 a[164:167], v[68:71], v[20:23], a[164:167]\n"
    "buffer_load_dwordx4 v133, s[44:47], s49 offen offset:1024 nt lds\n"
    "v_mfma_f32_16x16x32_bf16 a[192:195], v[64:67], v[24:27], a[192:195]\n"
    "ds_read_b128 v[48:51], v136 offset:36864\n"
    "v_mfma_f32_16x16x32_bf16 a[196:199], v[68:71], v[24:27], a[196:199]\n"
    "s_sub_u32 s24, s24, 16\n"
    "s_max_i32 s24, s24, s35\n"
    "v_mfma_f32_16x16x32_bf16 a[224:227], v[64:67], v[28:31], a[224:227]\n"
    "ds_read_b128 v[52:55], v136 offset:37888\n"
    "v_mfma_f32_16x16x32_bf16 a[228:231], v[68:71], v[28:31], a[228:231]\n"
    "v_mfma_f32_16x16x32_bf16 a[232:235], v[72:75], v[28:31], a[232:235]\n"
    "ds_read_b128 v[56:59], v136 offset:38912\n"
    "v_mfma_f32_16x16x32_bf16 a[236:239], v[76:79], v[28:31], a[236:239]\n"
    "v_mfma_f32_16x16x32_bf16 a[200:203], v[72:75], v[24:27], a[200:203]\n"
    "ds_read_b128 v[60:63], v136 offset:39936\n"
    "v_mfma_f32_16x16x32_bf16 a[204:207], v[76:79], v[24:27], a[204:207]\n"
    "buffer_load_dwordx4 v134, s[44:47], s49 offen offset:2048 nt lds\n"
    "v_mfma_f32_16x16x32_bf16 a[168:171], v[72:75], v[20:23], a[168:171]\n"
    "ds_read_b64_tr_b16 v[100:101], v141 offset:32768\n"
    "v_mfma_f32_16x16x32_bf16 a[172:175], v[76:79], v[20:23], a[172:175]\n"
    "v_mfma_f32_16x16x32_bf16 a[136:139], v[72:75], v[16:19], a[136:139]\n"
    "ds_read_b64_tr_b16 v[102:103], v141 offset:33280\n"
    "v_mfma_f32_16x16x32_bf16 a[140:143], v[76:79], v[16:19], a[140:143]\n"
    "v_mfma_f32_16x16x32_bf16 a[104:107], v[72:75], v[12:15], a[104:107]\n"
    "ds_read_b64_tr_b16 v[104:105], v142 offset:32768\n"
    "v_mfma_f32_16x16x32_bf16 a[108:111], v[76:79], v[12:15], a[108:111]\n"
    "s_sub_u32 s51, s51, 1\n"
    "v_mfma_f32_16x16x32_bf16 a[72:75], v[72:75], v[8:11], a[72:75]\n"
    "ds_read_b64_tr_b16 v[106:107], v142 offset:33280\n"
    "v_mfma_f32_16x16x32_bf16 a[76:79], v[76:79], v[8:11], a[76:79]\n"
    "buffer_load_dwordx4 v135, s[44:47], s49 offen offset:3072 nt lds\n"
    "v_mfma_f32_16x16x32_bf16 a[40:43], v[72:75], v[4:7], a[40:43]\n"
    "ds_read_b64_tr_b16 v[108:109], v143 offset:32768\n"
    "v_mfma_f32_16x16x32_bf16 a[44:47], v[76:79], v[4:7], a[44:47]\n"
    "s_add_u32 s53, s53, 0x24924925\n"
    "s_cselect_b32 s55, s79, s80\n"
    "v_mfma_f32_16x16x32_bf16 a[8:11], v[72:75], v[0:3], a[8:11]\n"
    "ds_read_b64_tr_b16 v[110:111], v143 offset:33280\n"
    "v_mfma_f32_16x16x32_bf16 a[12:15], v[76:79], v[0:3], a[12:15]\n"
    "s_add_u32 s49, s49, s55\n"
    "v_mfma_f32_16x16x32_bf16 a[16:19], v[80:83], v[0:3], a[16:19]\n"
    "ds_read_b64_tr_b16 v[112:113], v140 offset:36864\n"
    "v_mfma_f32_16x16x32_bf16 a[20:23], v[84:87], v[0:3], a[20:23]\n"
    "s_and_b32 s49, s49, 0xfffffff\n"
    "v_mfma_f32_16x16x32_bf16 a[48:51], v[80:83], v[4:7], a[48:51]\n"
    "ds_read_b64_tr_b16 v[114:115], v140 offset:37376\n"
    "s_add_u32 m0, s50, 0\n"
    "v_mfma_f32_16x16x32_bf16 a[52:55], v[84:87], v[4:7], a[52:55]\n"
    "buffer_load_dwordx4 v138, s[56:59], s52 offen offset:0 sc0 sc1 lds\n"
    "v_mfma_f32_16x16x32_bf16 a[80:83], v[80:83], v[8:11], a[80:83]\n"
    "ds_read_b64_tr_b16 v[116:117], v141 offset:36864\n"
    "v_mfma_f32_16x16x32_bf16 a[84:87], v[84:87], v[8:11], a[84:87]\n"
    "v_mfma_f32_16x16x32_bf16 a[112:115], v[80:83], v[12:15], a[112:115]\n"
    "ds_read_b64_tr_b16 v[118:119], v141 offset:37376\n"
    "v_mfma_f32_16x16x32_bf16 a[116:119], v[84:87], v[12:15], a[116:119]\n"
    "v_mfma_f32_16x16x32_bf16 a[144:147], v[80:83], v[16:19], a[144:147]\n"
    "ds_read_b64_tr_b16 v[120:121], v142 offset:36864\n"
    "v_mfma_f32_16x16x32_bf16 a[148:151], v[84:87], v[16:19], a[148:151]\n"
    "v_mfma_f32_16x16x32_bf16 a[176:179], v[80:83], v[20:23], a[176:179]\n"
    "ds_read_b64_tr_b16 v[122:123], v142 offset:37376\n"
    "v_mfma_f32_16x16x32_bf16 a[180:183], v[84:87], v[20:23], a[180:183]\n"
    "buffer_load_dwordx4 v138, s[56:59], s52 offen offset:1024 sc0 sc1 lds\n"
    "v_mfma_f32_16x16x32_bf16 a[208:211], v[80:83], v[24:27], a[208:211]\n"
    "ds_read_b64_tr_b16 v[124:125], v143 offset:36864\n"
    "v_mfma_f32_16x16x32_bf16 a[212:215], v[84:87], v[24:27], a[212:215]\n"
    "v_mfma_f32_16x16x32_bf16 a[240:243], v[80:83], v[28:31], a[240:243]\n"
    "ds_read_b64_tr_b16 v[126:127], v143 offset:37376\n"
    "v_mfma_f32_16x16x32_bf16 a[244:247], v[84:87], v[28:31], a[244:247]\n"
    "v_mfma_f32_16x16x32_bf16 a[248:251], v[88:91], v[28:31], a[248:251]\n"
    "v_mfma_f32_16x16x32_bf16 a[252:255], v[92:95], v[28:31], a[252:255]\n"
    "v_mfma_f32_16x16x32_bf16 a[216:219], v[88:91], v[24:27], a[216:219]\n"
    "v_mfma_f32_16x16x32_bf16 a[220:223], v[92:95], v[24:27], a[220:223]\n"
    "buffer_load_dwordx4 v138, s[56:59], s52 offen offset:2048 sc0 sc1 lds\n"
    "v_mfma_f32_16x16x32_bf16 a[184:187], v[88:91], v[20:23], a[184:187]\n"
    "v_mfma_f32_16x16x32_bf16 a[188:191], v[92:95], v[20:23], a[188:191]\n"
    "v_mfma_f32_16x16x32_bf16 a[152:155], v[88:91], v[16:19], a[152:155]\n"
    "v_mfma_f32_16x16x32_bf16 a[156:159], v[92:95], v[16:19], a[156:159]\n"
    "v_mfma_f32_16x16x32_bf16 a[120:123], v[88:91], v[12:15], a[120:123]\n"
    "v_mfma_f32_16x16x32_bf16 a[124:127], v[92:95], v[12:15], a[124:127]\n"
    "v_mfma_f32_16x16x32_bf16 a[88:91], v[88:91], v[8:11], a[88:91]\n"
    "v_mfma_f32_16x16x32_bf16 a[92:95], v[92:95], v[8:11], a[92:95]\n"
    "buffer_load_dwordx4 v138, s[56:59], s52 offen offset:3072 sc0 sc1 lds\n"
    "v_mfma_f32_16x16x32_bf16 a[56:59], v[88:91], v[4:7], a[56:59]\n"
    "s_add_u32 s52, s52, 0x4000\n"
    "v_mfma_f32_16x16x32_bf16 a[60:63], v[92:95], v[4:7], a[60:63]\n"
    "v_mfma_f32_16x16x32_bf16 a[24:27], v[88:91], v[0:3], a[24:27]\n"
    "s_waitcnt vmcnt(18) lgkmcnt(0)\n"
    "v_mfma_f32_16x16x32_bf16 a[28:31], v[92:95], v[0:3], a[28:31]\n"
    "s_barrier\n"
    "v_mfma_f32_16x16x32_bf16 a[0:3], v[96:99], v[32:35], a[0:3]\n"
    "ds_read_b64_tr_b16 v[64:65], v144 offset:0\n"
    "v_mfma_f32_16x16x32_bf16 a[4:7], v[100:103], v[32:35], a[4:7]\n"
    "s_mov_b64 exec, s[70:71]\n"
    "global_load_dwordx4 v[176:179], v139, s[74:75] sc0 sc1\n"
    "s_mov_b64 exec, -1\n"
    "v_mfma_f32_16x16x32_bf16 a[32:35], v[96:99], v[36:39], a[32:35]\n"
    "ds_read_b64_tr_b16 v[66:67], v144 offset:512\n"
    "s_add_u32 m0, s50, 49152\n"
    "s_mov_b64 exec, s[30:31]\n"
    "global_store_dwordx4 v138, v[164:167], s[26:27] offset:1024 sc0 sc1\n"
    "s_mov_b64 exec, -1\n"
    "v_mfma_f32_16x16x32_bf16 a[36:39], v[100:103], v[36:39], a[36:39]\n"
    "buffer_load_dwordx4 v132, s[44:47], s49 offen offset:0 nt lds\n"
    "v_mfma_f32_16x16x32_bf16 a[64:67], v[96:99], v[40:43], a[64:67]\n"
    "ds_read_b128 v[0:3], v137 offset:0\n"
    "v_mfma_f32_16x16x32_bf16 a[68:71], v[100:103], v[40:43], a[68:71]\n"
    "v_mfma_f32_16x16x32_bf16 a[96:99], v[96:99], v[44:47], a[96:99]\n"
    "ds_read_b128 v[4:7], v137 offset:1024\n"
    "v_mfma_f32_16x16x32_bf16 a[100:103], v[100:103], v[44:47], a[100:103]\n"
    "v_mfma_f32_16x16x32_bf16 a[128:131], v[96:99], v[48:51], a[128:131]\n"
    "ds_read_b128 v[8:11], v137 offset:2048\n"
    "v_mfma_f32_16x16x32_bf16 a[132:135], v[100:103], v[48:51], a[132:135]\n"
    "v_mfma_f32_16x16x32_bf16 a[160:163], v[96:99], v[52:55], a[160:163]\n"
    "ds_read_b128 v[12:15], v137 offset:3072\n"
    "v_mfma_f32_16x16x32_bf16 a[164:167], v[100:103], v[52:55], a[164:167]\n"
    "buffer_load_dwordx4 v133, s[44:47], s49 offen offset:1024 nt lds\n"
    "v_mfma_f32_16x16x32_bf16 a[192:195], v[96:99], v[56:59], a[192:195]\n"
    "ds_read_b128 v[16:19], v137 offset:4096\n"
    "v_mfma_f32_16x16x32_bf16 a[196:199], v[100:103], v[56:59], a[196:199]\n"
    "v_mfma_f32_16x16x32_bf16 a[224:227], v[96:99], v[60:63], a[224:227]\n"
    "ds_read_b128 v[20:23], v137 offset:5120\n"
    "v_mfma_f32_16x16x32_bf16 a[228:231], v[100:103], v[60:63], a[228:231]\n"
    "v_mfma_f32_16x16x32_bf16 a[232:235], v[104:107], v[60:63], a[232:235]\n"
    "ds_read_b128 v[24:27], v137 offset:6144\n"
    "v_mfma_f32_16x16x32_bf16 a[236:239], v[108:111], v[60:63], a[236:239]\n"
    "v_mfma_f32_16x16x32_bf16 a[200:203], v[104:107], v[56:59], a[200:203]\n"
    "ds_read_b128 v[28:31], v137 offset:7168\n"
    "v_mfma_f32_16x16x32_bf16 a[204:207], v[108:111], v[56:59], a[204:207]\n"
    "buffer_load_dwordx4 v134, s[44:47], s49 offen offset:2048 nt lds\n"
    "v_mfma_f32_16x16x32_bf16 a[168:171], v[104:107], v[52:55], a[168:171]\n"
    "ds_read_b64_tr_b16 v[68:69], v145 offset:0\n"
    "v_mfma_f32_16x16x32_bf16 a[172:175], v[108:111], v[52:55], a[172:175]\n"
    "v_mfma_f32_16x16x32_bf16 a[136:139], v[104:107], v[48:51], a[136:139]\n"
    "ds_read_b64_tr_b16 v[70:71], v145 offset:512\n"
    "v_mfma_f32_16x16x32_bf16 a[140:143], v[108:111], v[48:51], a[140:143]\n"
    "v_mfma_f32_16x16x32_bf16 a[104:107], v[104:107], v[44:47], a[104:107]\n"
    "ds_read_b64_tr_b16 v[72:73], v146 offset:0\n"
    "v_mfma_f32_16x16x32_bf16 a[108:111], v[108:111], v[44:47], a[108:111]\n"
    "v_mfma_f32_16x16x32_bf16 a[72:75], v[104:107], v[40:43], a[72:75]\n"
    "ds_read_b64_tr_b16 v[74:75], v146 offset:512\n"
    "v_mfma_f32_16x16x32_bf16 a[76:79], v[108:111], v[40:43], a[76:79]\n"
    "buffer_load_dwordx4 v135, s[44:47], s49 offen offset:3072 nt lds\n"
    "v_mfma_f32_16x16x32_bf16 a[40:43], v[104:107], v[36:39], a[40:43]\n"
    "ds_read_b64_tr_b16 v[76:77], v147 offset:0\n"
    "v_mfma_f32_16x16x32_bf16 a[44:47], v[108:111], v[36:39], a[44:47]\n"
    "s_add_u32 s53, s53, 0x24924925\n"
    "s_cselect_b32 s55, s79, s80\n"
    "v_mfma_f32_16x16x32_bf16 a[8:11], v[104:107], v[32:35], a[8:11]\n"
    "ds_read_b64_tr_b16 v[78:79], v147 offset:512\n"
    "v_mfma_f32_16x16x32_bf16 a[12:15], v[108:111], v[32:35], a[12:15]\n"
    "s_add_u32 s49, s49, s55\n"
    "v_mfma_f32_16x16x32_bf16 a[16:19], v[112:115], v[32:35], a[16:19]\n"
    "ds_read_b64_tr_b16 v[80:81], v144 offset:4096\n"
    "v_mfma_f32_16x16x32_bf16 a[20:23], v[116:119], v[32:35], a[20:23]\n"
    "s_and_b32 s49, s49, 0xfffffff\n"
    "v_mfma_f32_16x16x32_bf16 a[48:51], v[112:115], v[36:39], a[48:51]\n"
    "ds_read_b64_tr_b16 v[82:83], v144 offset:4608\n"
    "s_add_u32 m0, s50, 32768\n"
    "v_mfma_f32_16x16x32_bf16 a[52:55], v[116:119], v[36:39], a[52:55]\n"
    "buffer_load_dwordx4 v138, s[56:59], s52 offen offset:0 sc0 sc1 lds\n"
    "v_mfma_f32_16x16x32_bf16 a[80:83], v[112:115], v[40:43], a[80:83]\n"
    "ds_read_b64_tr_b16 v[84:85], v145 offset:4096\n"
    "v_mfma_f32_16x16x32_bf16 a[84:87], v[116:119], v[40:43], a[84:87]\n"
    "v_mfma_f32_16x16x32_bf16 a[112:115], v[112:115], v[44:47], a[112:115]\n"
    "ds_read_b64_tr_b16 v[86:87], v145 offset:4608\n"
    "v_mfma_f32_16x16x32_bf16 a[116:119], v[116:119], v[44:47], a[116:119]\n"
    "s_mov_b32 s54, s25\n"
    "v_mfma_f32_16x16x32_bf16 a[144:147], v[112:115], v[48:51], a[144:147]\n"
    "ds_read_b64_tr_b16 v[88:89], v146 offset:4096\n"
    "v_mfma_f32_16x16x32_bf16 a[148:151], v[116:119], v[48:51], a[148:151]\n"
    "s_cmp_eq_u32 s67, 8\n"
    "s_cselect_b32 s24, s54, s24\n"
    "s_cselect_b32 s35, s54, s35\n"
    "v_mfma_f32_16x16x32_bf16 a[176:179], v[112:115], v[52:55], a[176:179]\n"
    "ds_read_b64_tr_b16 v[90:91], v146 offset:4608\n"
    "v_mfma_f32_16x16x32_bf16 a[180:183], v[116:119], v[52:55], a[180:183]\n"
    "buffer_load_dwordx4 v138, s[56:59], s52 offen offset:1024 sc0 sc1 lds\n"
    "v_mfma_f32_16x16x32_bf16 a[208:211], v[112:115], v[56:59], a[208:211]\n"
    "ds_read_b64_tr_b16 v[92:93], v147 offset:4096\n"
    "v_mfma_f32_16x16x32_bf16 a[212:215], v[116:119], v[56:59], a[212:215]\n"
    "v_mfma_f32_16x16x32_bf16 a[240:243], v[112:115], v[60:63], a[240:243]\n"
    "ds_read_b64_tr_b16 v[94:95], v147 offset:4608\n"
    "v_mfma_f32_16x16x32_bf16 a[244:247], v[116:119], v[60:63], a[244:247]\n"
    "v_mfma_f32_16x16x32_bf16 a[248:251], v[120:123], v[60:63], a[248:251]\n"
    "v_mfma_f32_16x16x32_bf16 a[252:255], v[124:127], v[60:63], a[252:255]\n"
    "v_mfma_f32_16x16x32_bf16 a[216:219], v[120:123], v[56:59], a[216:219]\n"
    "v_mfma_f32_16x16x32_bf16 a[220:223], v[124:127], v[56:59], a[220:223]\n"
    "buffer_load_dwordx4 v138, s[56:59], s52 offen offset:2048 sc0 sc1 lds\n"
    "v_mfma_f32_16x16x32_bf16 a[184:187], v[120:123], v[52:55], a[184:187]\n"
    "v_mfma_f32_16x16x32_bf16 a[188:191], v[124:127], v[52:55], a[188:191]\n"
    "v_mfma_f32_16x16x32_bf16 a[152:155], v[120:123], v[48:51], a[152:155]\n"
    "s_memrealtime s[28:29]\n"
    "v_mfma_f32_16x16x32_bf16 a[156:159], v[124:127], v[48:51], a[156:159]\n"
    "v_mfma_f32_16x16x32_bf16 a[120:123], v[120:123], v[44:47], a[120:123]\n"
    "v_mfma_f32_16x16x32_bf16 a[124:127], v[124:127], v[44:47], a[124:127]\n"
    "v_mfma_f32_16x16x32_bf16 a[88:91], v[120:123], v[40:43], a[88:91]\n"
    "v_mfma_f32_16x16x32_bf16 a[92:95], v[124:127], v[40:43], a[92:95]\n"
    "buffer_load_dwordx4 v138, s[56:59], s52 offen offset:3072 sc0 sc1 lds\n"
    "v_mfma_f32_16x16x32_bf16 a[56:59], v[120:123], v[36:39], a[56:59]\n"
    "s_add_u32 s52, s52, 0x4000\n"
    "v_mfma_f32_16x16x32_bf16 a[60:63], v[124:127], v[36:39], a[60:63]\n"
    "v_mfma_f32_16x16x32_bf16 a[24:27], v[120:123], v[32:35], a[24:27]\n"
    "s_waitcnt vmcnt(18) lgkmcnt(0)\n"
    "v_mfma_f32_16x16x32_bf16 a[28:31], v[124:127], v[32:35], a[28:31]\n"
    "s_barrier\n"
    "v_mfma_f32_16x16x32_bf16 a[0:3], v[64:67], v[0:3], a[0:3]\n"
    "ds_read_b64_tr_b16 v[96:97], v144 offset:32768\n"
    "v_mfma_f32_16x16x32_bf16 a[4:7], v[68:71], v[0:3], a[4:7]\n"
    "v_mfma_f32_16x16x32_bf16 a[32:35], v[64:67], v[4:7], a[32:35]\n"
    "ds_read_b64_tr_b16 v[98:99], v144 offset:33280\n"
    "s_add_u32 m0, s50, 81920\n"
    "s_mov_b64 exec, s[30:31]\n"
    "global_store_dwordx4 v138, v[168:171], s[26:27] offset:2048 sc0 sc1\n"
    "s_mov_b64 exec, -1\n"
    "v_mfma_f32_16x16x32_bf16 a[36:39], v[68:71], v[4:7], a[36:39]\n"
    "buffer_load_dwordx4 v132, s[44:47], s49 offen offset:0 nt lds\n"
    "v_mfma_f32_16x16x32_bf16 a[64:67], v[64:67], v[8:11], a[64:67]\n"
    "ds_read_b128 v[32:35], v137 offset:32768\n"
    "v_mfma_f32_16x16x32_bf16 a[68:71], v[68:71], v[8:11], a[68:71]\n"
    "s_lshr_b32 s54, s24, 4\n"
    "s_add_u32 s33, s33, s54\n"
    "v_mfma_f32_16x16x32_bf16 a[96:99], v[64:67], v[12:15], a[96:99]\n"
    "ds_read_b128 v[36:39], v137 offset:33792\n"
    "s_add_u32 s32, s32, s54\n"
    "v_mfma_f32_16x16x32_bf16 a[100:103], v[68:71], v[12:15], a[100:103]\n"
    "s_cmp_eq_u32 s73, s67\n"
    "s_cselect_b64 exec, 1, 0\n"
    "global_store_dword v148, v152, s[64:65] sc0 sc1\n"
    "s_mov_b64 exec, -1\n"
    "s_sub_u32 s54, s33, s28\n"
    "s_max_i32 s54, s54, -250\n"
    "v_mfma_f32_16x16x32_bf16 a[128:131], v[64:67], v[16:19], a[128:131]\n"
    "ds_read_b128 v[40:43], v137 offset:34816\n"
    "v_mfma_f32_16x16x32_bf16 a[132:135], v[68:71], v[16:19], a[132:135]\n"
    "s_add_u32 s33, s28, s54\n"
    "s_cmp_gt_i32 s54, 0\n"
    "v_mfma_f32_16x16x32_bf16 a[160:163], v[64:67], v[20:23], a[160:163]\n"
    "ds_read_b128 v[44:47], v137 offset:35840\n"
    "s_cbranch_scc1 PACEA2_%=\n"
    "PACEBA2_%=:\n"
    "v_mfma_f32_16x16x32_bf16 a[164:167], v[68:71], v[20:23], a[164:167]\n"
    "buffer_load_dwordx4 v133, s[44:47], s49 offen offset:1024 nt lds\n"
    "v_mfma_f32_16x16x32_bf16 a[192:195], v[64:67], v[24:27], a[192:195]\n"
    "ds_read_b128 v[48:51], v137 offset:36864\n"
    "v_mfma_f32_16x16x32_bf16 a[196:199], v[68:71], v[24:27], a[196:199]\n"
    "s_sub_u32 s24, s24, 16\n"
    "s_max_i32 s24, s24, s35\n"
    "v_mfma_f32_16x16x32_bf16 a[224:227], v[64:67], v[28:31], a[224:227]\n"
    "ds_read_b128 v[52:55], v137 offset:37888\n"
    "v_mfma_f32_16x16x32_bf16 a[228:231], v[68:71], v[28:31], a[228:231]\n"
    "v_mfma_f32_16x16x32_bf16 a[232:235], v[72:75], v[28:31], a[232:235]\n"
    "ds_read_b128 v[56:59], v137 offset:38912\n"
    "v_mfma_f32_16x16x32_bf16 a[236:239], v[76:79], v[28:31], a[236:239]\n"
    "v_mfma_f32_16x16x32_bf16 a[200:203], v[72:75], v[24:27], a[200:203]\n"
    "ds_read_b128 v[60:63], v137 offset:39936\n"
    "v_mfma_f32_16x16x32_bf16 a[204:207], v[76:79], v[24:27], a[204:207]\n"
    "buffer_load_dwordx4 v134, s[44:47], s49 offen offset:2048 nt lds\n"
    "v_mfma_f32_16x16x32_bf16 a[168:171], v[72:75], v[20:23], a[168:171]\n"
    "ds_read_b64_tr_b16 v[100:101], v145 offset:32768\n"
    "v_mfma_f32_16x16x32_bf16 a[172:175], v[76:79], v[20:23], a[172:175]\n"
    "v_mfma_f32_16x16x32_bf16 a[136:139], v[72:75], v[16:19], a[136:139]\n"
    "ds_read_b64_tr_b16 v[102:103], v145 offset:33280\n"
    "v_mfma_f32_16x16x32_bf16 a[140:143], v[76:79], v[16:19], a[140:143]\n"
    "v_mfma_f32_16x16x32_bf16 a[104:107], v[72:75], v[12:15], a[104:107]\n"
    "ds_read_b64_tr_b16 v[104:105], v146 offset:32768\n"
    "v_mfma_f32_16x16x32_bf16 a[108:111], v[76:79], v[12:15], a[108:111]\n"
    "v_mfma_f32_16x16x32_bf16 a[72:75], v[72:75], v[8:11], a[72:75]\n"
    "ds_read_b64_tr_b16 v[106:107], v146 offset:33280\n"
    "v_mfma_f32_16x16x32_bf16 a[76:79], v[76:79], v[8:11], a[76:79]\n"
    "buffer_load_dwordx4 v135, s[44:47], s49 offen offset:3072 nt lds\n"
    "v_mfma_f32_16x16x32_bf16 a[40:43], v[72:75], v[4:7], a[40:43]\n"
    "ds_read_b64_tr_b16 v[108:109], v147 offset:32768\n"
    "v_mfma_f32_16x16x32_bf16 a[44:47], v[76:79], v[4:7], a[44:47]\n"
    "s_add_u32 s53, s53, 0x24924925\n"
    "s_cselect_b32 s55, s79, s80\n"
    "v_mfma_f32_16x16x32_bf16 a[8:11], v[72:75], v[0:3], a[8:11]\n"
    "ds_read_b64_tr_b16 v[110:111], v147 offset:33280\n"
    "v_mfma_f32_16x16x32_bf16 a[12:15], v[76:79], v[0:3], a[12:15]\n"
    "s_add_u32 s49, s49, s55\n"
    "v_mfma_f32_16x16x32_bf16 a[16:19], v[80:83], v[0:3], a[16:19]\n"
    "ds_read_b64_tr_b16 v[112:113], v144 offset:36864\n"
    "v_mfma_f32_16x16x32_bf16 a[20:23], v[84:87], v[0:3], a[20:23]\n"
    "s_and_b32 s49, s49, 0xfffffff\n"
    "v_mfma_f32_16x16x32_bf16 a[48:51], v[80:83], v[4:7], a[48:51]\n"
    "ds_read_b64_tr_b16 v[114:115], v144 offset:37376\n"
    "s_add_u32 m0, s50, 65536\n"
    "v_mfma_f32_16x16x32_bf16 a[52:55], v[84:87], v[4:7], a[52:55]\n"
    "buffer_load_dwordx4 v138, s[56:59], s52 offen offset:0 sc0 sc1 lds\n"
    "v_mfma_f32_16x16x32_bf16 a[80:83], v[80:83], v[8:11], a[80:83]\n"
    "ds_read_b64_tr_b16 v[116:117], v145 offset:36864\n"
    "v_mfma_f32_16x16x32_bf16 a[84:87], v[84:87], v[8:11], a[84:87]\n"
    "v_mfma_f32_16x16x32_bf16 a[112:115], v[80:83], v[12:15], a[112:115]\n"
    "ds_read_b64_tr_b16 v[118:119], v145 offset:37376\n"
    "v_mfma_f32_16x16x32_bf16 a[116:119], v[84:87], v[12:15], a[116:119]\n"
    "v_mfma_f32_16x16x32_bf16 a[144:147], v[80:83], v[16:19], a[144:147]\n"
    "ds_read_b64_tr_b16 v[120:121], v146 offset:36864\n"
    "v_mfma_f32_16x16x32_bf16 a[148:151], v[84:87], v[16:19], a[148:151]\n"
    "v_mfma_f32_16x16x32_bf16 a[176:179], v[80:83], v[20:23], a[176:179]\n"
    "ds_read_b64_tr_b16 v[122:123], v146 offset:37376\n"
    "v_mfma_f32_16x16x32_bf16 a[180:183], v[84:87], v[20:23], a[180:183]\n"
    "buffer_load_dwordx4 v138, s[56:59], s52 offen offset:1024 sc0 sc1 lds\n"
    "v_mfma_f32_16x16x32_bf16 a[208:211], v[80:83], v[24:27], a[208:211]\n"
    "ds_read_b64_tr_b16 v[124:125], v147 offset:36864\n"
    "v_mfma_f32_16x16x32_bf16 a[212:215], v[84:87], v[24:27], a[212:215]\n"
    "v_mfma_f32_16x16x32_bf16 a[240:243], v[80:83], v[28:31], a[240:243]\n"
    "ds_read_b64_tr_b16 v[126:127], v147 offset:37376\n"
    "v_mfma_f32_16x16x32_bf16 a[244:247], v[84:87], v[28:31], a[244:247]\n"
    "v_mfma_f32_16x16x32_bf16 a[248:251], v[88:91], v[28:31], a[248:251]\n"
    "v_mfma_f32_16x16x32_bf16 a[252:255], v[92:95], v[28:31], a[252:255]\n"
    "v_mfma_f32_16x16x32_bf16 a[216:219], v[88:91], v[24:27], a[216:219]\n"
    "v_mfma_f32_16x16x32_bf16 a[220:223], v[92:95], v[24:27], a[220:223]\n"
    "buffer_load_dwordx4 v138, s[56:59], s52 offen offset:2048 sc0 sc1 lds\n"
    "v_mfma_f32_16x16x32_bf16 a[184:187], v[88:91], v[20:23], a[184:187]\n"
    "v_mfma_f32_16x16x32_bf16 a[188:191], v[92:95], v[20:23], a[188:191]\n"
    "v_mfma_f32_16x16x32_bf16 a[152:155], v[88:91], v[16:19], a[152:155]\n"
    "v_mfma_f32_16x16x32_bf16 a[156:159], v[92:95], v[16:19], a[156:159]\n"
    "v_mfma_f32_16x16x32_bf16 a[120:123], v[88:91], v[12:15], a[120:123]\n"
    "v_mfma_f32_16x16x32_bf16 a[124:127], v[92:95], v[12:15], a[124:127]\n"
    "v_mfma_f32_16x16x32_bf16 a[88:91], v[88:91], v[8:11], a[88:91]\n"
    "v_mfma_f32_16x16x32_bf16 a[92:95], v[92:95], v[8:11], a[92:95]\n"
    "buffer_load_dwordx4 v138, s[56:59], s52 offen offset:3072 sc0 sc1 lds\n"
    "v_mfma_f32_16x16x32_bf16 a[56:59], v[88:91], v[4:7], a[56:59]\n"
    "s_add_u32 s52, s52, 0x4000\n"
    "v_mfma_f32_16x16x32_bf16 a[60:63], v[92:95], v[4:7], a[60:63]\n"
    "v_mfma_f32_16x16x32_bf16 a[24:27], v[88:91], v[0:3], a[24:27]\n"
    "s_waitcnt vmcnt(18) lgkmcnt(0)\n"
    "v_mfma_f32_16x16x32_bf16 a[28:31], v[92:95], v[0:3], a[28:31]\n"
    "s_barrier\n"
    "v_mfma_f32_16x16x32_bf16 a[0:3], v[96:99], v[32:35], a[0:3]\n"
    "ds_read_b64_tr_b16 v[64:65], v140 offset:0\n"
    "v_mfma_f32_16x16x32_bf16 a[4:7], v[100:103], v[32:35], a[4:7]\n"
    "v_min_u32 v176, v176, v177\n"
    "v_mfma_f32_16x16x32_bf16 a[32:35], v[96:99], v[36:39], a[32:35]\n"
    "ds_read_b64_tr_b16 v[66:67], v140 offset:512\n"
    "s_add_u32 m0, s50, 114688\n"
    "s_mov_b64 exec, s[30:31]\n"
    "global_store_dwordx4 v138, v[172:175], s[26:27] offset:3072 sc0 sc1\n"
    "s_mov_b64 exec, -1\n"
    "v_mfma_f32_16x16x32_bf16 a[36:39], v[100:103], v[36:39], a[36:39]\n"
    "buffer_load_dwordx4 v132, s[44:47], s49 offen offset:0 nt lds\n"
    "v_mfma_f32_16x16x32_bf16 a[64:67], v[96:99], v[40:43], a[64:67]\n"
    "ds_read_b128 v[0:3], v136 offset:0\n"
    "v_mfma_f32_16x16x32_bf16 a[68:71], v[100:103], v[40:43], a[68:71]\n"
    "v_min3_u32 v176, v176, v178, v179\n"
    "v_mfma_f32_16x16x32_bf16 a[96:99], v[96:99], v[44:47], a[96:99]\n"
    "ds_read_b128 v[4:7], v136 offset:1024\n"
    "v_mfma_f32_16x16x32_bf16 a[100:103], v[100:103], v[44:47], a[100:103]\n"
    "v_cmp_le_u32 vcc, s68, v176\n"
    "v_mfma_f32_16x16x32_bf16 a[128:131], v[96:99], v[48:51], a[128:131]\n"
    "ds_read_b128 v[8:11], v136 offset:2048\n"
    "v_mfma_f32_16x16x32_bf16 a[132:135], v[100:103], v[48:51], a[132:135]\n"
    "v_mfma_f32_16x16x32_bf16 a[160:163], v[96:99], v[52:55], a[160:163]\n"
    "ds_read_b128 v[12:15], v136 offset:3072\n"
    "v_mfma_f32_16x16x32_bf16 a[164:167], v[100:103], v[52:55], a[164:167]\n"
    "buffer_load_dwordx4 v133, s[44:47], s49 offen offset:1024 nt lds\n"
    "v_mfma_f32_16x16x32_bf16 a[192:195], v[96:99], v[56:59], a[192:195]\n"
    "ds_read_b128 v[16:19], v136 offset:4096\n"
    "v_mfma_f32_16x16x32_bf16 a[196:199], v[100:103], v[56:59], a[196:199]\n"
    "s_and_b32 s54, vcc_lo, s70\n"
    "s_or_b32 s69, s69, s54\n"
    "v_mfma_f32_16x16x32_bf16 a[224:227], v[96:99], v[60:63], a[224:227]\n"
    "ds_read_b128 v[20:23], v136 offset:5120\n"
    "v_mfma_f32_16x16x32_bf16 a[228:231], v[100:103], v[60:63], a[228:231]\n"
    "s_andn2_b32 s70, s70, s54\n"
    "s_bitcmp1_b32 s69, s72\n"
    "v_mfma_f32_16x16x32_bf16 a[232:235], v[104:107], v[60:63], a[232:235]\n"
    "ds_read_b128 v[24:27], v136 offset:6144\n"
    "v_mfma_f32_16x16x32_bf16 a[236:239], v[108:111], v[60:63], a[236:239]\n"
    "s_cbranch_scc0 SLOWA_%=\n"
    "SLOWBACKA_%=:\n"
    "s_add_u32 s72, s72, 1\n"
    "v_mfma_f32_16x16x32_bf16 a[200:203], v[104:107], v[56:59], a[200:203]\n"
    "ds_read_b128 v[28:31], v136 offset:7168\n"
    "v_mfma_f32_16x16x32_bf16 a[204:207], v[108:111], v[56:59], a[204:207]\n"
    "buffer_load_dwordx4 v134, s[44:47], s49 offen offset:2048 nt lds\n"
    "v_mfma_f32_16x16x32_bf16 a[168:171], v[104:107], v[52:55], a[168:171]\n"
    "ds_read_b64_tr_b16 v[68:69], v141 offset:0\n"
    "v_mfma_f32_16x16x32_bf16 a[172:175], v[108:111], v[52:55], a[172:175]\n"
    "v_mfma_f32_16x16x32_bf16 a[136:139], v[104:107], v[48:51], a[136:139]\n"
    "ds_read_b64_tr_b16 v[70:71], v141 offset:512\n"
    "v_mfma_f32_16x16x32_bf16 a[140:143], v[108:111], v[48:51], a[140:143]\n"
    "v_mfma_f32_16x16x32_bf16 a[104:107], v[104:107], v[44:47], a[104:107]\n"
    "ds_read_b64_tr_b16 v[72:73], v142 offset:0\n"
    "v_mfma_f32_16x16x32_bf16 a[108:111], v[108:111], v[44:47], a[108:111]\n"
    "v_mfma_f32_16x16x32_bf16 a[72:75], v[104:107], v[40:43], a[72:75]\n"
    "ds_read_b64_tr_b16 v[74:75], v142 offset:512\n"
    "v_mfma_f32_16x16x32_bf16 a[76:79], v[108:111], v[40:43], a[76:79]\n"
    "buffer_load_dwordx4 v135, s[44:47], s49 offen offset:3072 nt lds\n"
    "v_mfma_f32_16x16x32_bf16 a[40:43], v[104:107], v[36:39], a[40:43]\n"
    "ds_read_b64_tr_b16 v[76:77], v143 offset:0\n"
    "v_mfma_f32_16x16x32_bf16 a[44:47], v[108:111], v[36:39], a[44:47]\n"
    "s_add_u32 s53, s53, 0x24924925\n"
    "s_cselect_b32 s55, s79, s80\n"
    "v_mfma_f32_16x16x32_bf16 a[8:11], v[104:107], v[32:35], a[8:11]\n"
    "ds_read_b64_tr_b16 v[78:79], v143 offset:512\n"
    "v_mfma_f32_16x16x32_bf16 a[12:15], v[108:111], v[32:35], a[12:15]\n"
    "s_add_u32 s49, s49, s55\n"
    "v_mfma_f32_16x16x32_bf16 a[16:19], v[112:115], v[32:35], a[16:19]\n"
    "ds_read_b64_tr_b16 v[80:81], v140 offset:4096\n"
    "v_mfma_f32_16x16x32_bf16 a[20:23], v[116:119], v[32:35], a[20:23]\n"
    "s_and_b32 s49, s49, 0xfffffff\n"
    "v_mfma_f32_16x16x32_bf16 a[48:51], v[112:115], v[36:39], a[48:51]\n"
    "ds_read_b64_tr_b16 v[82:83], v140 offset:4608\n"
    "s_add_u32 m0, s50, 98304\n"
    "v_mfma_f32_16x16x32_bf16 a[52:55], v[116:119], v[36:39], a[52:55]\n"
    "buffer_load_dwordx4 v138, s[56:59], s52 offen offset:0 sc0 sc1 lds\n"
    "v_mfma_f32_16x16x32_bf16 a[80:83], v[112:115], v[40:43], a[80:83]\n"
    "ds_read_b64_tr_b16 v[84:85], v141 offset:4096\n"
    "v_mfma_f32_16x16x32_bf16 a[84:87], v[116:119], v[40:43], a[84:87]\n"
    "v_mfma_f32_16x16x32_bf16 a[112:115], v[112:115], v[44:47], a[112:115]\n"
    "ds_read_b64_tr_b16 v[86:87], v141 offset:4608\n"
    "v_mfma_f32_16x16x32_bf16 a[116:119], v[116:119], v[44:47], a[116:119]\n"
    "v_mfma_f32_16x16x32_bf16 a[144:147], v[112:115], v[48:51], a[144:147]\n"
    "ds_read_b64_tr_b16 v[88:89], v142 offset:4096\n"
    "v_mfma_f32_16x16x32_bf16 a[148:151], v[116:119], v[48:51], a[148:151]\n"
    "v_mfma_f32_16x16x32_bf16 a[176:179], v[112:115], v[52:55], a[176:179]\n"
    "ds_read_b64_tr_b16 v[90:91], v142 offset:4608\n"
    "v_mfma_f32_16x16x32_bf16 a[180:183], v[116:119], v[52:55], a[180:183]\n"
    "buffer_load_dwordx4 v138, s[56:59], s52 offen offset:1024 sc0 sc1 lds\n"
    "v_mfma_f32_16x16x32_bf16 a[208:211], v[112:115], v[56:59], a[208:211]\n"
    "ds_read_b64_tr_b16 v[92:93], v143 offset:4096\n"
    "v_mfma_f32_16x16x32_bf16 a[212:215], v[116:119], v[56:59], a[212:215]\n"
    "v_mfma_f32_16x16x32_bf16 a[240:243], v[112:115], v[60:63], a[240:243]\n"
    "ds_read_b64_tr_b16 v[94:95], v143 offset:4608\n"
    "v_mfma_f32_16x16x32_bf16 a[244:247], v[116:119], v[60:63], a[244:247]\n"
    "v_mfma_f32_16x16x32_bf16 a[248:251], v[120:123], v[60:63], a[248:251]\n"
    "v_mfma_f32_16x16x32_bf16 a[252:255], v[124:127], v[60:63], a[252:255]\n"
    "v_mfma_f32_16x16x32_bf16 a[216:219], v[120:123], v[56:59], a[216:219]\n"
    "v_mfma_f32_16x16x32_bf16 a[220:223], v[124:127], v[56:59], a[220:223]\n"
    "buffer_load_dwordx4 v138, s[56:59], s52 offen offset:2048 sc0 sc1 lds\n"
    "v_mfma_f32_16x16x32_bf16 a[184:187], v[120:123], v[52:55], a[184:187]\n"
    "v_mfma_f32_16x16x32_bf16 a[188:191], v[124:127], v[52:55], a[188:191]\n"
    "v_mfma_f32_16x16x32_bf16 a[152:155], v[120:123], v[48:51], a[152:155]\n"
    "s_memrealtime s[28:29]\n"
    "v_mfma_f32_16x16x32_bf16 a[156:159], v[124:127], v[48:51], a[156:159]\n"
    "v_mfma_f32_16x16x32_bf16 a[120:123], v[120:123], v[44:47], a[120:123]\n"
    "v_mfma_f32_16x16x32_bf16 a[124:127], v[124:127], v[44:47], a[124:127]\n"
    "v_mfma_f32_16x16x32_bf16 a[88:91], v[120:123], v[40:43], a[88:91]\n"
    "v_mfma_f32_16x16x32_bf16 a[92:95], v[124:127], v[40:43], a[92:95]\n"
    "buffer_load_dwordx4 v138, s[56:59], s52 offen offset:3072 sc0 sc1 lds\n"
    "v_mfma_f32_16x16x32_bf16 a[56:59], v[120:123], v[36:39], a[56:59]\n"
    "s_add_u32 s52, s52, 0x4000\n"
    "v_mfma_f32_16x16x32_bf16 a[60:63], v[124:127], v[36:39], a[60:63]\n"
    "s_cmp_lg_u32 s51, 0\n"
    "v_mfma_f32_16x16x32_bf16 a[24:27], v[120:123], v[32:35], a[24:27]\n"
    "s_waitcnt vmcnt(18) lgkmcnt(0)\n"
    "v_mfma_f32_16x16x32_bf16 a[28:31], v[124:127], v[32:35], a[28:31]\n"
    "s_barrier\n"
    "s_cbranch_scc1 LOOPA_%=\n"
    "s_mov_b32 s51, 22\n"
    "LOOPB_%=:\n"
    "v_mfma_f32_16x16x32_bf16 a[0:3], v[64:67], v[0:3], a[0:3]\n"
    "ds_read_b64_tr_b16 v[96:97], v140 offset:32768\n"
    "s_add_u32 s67, s67, 1\n"
    "v_mfma_f32_16x16x32_bf16 a[4:7], v[68:71], v[0:3], a[4:7]\n"
    "v_mfma_f32_16x16x32_bf16 a[32:35], v[64:67], v[4:7], a[32:35]\n"
    "ds_read_b64_tr_b16 v[98:99], v140 offset:33280\n"
    "s_add_u32 m0, s50, 16384\n"
    "v_mfma_f32_16x16x32_bf16 a[36:39], v[68:71], v[4:7], a[36:39]\n"
    "buffer_load_dwordx4 v132, s[44:47], s49 offen offset:0 nt lds\n"
    "v_mfma_f32_16x16x32_bf16 a[64:67], v[64:67], v[8:11], a[64:67]\n"
    "ds_read_b128 v[32:35], v136 offset:32768\n"
    "v_mfma_f32_16x16x32_bf16 a[68:71], v[68:71], v[8:11], a[68:71]\n"
    "s_lshr_b32 s54, s24, 4\n"
    "s_add_u32 s33, s33, s54\n"
    "v_mfma_f32_16x16x32_bf16 a[96:99], v[64:67], v[12:15], a[96:99]\n"
    "ds_read_b128 v[36:39], v136 offset:33792\n"
    "s_add_u32 s32, s32, s54\n"
    "v_mfma_f32_16x16x32_bf16 a[100:103], v[68:71], v[12:15], a[100:103]\n"
    "s_sub_u32 s54, s33, s28\n"
    "s_max_i32 s54, s54, -250\n"
    "v_mfma_f32_16x16x32_bf16 a[128:131], v[64:67], v[16:19], a[128:131]\n"
    "ds_read_b128 v[40:43], v136 offset:34816\n"
    "v_mfma_f32_16x16x32_bf16 a[132:135], v[68:71], v[16:19], a[132:135]\n"
    "s_add_u32 s33, s28, s54\n"
    "s_cmp_gt_i32 s54, 0\n"
    "v_mfma_f32_16x16x32_bf16 a[160:163], v[64:67], v[20:23], a[160:163]\n"
    "ds_read_b128 v[44:47], v136 offset:35840\n"
    "s_cbranch_scc1 PACEB0_%=\n"
    "PACEBB0_%=:\n"
    "v_mfma_f32_16x16x32_bf16 a[164:167], v[68:71], v[20:23], a[164:167]\n"
    "buffer_load_dwordx4 v133, s[44:47], s49 offen offset:1024 nt lds\n"
    "v_mfma_f32_16x16x32_bf16 a[192:195], v[64:67], v[24:27], a[192:195]\n"
    "ds_read_b128 v[48:51], v136 offset:36864\n"
    "v_mfma_f32_16x16x32_bf16 a[196:199], v[68:71], v[24:27], a[196:199]\n"
    "s_sub_u32 s24, s24, 16\n"
    "s_max_i32 s24, s24, s35\n"
    "v_mfma_f32_16x16x32_bf16 a[224:227], v[64:67], v[28:31], a[224:227]\n"
    "ds_read_b128 v[52:55], v136 offset:37888\n"
    "v_mfma_f32_16x16x32_bf16 a[228:231], v[68:71], v[28:31], a[228:231]\n"
    "v_mfma_f32_16x16x32_bf16 a[232:235], v[72:75], v[28:31], a[232:235]\n"
    "ds_read_b128 v[56:59], v136 offset:38912\n"
    "v_mfma_f32_16x16x32_bf16 a[236:239], v[76:79], v[28:31], a[236:239]\n"
    "v_mfma_f32_16x16x32_bf16 a[200:203], v[72:75], v[24:27], a[200:203]\n"
    "ds_read_b128 v[60:63], v136 offset:39936\n"
    "v_mfma_f32_16x16x32_bf16 a[204:207], v[76:79], v[24:27], a[204:207]\n"
    "buffer_load_dwordx4 v134, s[44:47], s49 offen offset:2048 nt lds\n"
    "v_mfma_f32_16x16x32_bf16 a[168:171], v[72:75], v[20:23], a[168:171]\n"
    "ds_read_b64_tr_b16 v[100:101], v141 offset:32768\n"
    "v_mfma_f32_16x16x32_bf16 a[172:175], v[76:79], v[20:23], a[172:175]\n"
    "v_mfma_f32_16x16x32_bf16 a[136:139], v[72:75], v[16:19], a[136:139]\n"
    "ds_read_b64_tr_b16 v[102:103], v141 offset:33280\n"
    "v_mfma_f32_16x16x32_bf16 a[140:143], v[76:79], v[16:19], a[140:143]\n"
    "v_mfma_f32_16x16x32_bf16 a[104:107], v[72:75], v[12:15], a[104:107]\n"
    "ds_read_b64_tr_b16 v[104:105], v142 offset:32768\n"
    "v_mfma_f32_16x16x32_bf16 a[108:111], v[76:79], v[12:15], a[108:111]\n"
    "s_sub_u32 s51, s51, 1\n"
    "v_mfma_f32_16x16x32_bf16 a[72:75], v[72:75], v[8:11], a[72:75]\n"
    "ds_read_b64_tr_b16 v[106:107], v142 offset:33280\n"
    "v_mfma_f32_16x16x32_bf16 a[76:79], v[76:79], v[8:11], a[76:79]\n"
    "buffer_load_dwordx4 v135, s[44:47], s49 offen offset:3072 nt lds\n"
    "v_mfma_f32_16x16x32_bf16 a[40:43], v[72:75], v[4:7], a[40:43]\n"
    "ds_read_b64_tr_b16 v[108:109], v143 offset:32768\n"
    "v_mfma_f32_16x16x32_bf16 a[44:47], v[76:79], v[4:7], a[44:47]\n"
    "s_add_u32 s53, s53, 0x24924925\n"
    "s_cselect_b32 s55, s79, s80\n"
    "v_mfma_f32_16x16x32_bf16 a[8:11], v[72:75], v[0:3], a[8:11]\n"
    "ds_read_b64_tr_b16 v[110:111], v143 offset:33280\n"
    "v_mfma_f32_16x16x32_bf16 a[12:15], v[76:79], v[0:3], a[12:15]\n"
    "s_add_u32 s49, s49, s55\n"
    "v_mfma_f32_16x16x32_bf16 a[16:19], v[80:83], v[0:3], a[16:19]\n"
    "ds_read_b64_tr_b16 v[112:113], v140 offset:36864\n"
    "v_mfma_f32_16x16x32_bf16 a[20:23], v[84:87], v[0:3], a[20:23]\n"
    "s_and_b32 s49, s49, 0xfffffff\n"
    "v_mfma_f32_16x16x32_bf16 a[48:51], v[80:83], v[4:7], a[48:51]\n"
    "ds_read_b64_tr_b16 v[114:115], v140 offset:37376\n"
    "s_add_u32 m0, s50, 0\n"
    "v_mfma_f32_16x16x32_bf16 a[52:55], v[84:87], v[4:7], a[52:55]\n"
    "buffer_load_dwordx4 v138, s[56:59], s52 offen offset:0 sc0 sc1 lds\n"
    "v_mfma_f32_16x16x32_bf16 a[80:83], v[80:83], v[8:11], a[80:83]\n"
    "ds_read_b64_tr_b16 v[116:117], v141 offset:36864\n"
    "v_mfma_f32_16x16x32_bf16 a[84:87], v[84:87], v[8:11], a[84:87]\n"
    "v_mfma_f32_16x16x32_bf16 a[112:115], v[80:83], v[12:15], a[112:115]\n"
    "ds_read_b64_tr_b16 v[118:119], v141 offset:37376\n"
    "v_mfma_f32_16x16x32_bf16 a[116:119], v[84:87], v[12:15], a[116:119]\n"
    "v_mfma_f32_16x16x32_bf16 a[144:147], v[80:83], v[16:19], a[144:147]\n"
    "ds_read_b64_tr_b16 v[120:121], v142 offset:36864\n"
    "v_mfma_f32_16x16x32_bf16 a[148:151], v[84:87], v[16:19], a[148:151]\n"
    "v_mfma_f32_16x16x32_bf16 a[176:179], v[80:83], v[20:23], a[176:179]\n"
    "ds_read_b64_tr_b16 v[122:123], v142 offset:37376\n"
    "v_mfma_f32_16x16x32_bf16 a[180:183], v[84:87], v[20:23], a[180:183]\n"
    "buffer_load_dwordx4 v138, s[56:59], s52 offen offset:1024 sc0 sc1 lds\n"
    "v_mfma_f32_16x16x32_bf16 a[208:211], v[80:83], v[24:27], a[208:211]\n"
    "ds_read_b64_tr_b16 v[124:125], v143 offset:36864\n"
    "v_mfma_f32_16x16x32_bf16 a[212:215], v[84:87], v[24:27], a[212:215]\n"
    "v_mfma_f32_16x16x32_bf16 a[240:243], v[80:83], v[28:31], a[240:243]\n"
    "ds_read_b64_tr_b16 v[126:127], v143 offset:37376\n"
    "v_mfma_f32_16x16x32_bf16 a[244:247], v[84:87], v[28:31], a[244:247]\n"
    "v_mfma_f32_16x16x32_bf16 a[248:251], v[88:91], v[28:31], a[248:251]\n"
    "v_mfma_f32_16x16x32_bf16 a[252:255], v[92:95], v[28:31], a[252:255]\n"
    "v_mfma_f32_16x16x32_bf16 a[216:219], v[88:91], v[24:27], a[216:219]\n"
    "v_mfma_f32_16x16x32_bf16 a[220:223], v[92:95], v[24:27], a[220:223]\n"
    "buffer_load_dwordx4 v138, s[56:59], s52 offen offset:2048 sc0 sc1 lds\n"
    "v_mfma_f32_16x16x32_bf16 a[184:187], v[88:91], v[20:23], a[184:187]\n"
    "v_mfma_f32_16x16x32_bf16 a[188:191], v[92:95], v[20:23], a[188:191]\n"
    "s_mov_b64 exec, s[70:71]\n"
    "v_mfma_f32_16x16x32_bf16 a[152:155], v[88:91], v[16:19], a[152:155]\n"
    "global_load_dwordx4 v[176:179], v139, s[74:75] sc0 sc1\n"
    "v_mfma_f32_16x16x32_bf16 a[156:159], v[92:95], v[16:19], a[156:159]\n"
    "s_mov_b64 exec, -1\n"
    "v_mfma_f32_16x16x32_bf16 a[120:123], v[88:91], v[12:15], a[120:123]\n"
    "v_mfma_f32_16x16x32_bf16 a[124:127], v[92:95], v[12:15], a[124:127]\n"
    "v_mfma_f32_16x16x32_bf16 a[88:91], v[88:91], v[8:11], a[88:91]\n"
    "v_mfma_f32_16x16x32_bf16 a[92:95], v[92:95], v[8:11], a[92:95]\n"
    "buffer_load_dwordx4 v138, s[56:59], s52 offen offset:3072 sc0 sc1 lds\n"
    "v_mfma_f32_16x16x32_bf16 a[56:59], v[88:91], v[4:7], a[56:59]\n"
    "s_add_u32 s52, s52, 0x4000\n"
    "v_mfma_f32_16x16x32_bf16 a[60:63], v[92:95], v[4:7], a[60:63]\n"
    "v_mfma_f32_16x16x32_bf16 a[24:27], v[88:91], v[0:3], a[24:27]\n"
    "s_waitcnt vmcnt(16) lgkmcnt(0)\n"
    "v_mfma_f32_16x16x32_bf16 a[28:31], v[92:95], v[0:3], a[28:31]\n"
    "s_barrier\n"
    "v_mfma_f32_16x16x32_bf16 a[0:3], v[96:99], v[32:35], a[0:3]\n"
    "ds_read_b64_tr_b16 v[64:65], v144 offset:0\n"
    "v_mfma_f32_16x16x32_bf16 a[4:7], v[100:103], v[32:35], a[4:7]\n"
    "v_mfma_f32_16x16x32_bf16 a[32:35], v[96:99], v[36:39], a[32:35]\n"
    "ds_read_b64_tr_b16 v[66:67], v144 offset:512\n"
    "s_add_u32 m0, s50, 49152\n"
    "v_mfma_f32_16x16x32_bf16 a[36:39], v[100:103], v[36:39], a[36:39]\n"
    "buffer_load_dwordx4 v132, s[44:47], s49 offen offset:0 nt lds\n"
    "v_mfma_f32_16x16x32_bf16 a[64:67], v[96:99], v[40:43], a[64:67]\n"
    "ds_read_b128 v[0:3], v137 offset:0\n"
    "v_mfma_f32_16x16x32_bf16 a[68:71], v[100:103], v[40:43], a[68:71]\n"
    "v_mfma_f32_16x16x32_bf16 a[96:99], v[96:99], v[44:47], a[96:99]\n"
    "ds_read_b128 v[4:7], v137 offset:1024\n"
    "v_mfma_f32_16x16x32_bf16 a[100:103], v[100:103], v[44:47], a[100:103]\n"
    "v_mfma_f32_16x16x32_bf16 a[128:131], v[96:99], v[48:51], a[128:131]\n"
    "ds_read_b128 v[8:11], v137 offset:2048\n"
    "v_mfma_f32_16x16x32_bf16 a[132:135], v[100:103], v[48:51], a[132:135]\n"
    "v_mfma_f32_16x16x32_bf16 a[160:163], v[96:99], v[52:55], a[160:163]\n"
    "ds_read_b128 v[12:15], v137 offset:3072\n"
    "v_mfma_f32_16x16x32_bf16 a[164:167], v[100:103], v[52:55], a[164:167]\n"
    "buffer_load_dwordx4 v133, s[44:47], s49 offen offset:1024 nt lds\n"
    "v_mfma_f32_16x16x32_bf16 a[192:195], v[96:99], v[56:59], a[192:195]\n"
    "ds_read_b128 v[16:19], v137 offset:4096\n"
    "v_mfma_f32_16x16x32_bf16 a[196:199], v[100:103], v[56:59], a[196:199]\n"
    "v_mfma_f32_16x16x32_bf16 a[224:227], v[96:99], v[60:63], a[224:227]\n"
    "ds_read_b128 v[20:23], v137 offset:5120\n"
    "v_mfma_f32_16x16x32_bf16 a[228:231], v[100:103], v[60:63], a[228:231]\n"
    "v_mfma_f32_16x16x32_bf16 a[232:235], v[104:107], v[60:63], a[232:235]\n"
    "ds_read_b128 v[24:27], v137 offset:6144\n"
    "v_mfma_f32_16x16x32_bf16 a[236:239], v[108:111], v[60:63], a[236:239]\n"
    "v_mfma_f32_16x16x32_bf16 a[200:203], v[104:107], v[56:59], a[200:203]\n"
    "ds_read_b128 v[28:31], v137 offset:7168\n"
    "v_mfma_f32_16x16x32_bf16 a[204:207], v[108:111], v[56:59], a[204:207]\n"
    "buffer_load_dwordx4 v134, s[44:47], s49 offen offset:2048 nt lds\n"
    "v_mfma_f32_16x16x32_bf16 a[168:171], v[104:107], v[52:55], a[168:171]\n"
    "ds_read_b64_tr_b16 v[68:69], v145 offset:0\n"
    "v_mfma_f32_16x16x32_bf16 a[172:175], v[108:111], v[52:55], a[172:175]\n"
    "v_mfma_f32_16x16x32_bf16 a[136:139], v[104:107], v[48:51], a[136:139]\n"
    "ds_read_b64_tr_b16 v[70:71], v145 offset:512\n"
    "v_mfma_f32_16x16x32_bf16 a[140:143], v[108:111], v[48:51], a[140:143]\n"
    "v_mfma_f32_16x16x32_bf16 a[104:107], v[104:107], v[44:47], a[104:107]\n"
    "ds_read_b64_tr_b16 v[72:73], v146 offset:0\n"
    "v_mfma_f32_16x16x32_bf16 a[108:111], v[108:111], v[44:47], a[108:111]\n"
    "v_mfma_f32_16x16x32_bf16 a[72:75], v[104:107], v[40:43], a[72:75]\n"
    "ds_read_b64_tr_b16 v[74:75], v146 offset:512\n"
    "v_mfma_f32_16x16x32_bf16 a[76:79], v[108:111], v[40:43], a[76:79]\n"
    "buffer_load_dwordx4 v135, s[44:47], s49 offen offset:3072 nt lds\n"
    "v_mfma_f32_16x16x32_bf16 a[40:43], v[104:107], v[36:39], a[40:43]\n"
    "ds_read_b64_tr_b16 v[76:77], v147 offset:0\n"
    "v_mfma_f32_16x16x32_bf16 a[44:47], v[108:111], v[36:39], a[44:47]\n"
    "s_add_u32 s53, s53, 0x24924925\n"
    "s_cselect_b32 s55, s79, s80\n"
    "v_mfma_f32_16x16x32_bf16 a[8:11], v[104:107], v[32:35], a[8:11]\n"
    "ds_read_b64_tr_b16 v[78:79], v147 offset:512\n"
    "v_mfma_f32_16x16x32_bf16 a[12:15], v[108:111], v[32:35], a[12:15]\n"
    "s_add_u32 s49, s49, s55\n"
    "s_and_b32 s54, s37, 1856\n"
    "v_mfma_f32_16x16x32_bf16 a[16:19], v[112:115], v[32:35], a[16:19]\n"
    "ds_read_b64_tr_b16 v[80:81], v144 offset:4096\n"
    "v_mfma_f32_16x16x32_bf16 a[20:23], v[116:119], v[32:35], a[20:23]\n"
    "s_and_b32 s49, s49, 0xfffffff\n"
    "v_mfma_f32_16x16x32_bf16 a[48:51], v[112:115], v[36:39], a[48:51]\n"
    "ds_read_b64_tr_b16 v[82:83], v144 offset:4608\n"
    "s_add_u32 m0, s50, 32768\n"
    "v_mfma_f32_16x16x32_bf16 a[52:55], v[116:119], v[36:39], a[52:55]\n"
    "buffer_load_dwordx4 v138, s[56:59], s52 offen offset:0 sc0 sc1 lds\n"
    "v_mfma_f32_16x16x32_bf16 a[80:83], v[112:115], v[40:43], a[80:83]\n"
    "ds_read_b64_tr_b16 v[84:85], v145 offset:4096\n"
    "v_mfma_f32_16x16x32_bf16 a[84:87], v[116:119], v[40:43], a[84:87]\n"
    "s_cmp_eq_u32 s81, 24\n"
    "s_cselect_b32 s54, s24, s54\n"
    "v_mfma_f32_16x16x32_bf16 a[112:115], v[112:115], v[44:47], a[112:115]\n"
    "ds_read_b64_tr_b16 v[86:87], v145 offset:4608\n"
    "v_mfma_f32_16x16x32_bf16 a[116:119], v[116:119], v[44:47], a[116:119]\n"
    "s_sub_u32 s55, s28, s32\n"
    "s_cmp_eq_u32 s36, 0\n"
    "v_mfma_f32_16x16x32_bf16 a[144:147], v[112:115], v[48:51], a[144:147]\n"
    "ds_read_b64_tr_b16 v[88:89], v146 offset:4096\n"
    "v_mfma_f32_16x16x32_bf16 a[148:151], v[116:119], v[48:51], a[148:151]\n"
    "s_cselect_b32 s55, s55, 0\n"
    "s_cmp_gt_i32 s55, 500\n"
    "v_mfma_f32_16x16x32_bf16 a[176:179], v[112:115], v[52:55], a[176:179]\n"
    "ds_read_b64_tr_b16 v[90:91], v146 offset:4608\n"
    "s_cselect_b32 s54, s24, s54\n"
    "v_mfma_f32_16x16x32_bf16 a[180:183], v[116:119], v[52:55], a[180:183]\n"
    "buffer_load_dwordx4 v138, s[56:59], s52 offen offset:1024 sc0 sc1 lds\n"
    "v_mfma_f32_16x16x32_bf16 a[208:211], v[112:115], v[56:59], a[208:211]\n"
    "ds_read_b64_tr_b16 v[92:93], v147 offset:4096\n"
    "v_mfma_f32_16x16x32_bf16 a[212:215], v[116:119], v[56:59], a[212:215]\n"
    "s_cmp_eq_u32 s67, 28\n"
    "s_cselect_b32 s25, s55, s25\n"
    "v_mfma_f32_16x16x32_bf16 a[240:243], v[112:115], v[60:63], a[240:243]\n"
    "ds_read_b64_tr_b16 v[94:95], v147 offset:4608\n"
    "v_mfma_f32_16x16x32_bf16 a[244:247], v[116:119], v[60:63], a[244:247]\n"
    "s_cselect_b32 s24, s54, s24\n"
    "s_cselect_b32 s35, s54, s35\n"
    "v_mfma_f32_16x16x32_bf16 a[248:251], v[120:123], v[60:63], a[248:251]\n"
    "v_mfma_f32_16x16x32_bf16 a[252:255], v[124:127], v[60:63], a[252:255]\n"
    "v_mfma_f32_16x16x32_bf16 a[216:219], v[120:123], v[56:59], a[216:219]\n"
    "v_mfma_f32_16x16x32_bf16 a[220:223], v[124:127], v[56:59], a[220:223]\n"
    "buffer_load_dwordx4 v138, s[56:59], s52 offen offset:2048 sc0 sc1 lds\n"
    "v_mfma_f32_16x16x32_bf16 a[184:187], v[120:123], v[52:55], a[184:187]\n"
    "v_mfma_f32_16x16x32_bf16 a[188:191], v[124:127], v[52:55], a[188:191]\n"
    "v_mfma_f32_16x16x32_bf16 a[152:155], v[120:123], v[48:51], a[152:155]\n"
    "s_memrealtime s[28:29]\n"
    "v_mfma_f32_16x16x32_bf16 a[156:159], v[124:127], v[48:51], a[156:159]\n"
    "v_mfma_f32_16x16x32_bf16 a[120:123], v[120:123], v[44:47], a[120:123]\n"
    "v_mfma_f32_16x16x32_bf16 a[124:127], v[124:127], v[44:47], a[124:127]\n"
    "v_mfma_f32_16x16x32_bf16 a[88:91], v[120:123], v[40:43], a[88:91]\n"
    "v_mfma_f32_16x16x32_bf16 a[92:95], v[124:127], v[40:43], a[92:95]\n"
    "buffer_load_dwordx4 v138, s[56:59], s52 offen offset:3072 sc0 sc1 lds\n"
    "v_mfma_f32_16x16x32_bf16 a[56:59], v[120:123], v[36:39], a[56:59]\n"
    "s_add_u32 s52, s52, 0x4000\n"
    "v_mfma_f32_16x16x32_bf16 a[60:63], v[124:127], v[36:39], a[60:63]\n"
    "v_mfma_f32_16x16x32_bf16 a[24:27], v[120:123], v[32:35], a[24:27]\n"
    "s_waitcnt vmcnt(16) lgkmcnt(0)\n"
    "v_mfma_f32_16x16x32_bf16 a[28:31], v[124:127], v[32:35], a[28:31]\n"
    "s_barrier\n"
    "v_mfma_f32_16x16x32_bf16 a[0:3], v[64:67], v[0:3], a[0:3]\n"
    "ds_read_b64_tr_b16 v[96:97], v144 offset:32768\n"
    "v_mfma_f32_16x16x32_bf16 a[4:7], v[68:71], v[0:3], a[4:7]\n"
    "v_mfma_f32_16x16x32_bf16 a[32:35], v[64:67], v[4:7], a[32:35]\n"
    "ds_read_b64_tr_b16 v[98:99], v144 offset:33280\n"
    "s_add_u32 m0, s50, 81920\n"
    "v_mfma_f32_16x16x32_bf16 a[36:39], v[68:71], v[4:7], a[36:39]\n"
    "buffer_load_dwordx4 v132, s[44:47], s49 offen offset:0 nt lds\n"
    "v_mfma_f32_16x16x32_bf16 a[64:67], v[64:67], v[8:11], a[64:67]\n"
    "ds_read_b128 v[32:35], v137 offset:32768\n"
    "v_mfma_f32_16x16x32_bf16 a[68:71], v[68:71], v[8:11], a[68:71]\n"
    "s_lshr_b32 s54, s24, 4\n"
    "s_add_u32 s33, s33, s54\n"
    "v_mfma_f32_16x16x32_bf16 a[96:99], v[64:67], v[12:15], a[96:99]\n"
    "ds_read_b128 v[36:39], v137 offset:33792\n"
    "s_add_u32 s32, s32, s54\n"
    "v_mfma_f32_16x16x32_bf16 a[100:103], v[68:71], v[12:15], a[100:103]\n"
    "s_sub_u32 s54, s33, s28\n"
    "s_max_i32 s54, s54, -250\n"
    "v_mfma_f32_16x16x32_bf16 a[128:131], v[64:67], v[16:19], a[128:131]\n"
    "ds_read_b128 v[40:43], v137 offset:34816\n"
    "v_mfma_f32_16x16x32_bf16 a[132:135], v[68:71], v[16:19], a[132:135]\n"
    "s_add_u32 s33, s28, s54\n"
    "s_cmp_gt_i32 s54, 0\n"
    "v_mfma_f32_16x16x32_bf16 a[160:163], v[64:67], v[20:23], a[160:163]\n"
    "ds_read_b128 v[44:47], v137 offset:35840\n"
    "s_cbranch_scc1 PACEB2_%=\n"
    "PACEBB2_%=:\n"
    "v_mfma_f32_16x16x32_bf16 a[164:167], v[68:71], v[20:23], a[164:167]\n"
    "buffer_load_dwordx4 v133, s[44:47], s49 offen offset:1024 nt lds\n"
    "v_mfma_f32_16x16x32_bf16 a[192:195], v[64:67], v[24:27], a[192:195]\n"
    "ds_read_b128 v[48:51], v137 offset:36864\n"
    "v_mfma_f32_16x16x32_bf16 a[196:199], v[68:71], v[24:27], a[196:199]\n"
    "s_sub_u32 s24, s24, 16\n"
    "s_max_i32 s24, s24, s35\n"
    "v_mfma_f32_16x16x32_bf16 a[224:227], v[64:67], v[28:31], a[224:227]\n"
    "ds_read_b128 v[52:55], v137 offset:37888\n"
    "v_mfma_f32_16x16x32_bf16 a[228:231], v[68:71], v[28:31], a[228:231]\n"
    "v_mfma_f32_16x16x32_bf16 a[232:235], v[72:75], v[28:31], a[232:235]\n"
    "ds_read_b128 v[56:59], v137 offset:38912\n"
    "v_mfma_f32_16x16x32_bf16 a[236:239], v[76:79], v[28:31], a[236:239]\n"
    "v_mfma_f32_16x16x32_bf16 a[200:203], v[72:75], v[24:27], a[200:203]\n"
    "ds_read_b128 v[60:63], v137 offset:39936\n"
    "v_mfma_f32_16x16x32_bf16 a[204:207], v[76:79], v[24:27], a[204:207]\n"
    "buffer_load_dwordx4 v134, s[44:47], s49 offen offset:2048 nt lds\n"
    "v_mfma_f32_16x16x32_bf16 a[168:171], v[72:75], v[20:23], a[168:171]\n"
    "ds_read_b64_tr_b16 v[100:101], v145 offset:32768\n"
    "v_mfma_f32_16x16x32_bf16 a[172:175], v[76:79], v[20:23], a[172:175]\n"
    "v_mfma_f32_16x16x32_bf16 a[136:139], v[72:75], v[16:19], a[136:139]\n"
    "ds_read_b64_tr_b16 v[102:103], v145 offset:33280\n"
    "v_mfma_f32_16x16x32_bf16 a[140:143], v[76:79], v[16:19], a[140:143]\n"
    "v_mfma_f32_16x16x32_bf16 a[104:107], v[72:75], v[12:15], a[104:107]\n"
    "ds_read_b64_tr_b16 v[104:105], v146 offset:32768\n"
    "v_mfma_f32_16x16x32_bf16 a[108:111], v[76:79], v[12:15], a[108:111]\n"
    "v_mfma_f32_16x16x32_bf16 a[72:75], v[72:75], v[8:11], a[72:75]\n"
    "ds_read_b64_tr_b16 v[106:107], v146 offset:33280\n"
    "v_mfma_f32_16x16x32_bf16 a[76:79], v[76:79], v[8:11], a[76:79]\n"
    "buffer_load_dwordx4 v135, s[44:47], s49 offen offset:3072 nt lds\n"
    "v_mfma_f32_16x16x32_bf16 a[40:43], v[72:75], v[4:7], a[40:43]\n"
    "ds_read_b64_tr_b16 v[108:109], v147 offset:32768\n"
    "v_mfma_f32_16x16x32_bf16 a[44:47], v[76:79], v[4:7], a[44:47]\n"
    "s_add_u32 s53, s53, 0x24924925\n"
    "s_cselect_b32 s55, s79, s80\n"
    "v_mfma_f32_16x16x32_bf16 a[8:11], v[72:75], v[0:3], a[8:11]\n"
    "ds_read_b64_tr_b16 v[110:111], v147 offset:33280\n"
    "v_mfma_f32_16x16x32_bf16 a[12:15], v[76:79], v[0:3], a[12:15]\n"
    "s_add_u32 s49, s49, s55\n"
    "v_mfma_f32_16x16x32_bf16 a[16:19], v[80:83], v[0:3], a[16:19]\n"
    "ds_read_b64_tr_b16 v[112:113], v144 offset:36864\n"
    "v_mfma_f32_16x16x32_bf16 a[20:23], v[84:87], v[0:3], a[20:23]\n"
    "s_and_b32 s49, s49, 0xfffffff\n"
    "v_mfma_f32_16x16x32_bf16 a[48:51], v[80:83], v[4:7], a[48:51]\n"
    "ds_read_b64_tr_b16 v[114:115], v144 offset:37376\n"
    "s_add_u32 m0, s50, 65536\n"
    "v_mfma_f32_16x16x32_bf16 a[52:55], v[84:87], v[4:7], a[52:55]\n"
    "buffer_load_dwordx4 v138, s[56:59], s52 offen offset:0 sc0 sc1 lds\n"
    "v_mfma_f32_16x16x32_bf16 a[80:83], v[80:83], v[8:11], a[80:83]\n"
    "ds_read_b64_tr_b16 v[116:117], v145 offset:36864\n"
    "v_mfma_f32_16x16x32_bf16 a[84:87], v[84:87], v[8:11], a[84:87]\n"
    "v_mfma_f32_16x16x32_bf16 a[112:115], v[80:83], v[12:15], a[112:115]\n"
    "ds_read_b64_tr_b16 v[118:119], v145 offset:37376\n"
    "v_mfma_f32_16x16x32_bf16 a[116:119], v[84:87], v[12:15], a[116:119]\n"
    "v_mfma_f32_16x16x32_bf16 a[144:147], v[80:83], v[16:19], a[144:147]\n"
    "ds_read_b64_tr_b16 v[120:121], v146 offset:36864\n"
    "v_mfma_f32_16x16x32_bf16 a[148:151], v[84:87], v[16:19], a[148:151]\n"
    "v_mfma_f32_16x16x32_bf16 a[176:179], v[80:83], v[20:23], a[176:179]\n"
    "ds_read_b64_tr_b16 v[122:123], v146 offset:37376\n"
    "v_mfma_f32_16x16x32_bf16 a[180:183], v[84:87], v[20:23], a[180:183]\n"
    "buffer_load_dwordx4 v138, s[56:59], s52 offen offset:1024 sc0 sc1 lds\n"
    "v_mfma_f32_16x16x32_bf16 a[208:211], v[80:83], v[24:27], a[208:211]\n"
    "ds_read_b64_tr_b16 v[124:125], v147 offset:36864\n"
    "v_mfma_f32_16x16x32_bf16 a[212:215], v[84:87], v[24:27], a[212:215]\n"
    "v_mfma_f32_16x16x32_bf16 a[240:243], v[80:83], v[28:31], a[240:243]\n"
    "ds_read_b64_tr_b16 v[126:127], v147 offset:37376\n"
    "v_mfma_f32_16x16x32_bf16 a[244:247], v[84:87], v[28:31], a[244:247]\n"
    "v_mfma_f32_16x16x32_bf16 a[248:251], v[88:91], v[28:31], a[248:251]\n"
    "v_mfma_f32_16x16x32_bf16 a[252:255], v[92:95], v[28:31], a[252:255]\n"
    "v_mfma_f32_16x16x32_bf16 a[216:219], v[88:91], v[24:27], a[216:219]\n"
    "v_mfma_f32_16x16x32_bf16 a[220:223], v[92:95], v[24:27], a[220:223]\n"
    "buffer_load_dwordx4 v138, s[56:59], s52 offen offset:2048 sc0 sc1 lds\n"
    "v_mfma_f32_16x16x32_bf16 a[184:187], v[88:91], v[20:23], a[184:187]\n"
    "v_mfma_f32_16x16x32_bf16 a[188:191], v[92:95], v[20:23], a[188:191]\n"
    "v_mfma_f32_16x16x32_bf16 a[152:155], v[88:91], v[16:19], a[152:155]\n"
    "v_mfma_f32_16x16x32_bf16 a[156:159], v[92:95], v[16:19], a[156:159]\n"
    "v_mfma_f32_16x16x32_bf16 a[120:123], v[88:91], v[12:15], a[120:123]\n"
    "v_mfma_f32_16x16x32_bf16 a[124:127], v[92:95], v[12:15], a[124:127]\n"
    "v_mfma_f32_16x16x32_bf16 a[88:91], v[88:91], v[8:11], a[88:91]\n"
    "v_mfma_f32_16x16x32_bf16 a[92:95], v[92:95], v[8:11], a[92:95]\n"
    "buffer_load_dwordx4 v138, s[56:59], s52 offen offset:3072 sc0 sc1 lds\n"
    "v_mfma_f32_16x16x32_bf16 a[56:59], v[88:91], v[4:7], a[56:59]\n"
    "s_add_u32 s52, s52, 0x4000\n"
    "v_mfma_f32_16x16x32_bf16 a[60:63], v[92:95], v[4:7], a[60:63]\n"
    "v_mfma_f32_16x16x32_bf16 a[24:27], v[88:91], v[0:3], a[24:27]\n"
    "s_waitcnt vmcnt(16) lgkmcnt(0)\n"
    "v_mfma_f32_16x16x32_bf16 a[28:31], v[92:95], v[0:3], a[28:31]\n"
    "s_barrier\n"
    "v_mfma_f32_16x16x32_bf16 a[0:3], v[96:99], v[32:35], a[0:3]\n"
    "ds_read_b64_tr_b16 v[64:65], v140 offset:0\n"
    "v_mfma_f32_16x16x32_bf16 a[4:7], v[100:103], v[32:35], a[4:7]\n"
    "v_min_u32 v176, v176, v177\n"
    "v_mfma_f32_16x16x32_bf16 a[32:35], v[96:99], v[36:39], a[32:35]\n"
    "ds_read_b64_tr_b16 v[66:67], v140 offset:512\n"
    "s_add_u32 m0, s50, 114688\n"
    "v_mfma_f32_16x16x32_bf16 a[36:39], v[100:103], v[36:39], a[36:39]\n"
    "buffer_load_dwordx4 v132, s[44:47], s49 offen offset:0 nt lds\n"
    "v_mfma_f32_16x16x32_bf16 a[64:67], v[96:99], v[40:43], a[64:67]\n"
    "ds_read_b128 v[0:3], v136 offset:0\n"
    "v_mfma_f32_16x16x32_bf16 a[68:71], v[100:103], v[40:43], a[68:71]\n"
    "v_min3_u32 v176, v176, v178, v179\n"
    "v_mfma_f32_16x16x32_bf16 a[96:99], v[96:99], v[44:47], a[96:99]\n"
    "ds_read_b128 v[4:7], v136 offset:1024\n"
    "v_mfma_f32_16x16x32_bf16 a[100:103], v[100:103], v[44:47], a[100:103]\n"
    "v_cmp_le_u32 vcc, s68, v176\n"
    "v_mfma_f32_16x16x32_bf16 a[128:131], v[96:99], v[48:51], a[128:131]\n"
    "ds_read_b128 v[8:11], v136 offset:2048\n"
    "v_mfma_f32_16x16x32_bf16 a[132:135], v[100:103], v[48:51], a[132:135]\n"
    "v_mfma_f32_16x16x32_bf16 a[160:163], v[96:99], v[52:55], a[160:163]\n"
    "ds_read_b128 v[12:15], v136 offset:3072\n"
    "v_mfma_f32_16x16x32_bf16 a[164:167], v[100:103], v[52:55], a[164:167]\n"
    "buffer_load_dwordx4 v133, s[44:47], s49 offen offset:1024 nt lds\n"
    "v_mfma_f32_16x16x32_bf16 a[192:195], v[96:99], v[56:59], a[192:195]\n"
    "ds_read_b128 v[16:19], v136 offset:4096\n"
    "v_mfma_f32_16x16x32_bf16 a[196:199], v[100:103], v[56:59], a[196:199]\n"
    "s_and_b32 s54, vcc_lo, s70\n"
    "s_or_b32 s69, s69, s54\n"
    "v_mfma_f32_16x16x32_bf16 a[224:227], v[96:99], v[60:63], a[224:227]\n"
    "ds_read_b128 v[20:23], v136 offset:5120\n"
    "v_mfma_f32_16x16x32_bf16 a[228:231], v[100:103], v[60:63], a[228:231]\n"
    "s_andn2_b32 s70, s70, s54\n"
    "s_bitcmp1_b32 s69, s72\n"
    "v_mfma_f32_16x16x32_bf16 a[232:235], v[104:107], v[60:63], a[232:235]\n"
    "ds_read_b128 v[24:27], v136 offset:6144\n"
    "v_mfma_f32_16x16x32_bf16 a[236:239], v[108:111], v[60:63], a[236:239]\n"
    "s_cbranch_scc0 SLOWB_%=\n"
    "SLOWBACKB_%=:\n"
    "s_add_u32 s72, s72, 1\n"
    "v_mfma_f32_16x16x32_bf16 a[200:203], v[104:107], v[56:59], a[200:203]\n"
    "ds_read_b128 v[28:31], v136 offset:7168\n"
    "v_mfma_f32_16x16x32_bf16 a[204:207], v[108:111], v[56:59], a[204:207]\n"
    "buffer_load_dwordx4 v134, s[44:47], s49 offen offset:2048 nt lds\n"
    "v_mfma_f32_16x16x32_bf16 a[168:171], v[104:107], v[52:55], a[168:171]\n"
    "ds_read_b64_tr_b16 v[68:69], v141 offset:0\n"
    "v_mfma_f32_16x16x32_bf16 a[172:175], v[108:111], v[52:55], a[172:175]\n"
    "v_mfma_f32_16x16x32_bf16 a[136:139], v[104:107], v[48:51], a[136:139]\n"
    "ds_read_b64_tr_b16 v[70:71], v141 offset:512\n"
    "v_mfma_f32_16x16x32_bf16 a[140:143], v[108:111], v[48:51], a[140:143]\n"
    "v_mfma_f32_16x16x32_bf16 a[104:107], v[104:107], v[44:47], a[104:107]\n"
    "ds_read_b64_tr_b16 v[72:73], v142 offset:0\n"
    "v_mfma_f32_16x16x32_bf16 a[108:111], v[108:111], v[44:47], a[108:111]\n"
    "v_mfma_f32_16x16x32_bf16 a[72:75], v[104:107], v[40:43], a[72:75]\n"
    "ds_read_b64_tr_b16 v[74:75], v142 offset:512\n"
    "v_mfma_f32_16x16x32_bf16 a[76:79], v[108:111], v[40:43], a[76:79]\n"
    "buffer_load_dwordx4 v135, s[44:47], s49 offen offset:3072 nt lds\n"
    "v_mfma_f32_16x16x32_bf16 a[40:43], v[104:107], v[36:39], a[40:43]\n"
    "ds_read_b64_tr_b16 v[76:77], v143 offset:0\n"
    "v_mfma_f32_16x16x32_bf16 a[44:47], v[108:111], v[36:39], a[44:47]\n"
    "s_add_u32 s53, s53, 0x24924925\n"
    "s_cselect_b32 s55, s79, s80\n"
    "v_mfma_f32_16x16x32_bf16 a[8:11], v[104:107], v[32:35], a[8:11]\n"
    "ds_read_b64_tr_b16 v[78:79], v143 offset:512\n"
    "v_mfma_f32_16x16x32_bf16 a[12:15], v[108:111], v[32:35], a[12:15]\n"
    "s_add_u32 s49, s49, s55\n"
    "v_mfma_f32_16x16x32_bf16 a[16:19], v[112:115], v[32:35], a[16:19]\n"
    "ds_read_b64_tr_b16 v[80:81], v140 offset:4096\n"
    "v_mfma_f32_16x16x32_bf16 a[20:23], v[116:119], v[32:35], a[20:23]\n"
    "s_and_b32 s49, s49, 0xfffffff\n"
    "v_mfma_f32_16x16x32_bf16 a[48:51], v[112:115], v[36:39], a[48:51]\n"
    "ds_read_b64_tr_b16 v[82:83], v140 offset:4608\n"
    "s_add_u32 m0, s50, 98304\n"
    "v_mfma_f32_16x16x32_bf16 a[52:55], v[116:119], v[36:39], a[52:55]\n"
    "buffer_load_dwordx4 v138, s[56:59], s52 offen offset:0 sc0 sc1 lds\n"
    "v_mfma_f32_16x16x32_bf16 a[80:83], v[112:115], v[40:43], a[80:83]\n"
    "ds_read_b64_tr_b16 v[84:85], v141 offset:4096\n"
    "v_mfma_f32_16x16x32_bf16 a[84:87], v[116:119], v[40:43], a[84:87]\n"
    "v_mfma_f32_16x16x32_bf16 a[112:115], v[112:115], v[44:47], a[112:115]\n"
    "ds_read_b64_tr_b16 v[86:87], v141 offset:4608\n"
    "v_mfma_f32_16x16x32_bf16 a[116:119], v[116:119], v[44:47], a[116:119]\n"
    "v_mfma_f32_16x16x32_bf16 a[144:147], v[112:115], v[48:51], a[144:147]\n"
    "ds_read_b64_tr_b16 v[88:89], v142 offset:4096\n"
    "v_mfma_f32_16x16x32_bf16 a[148:151], v[116:119], v[48:51], a[148:151]\n"
    "v_mfma_f32_16x16x32_bf16 a[176:179], v[112:115], v[52:55], a[176:179]\n"
    "ds_read_b64_tr_b16 v[90:91], v142 offset:4608\n"
    "v_mfma_f32_16x16x32_bf16 a[180:183], v[116:119], v[52:55], a[180:183]\n"
    "buffer_load_dwordx4 v138, s[56:59], s52 offen offset:1024 sc0 sc1 lds\n"
    "v_mfma_f32_16x16x32_bf16 a[208:211], v[112:115], v[56:59], a[208:211]\n"
    "ds_read_b64_tr_b16 v[92:93], v143 offset:4096\n"
    "v_mfma_f32_16x16x32_bf16 a[212:215], v[116:119], v[56:59], a[212:215]\n"
    "v_mfma_f32_16x16x32_bf16 a[240:243], v[112:115], v[60:63], a[240:243]\n"
    "ds_read_b64_tr_b16 v[94:95], v143 offset:4608\n"
    "v_mfma_f32_16x16x32_bf16 a[244:247], v[116:119], v[60:63], a[244:247]\n"
    "v_mfma_f32_16x16x32_bf16 a[248:251], v[120:123], v[60:63], a[248:251]\n"
    "v_mfma_f32_16x16x32_bf16 a[252:255], v[124:127], v[60:63], a[252:255]\n"
    "v_mfma_f32_16x16x32_bf16 a[216:219], v[120:123], v[56:59], a[216:219]\n"
    "v_mfma_f32_16x16x32_bf16 a[220:223], v[124:127], v[56:59], a[220:223]\n"
    "buffer_load_dwordx4 v138, s[56:59], s52 offen offset:2048 sc0 sc1 lds\n"
    "v_mfma_f32_16x16x32_bf16 a[184:187], v[120:123], v[52:55], a[184:187]\n"
    "v_mfma_f32_16x16x32_bf16 a[188:191], v[124:127], v[52:55], a[188:191]\n"
    "v_mfma_f32_16x16x32_bf16 a[152:155], v[120:123], v[48:51], a[152:155]\n"
    "s_memrealtime s[28:29]\n"
    "v_mfma_f32_16x16x32_bf16 a[156:159], v[124:127], v[48:51], a[156:159]\n"
    "v_mfma_f32_16x16x32_bf16 a[120:123], v[120:123], v[44:47], a[120:123]\n"
    "v_mfma_f32_16x16x32_bf16 a[124:127], v[124:127], v[44:47], a[124:127]\n"
    "v_mfma_f32_16x16x32_bf16 a[88:91], v[120:123], v[40:43], a[88:91]\n"
    "v_mfma_f32_16x16x32_bf16 a[92:95], v[124:127], v[40:43], a[92:95]\n"
    "buffer_load_dwordx4 v138, s[56:59], s52 offen offset:3072 sc0 sc1 lds\n"
    "v_mfma_f32_16x16x32_bf16 a[56:59], v[120:123], v[36:39], a[56:59]\n"
    "s_add_u32 s52, s52, 0x4000\n"
    "v_mfma_f32_16x16x32_bf16 a[60:63], v[124:127], v[36:39], a[60:63]\n"
    "s_cmp_lg_u32 s51, 0\n"
    "v_mfma_f32_16x16x32_bf16 a[24:27], v[120:123], v[32:35], a[24:27]\n"
    "s_waitcnt vmcnt(16) lgkmcnt(0)\n"
    "v_mfma_f32_16x16x32_bf16 a[28:31], v[124:127], v[32:35], a[28:31]\n"
    "s_barrier\n"
    "s_cbranch_scc1 LOOPB_%=\n"
    "v_mfma_f32_16x16x32_bf16 a[0:3], v[64:67], v[0:3], a[0:3]\n"
    "ds_read_b64_tr_b16 v[96:97], v140 offset:32768\n"
    "v_mfma_f32_16x16x32_bf16 a[4:7], v[68:71], v[0:3], a[4:7]\n"
    "v_mfma_f32_16x16x32_bf16 a[32:35], v[64:67], v[4:7], a[32:35]\n"
    "ds_read_b64_tr_b16 v[98:99], v140 offset:33280\n"
    "v_mfma_f32_16x16x32_bf16 a[36:39], v[68:71], v[4:7], a[36:39]\n"
    "v_mfma_f32_16x16x32_bf16 a[64:67], v[64:67], v[8:11], a[64:67]\n"
    "ds_read_b128 v[32:35], v136 offset:32768\n"
    "v_mfma_f32_16x16x32_bf16 a[68:71], v[68:71], v[8:11], a[68:71]\n"
    "s_lshr_b32 s54, s24, 4\n"
    "s_add_u32 s33, s33, s54\n"
    "v_mfma_f32_16x16x32_bf16 a[96:99], v[64:67], v[12:15], a[96:99]\n"
    "ds_read_b128 v[36:39], v136 offset:33792\n"
    "v_mfma_f32_16x16x32_bf16 a[100:103], v[68:71], v[12:15], a[100:103]\n"
    "s_sub_u32 s54, s33, s28\n"
    "s_max_i32 s54, s54, -250\n"
    "v_mfma_f32_16x16x32_bf16 a[128:131], v[64:67], v[16:19], a[128:131]\n"
    "ds_read_b128 v[40:43], v136 offset:34816\n"
    "v_mfma_f32_16x16x32_bf16 a[132:135], v[68:71], v[16:19], a[132:135]\n"
    "s_add_u32 s33, s28, s54\n"
    "s_cmp_gt_i32 s54, 0\n"
    "v_mfma_f32_16x16x32_bf16 a[160:163], v[64:67], v[20:23], a[160:163]\n"
    "ds_read_b128 v[44:47], v136 offset:35840\n"
    "s_cbranch_scc1 PACEL0_%=\n"
    "PACEBL0_%=:\n"
    "v_mfma_f32_16x16x32_bf16 a[164:167], v[68:71], v[20:23], a[164:167]\n"
    "v_mfma_f32_16x16x32_bf16 a[192:195], v[64:67], v[24:27], a[192:195]\n"
    "ds_read_b128 v[48:51], v136 offset:36864\n"
    "v_mfma_f32_16x16x32_bf16 a[196:199], v[68:71], v[24:27], a[196:199]\n"
    "s_sub_u32 s24, s24, 16\n"
    "s_max_i32 s24, s24, s35\n"
    "v_mfma_f32_16x16x32_bf16 a[224:227], v[64:67], v[28:31], a[224:227]\n"
    "ds_read_b128 v[52:55], v136 offset:37888\n"
    "v_mfma_f32_16x16x32_bf16 a[228:231], v[68:71], v[28:31], a[228:231]\n"
    "s_sub_u32 s54, s34, 1\n"
    "v_mfma_f32_16x16x32_bf16 a[232:235], v[72:75], v[28:31], a[232:235]\n"
    "ds_read_b128 v[56:59], v136 offset:38912\n"
    "v_mfma_f32_16x16x32_bf16 a[236:239], v[76:79], v[28:31], a[236:239]\n"
    "s_max_i32 s54, s54, 0\n"
    "s_cmp_gt_i32 s25, 800\n"
    "v_mfma_f32_16x16x32_bf16 a[200:203], v[72:75], v[24:27], a[200:203]\n"
    "ds_read_b128 v[60:63], v136 offset:39936\n"
    "v_mfma_f32_16x16x32_bf16 a[204:207], v[76:79], v[24:27], a[204:207]\n"
    "s_cselect_b32 s55, s81, 24\n"
    "v_mfma_f32_16x16x32_bf16 a[168:171], v[72:75], v[20:23], a[168:171]\n"
    "ds_read_b64_tr_b16 v[100:101], v141 offset:32768\n"
    "v_mfma_f32_16x16x32_bf16 a[172:175], v[76:79], v[20:23], a[172:175]\n"
    "s_add_u32 s55, s55, s34\n"
    "s_min_u32 s55, s55, 192\n"
    "v_mfma_f32_16x16x32_bf16 a[136:139], v[72:75], v[16:19], a[136:139]\n"
    "ds_read_b64_tr_b16 v[102:103], v141 offset:33280\n"
    "v_mfma_f32_16x16x32_bf16 a[140:143], v[76:79], v[16:19], a[140:143]\n"
    "s_cmp_gt_i32 s25, 500\n"
    "s_cselect_b32 s54, s55, s54\n"
    "v_mfma_f32_16x16x32_bf16 a[104:107], v[72:75], v[12:15], a[104:107]\n"
    "ds_read_b64_tr_b16 v[104:105], v142 offset:32768\n"
    "v_mfma_f32_16x16x32_bf16 a[108:111], v[76:79], v[12:15], a[108:111]\n"
    "v_mov_b32 v184, s68\n"
    "v_mov_b32 v185, s28\n"
    "v_mfma_f32_16x16x32_bf16 a[72:75], v[72:75], v[8:11], a[72:75]\n"
    "ds_read_b64_tr_b16 v[106:107], v142 offset:33280\n"
    "v_mfma_f32_16x16x32_bf16 a[76:79], v[76:79], v[8:11], a[76:79]\n"
    "v_mov_b32 v186, s54\n"
    "v_mfma_f32_16x16x32_bf16 a[40:43], v[72:75], v[4:7], a[40:43]\n"
    "ds_read_b64_tr_b16 v[108:109], v143 offset:32768\n"
    "v_mfma_f32_16x16x32_bf16 a[44:47], v[76:79], v[4:7], a[44:47]\n"
    "v_mfma_f32_16x16x32_bf16 a[8:11], v[72:75], v[0:3], a[8:11]\n"
    "ds_read_b64_tr_b16 v[110:111], v143 offset:33280\n"
    "v_mfma_f32_16x16x32_bf16 a[12:15], v[76:79], v[0:3], a[12:15]\n"
    "v_mfma_f32_16x16x32_bf16 a[16:19], v[80:83], v[0:3], a[16:19]\n"
    "ds_read_b64_tr_b16 v[112:113], v140 offset:36864\n"
    "v_mfma_f32_16x16x32_bf16 a[20:23], v[84:87], v[0:3], a[20:23]\n"
    "v_mfma_f32_16x16x32_bf16 a[48:51], v[80:83], v[4:7], a[48:51]\n"
    "ds_read_b64_tr_b16 v[114:115], v140 offset:37376\n"
    "v_mfma_f32_16x16x32_bf16 a[52:55], v[84:87], v[4:7], a[52:55]\n"
    "v_mfma_f32_16x16x32_bf16 a[80:83], v[80:83], v[8:11], a[80:83]\n"
    "ds_read_b64_tr_b16 v[116:117], v141 offset:36864\n"
    "v_mfma_f32_16x16x32_bf16 a[84:87], v[84:87], v[8:11], a[84:87]\n"
    "v_mfma_f32_16x16x32_bf16 a[112:115], v[80:83], v[12:15], a[112:115]\n"
    "ds_read_b64_tr_b16 v[118:119], v141 offset:37376\n"
    "v_mfma_f32_16x16x32_bf16 a[116:119], v[84:87], v[12:15], a[116:119]\n"
    "v_mfma_f32_16x16x32_bf16 a[144:147], v[80:83], v[16:19], a[144:147]\n"
    "ds_read_b64_tr_b16 v[120:121], v142 offset:36864\n"
    "v_mfma_f32_16x16x32_bf16 a[148:151], v[84:87], v[16:19], a[148:151]\n"
    "v_mfma_f32_16x16x32_bf16 a[176:179], v[80:83], v[20:23], a[176:179]\n"
    "ds_read_b64_tr_b16 v[122:123], v142 offset:37376\n"
    "v_mfma_f32_16x16x32_bf16 a[180:183], v[84:87], v[20:23], a[180:183]\n"
    "v_mfma_f32_16x16x32_bf16 a[208:211], v[80:83], v[24:27], a[208:211]\n"
    "ds_read_b64_tr_b16 v[124:125], v143 offset:36864\n"
    "v_mfma_f32_16x16x32_bf16 a[212:215], v[84:87], v[24:27], a[212:215]\n"
    "v_mfma_f32_16x16x32_bf16 a[240:243], v[80:83], v[28:31], a[240:243]\n"
    "ds_read_b64_tr_b16 v[126:127], v143 offset:37376\n"
    "v_mfma_f32_16x16x32_bf16 a[244:247], v[84:87], v[28:31], a[244:247]\n"
    "v_mfma_f32_16x16x32_bf16 a[248:251], v[88:91], v[28:31], a[248:251]\n"
    "v_mfma_f32_16x16x32_bf16 a[252:255], v[92:95], v[28:31], a[252:255]\n"
    "s_and_b32 exec_lo, s37, 1\n"
    "s_mov_b32 exec_hi, 0\n"
    "v_mfma_f32_16x16x32_bf16 a[216:219], v[88:91], v[24:27], a[216:219]\n"
    "global_store_dwordx3 v150, v[184:186], s[74:75] sc0 sc1\n"
    "v_mfma_f32_16x16x32_bf16 a[220:223], v[92:95], v[24:27], a[220:223]\n"
    "s_mov_b64 exec, -1\n"
    "v_mfma_f32_16x16x32_bf16 a[184:187], v[88:91], v[20:23], a[184:187]\n"
    "v_mfma_f32_16x16x32_bf16 a[188:191], v[92:95], v[20:23], a[188:191]\n"
    "v_mfma_f32_16x16x32_bf16 a[152:155], v[88:91], v[16:19], a[152:155]\n"
    "v_mfma_f32_16x16x32_bf16 a[156:159], v[92:95], v[16:19], a[156:159]\n"
    "v_mfma_f32_16x16x32_bf16 a[120:123], v[88:91], v[12:15], a[120:123]\n"
    "v_mfma_f32_16x16x32_bf16 a[124:127], v[92:95], v[12:15], a[124:127]\n"
    "v_mfma_f32_16x16x32_bf16 a[88:91], v[88:91], v[8:11], a[88:91]\n"
    "v_mfma_f32_16x16x32_bf16 a[92:95], v[92:95], v[8:11], a[92:95]\n"
    "v_mfma_f32_16x16x32_bf16 a[56:59], v[88:91], v[4:7], a[56:59]\n"
    "v_mfma_f32_16x16x32_bf16 a[60:63], v[92:95], v[4:7], a[60:63]\n"
    "v_mfma_f32_16x16x32_bf16 a[24:27], v[88:91], v[0:3], a[24:27]\n"
    "v_mfma_f32_16x16x32_bf16 a[28:31], v[92:95], v[0:3], a[28:31]\n"
    "s_waitcnt vmcnt(8) lgkmcnt(0)\n"
    "s_barrier\n"
    "v_mfma_f32_16x16x32_bf16 a[0:3], v[96:99], v[32:35], a[0:3]\n"
    "ds_read_b64_tr_b16 v[64:65], v144 offset:0\n"
    "v_mfma_f32_16x16x32_bf16 a[4:7], v[100:103], v[32:35], a[4:7]\n"
    "global_load_dword v180, v148, s[74:75] offset:2048 sc0 sc1\n"
    "v_mfma_f32_16x16x32_bf16 a[32:35], v[96:99], v[36:39], a[32:35]\n"
    "ds_read_b64_tr_b16 v[66:67], v144 offset:512\n"
    "v_mfma_f32_16x16x32_bf16 a[36:39], v[100:103], v[36:39], a[36:39]\n"
    "v_mfma_f32_16x16x32_bf16 a[64:67], v[96:99], v[40:43], a[64:67]\n"
    "ds_read_b128 v[0:3], v137 offset:0\n"
    "v_mfma_f32_16x16x32_bf16 a[68:71], v[100:103], v[40:43], a[68:71]\n"
    "v_mfma_f32_16x16x32_bf16 a[96:99], v[96:99], v[44:47], a[96:99]\n"
    "ds_read_b128 v[4:7], v137 offset:1024\n"
    "v_mfma_f32_16x16x32_bf16 a[100:103], v[100:103], v[44:47], a[100:103]\n"
    "v_mfma_f32_16x16x32_bf16 a[128:131], v[96:99], v[48:51], a[128:131]\n"
    "ds_read_b128 v[8:11], v137 offset:2048\n"
    "v_mfma_f32_16x16x32_bf16 a[132:135], v[100:103], v[48:51], a[132:135]\n"
    "v_mfma_f32_16x16x32_bf16 a[160:163], v[96:99], v[52:55], a[160:163]\n"
    "ds_read_b128 v[12:15], v137 offset:3072\n"
    "v_mfma_f32_16x16x32_bf16 a[164:167], v[100:103], v[52:55], a[164:167]\n"
    "v_mfma_f32_16x16x32_bf16 a[192:195], v[96:99], v[56:59], a[192:195]\n"
    "ds_read_b128 v[16:19], v137 offset:4096\n"
    "v_mfma_f32_16x16x32_bf16 a[196:199], v[100:103], v[56:59], a[196:199]\n"
    "v_mfma_f32_16x16x32_bf16 a[224:227], v[96:99], v[60:63], a[224:227]\n"
    "ds_read_b128 v[20:23], v137 offset:5120\n"
    "v_mfma_f32_16x16x32_bf16 a[228:231], v[100:103], v[60:63], a[228:231]\n"
    "v_mfma_f32_16x16x32_bf16 a[232:235], v[104:107], v[60:63], a[232:235]\n"
    "ds_read_b128 v[24:27], v137 offset:6144\n"
    "v_mfma_f32_16x16x32_bf16 a[236:239], v[108:111], v[60:63], a[236:239]\n"
    "v_mfma_f32_16x16x32_bf16 a[200:203], v[104:107], v[56:59], a[200:203]\n"
    "ds_read_b128 v[28:31], v137 offset:7168\n"
    "v_mfma_f32_16x16x32_bf16 a[204:207], v[108:111], v[56:59], a[204:207]\n"
    "v_mfma_f32_16x16x32_bf16 a[168:171], v[104:107], v[52:55], a[168:171]\n"
    "ds_read_b64_tr_b16 v[68:69], v145 offset:0\n"
    "v_mfma_f32_16x16x32_bf16 a[172:175], v[108:111], v[52:55], a[172:175]\n"
    "v_mfma_f32_16x16x32_bf16 a[136:139], v[104:107], v[48:51], a[136:139]\n"
    "ds_read_b64_tr_b16 v[70:71], v145 offset:512\n"
    "v_mfma_f32_16x16x32_bf16 a[140:143], v[108:111], v[48:51], a[140:143]\n"
    "v_mfma_f32_16x16x32_bf16 a[104:107], v[104:107], v[44:47], a[104:107]\n"
    "ds_read_b64_tr_b16 v[72:73], v146 offset:0\n"
    "v_mfma_f32_16x16x32_bf16 a[108:111], v[108:111], v[44:47], a[108:111]\n"
    "v_mfma_f32_16x16x32_bf16 a[72:75], v[104:107], v[40:43], a[72:75]\n"
    "ds_read_b64_tr_b16 v[74:75], v146 offset:512\n"
    "v_mfma_f32_16x16x32_bf16 a[76:79], v[108:111], v[40:43], a[76:79]\n"
    "v_mfma_f32_16x16x32_bf16 a[40:43], v[104:107], v[36:39], a[40:43]\n"
    "ds_read_b64_tr_b16 v[76:77], v147 offset:0\n"
    "v_mfma_f32_16x16x32_bf16 a[44:47], v[108:111], v[36:39], a[44:47]\n"
    "v_mfma_f32_16x16x32_bf16 a[8:11], v[104:107], v[32:35], a[8:11]\n"
    "ds_read_b64_tr_b16 v[78:79], v147 offset:512\n"
    "v_mfma_f32_16x16x32_bf16 a[12:15], v[108:111], v[32:35], a[12:15]\n"
    "v_mfma_f32_16x16x32_bf16 a[16:19], v[112:115], v[32:35], a[16:19]\n"
    "ds_read_b64_tr_b16 v[80:81], v144 offset:4096\n"
    "v_mfma_f32_16x16x32_bf16 a[20:23], v[116:119], v[32:35], a[20:23]\n"
    "v_mfma_f32_16x16x32_bf16 a[48:51], v[112:115], v[36:39], a[48:51]\n"
    "ds_read_b64_tr_b16 v[82:83], v144 offset:4608\n"
    "v_mfma_f32_16x16x32_bf16 a[52:55], v[116:119], v[36:39], a[52:55]\n"
    "v_mfma_f32_16x16x32_bf16 a[80:83], v[112:115], v[40:43], a[80:83]\n"
    "ds_read_b64_tr_b16 v[84:85], v145 offset:4096\n"
    "v_mfma_f32_16x16x32_bf16 a[84:87], v[116:119], v[40:43], a[84:87]\n"
    "v_mfma_f32_16x16x32_bf16 a[112:115], v[112:115], v[44:47], a[112:115]\n"
    "ds_read_b64_tr_b16 v[86:87], v145 offset:4608\n"
    "v_mfma_f32_16x16x32_bf16 a[116:119], v[116:119], v[44:47], a[116:119]\n"
    "v_mfma_f32_16x16x32_bf16 a[144:147], v[112:115], v[48:51], a[144:147]\n"
    "ds_read_b64_tr_b16 v[88:89], v146 offset:4096\n"
    "v_mfma_f32_16x16x32_bf16 a[148:151], v[116:119], v[48:51], a[148:151]\n"
    "v_mfma_f32_16x16x32_bf16 a[176:179], v[112:115], v[52:55], a[176:179]\n"
    "ds_read_b64_tr_b16 v[90:91], v146 offset:4608\n"
    "v_mfma_f32_16x16x32_bf16 a[180:183], v[116:119], v[52:55], a[180:183]\n"
    "v_mfma_f32_16x16x32_bf16 a[208:211], v[112:115], v[56:59], a[208:211]\n"
    "ds_read_b64_tr_b16 v[92:93], v147 offset:4096\n"
    "v_mfma_f32_16x16x32_bf16 a[212:215], v[116:119], v[56:59], a[212:215]\n"
    "v_mfma_f32_16x16x32_bf16 a[240:243], v[112:115], v[60:63], a[240:243]\n"
    "ds_read_b64_tr_b16 v[94:95], v147 offset:4608\n"
    "v_mfma_f32_16x16x32_bf16 a[244:247], v[116:119], v[60:63], a[244:247]\n"
    "v_mfma_f32_16x16x32_bf16 a[248:251], v[120:123], v[60:63], a[248:251]\n"
    "v_mfma_f32_16x16x32_bf16 a[252:255], v[124:127], v[60:63], a[252:255]\n"
    "v_mfma_f32_16x16x32_bf16 a[216:219], v[120:123], v[56:59], a[216:219]\n"
    "v_mfma_f32_16x16x32_bf16 a[220:223], v[124:127], v[56:59], a[220:223]\n"
    "v_mfma_f32_16x16x32_bf16 a[184:187], v[120:123], v[52:55], a[184:187]\n"
    "v_mfma_f32_16x16x32_bf16 a[188:191], v[124:127], v[52:55], a[188:191]\n"
    "v_mfma_f32_16x16x32_bf16 a[152:155], v[120:123], v[48:51], a[152:155]\n"
    "s_memrealtime s[28:29]\n"
    "v_mfma_f32_16x16x32_bf16 a[156:159], v[124:127], v[48:51], a[156:159]\n"
    "v_mfma_f32_16x16x32_bf16 a[120:123], v[120:123], v[44:47], a[120:123]\n"
    "v_mfma_f32_16x16x32_bf16 a[124:127], v[124:127], v[44:47], a[124:127]\n"
    "v_mfma_f32_16x16x32_bf16 a[88:91], v[120:123], v[40:43], a[88:91]\n"
    "v_mfma_f32_16x16x32_bf16 a[92:95], v[124:127], v[40:43], a[92:95]\n"
    "v_mfma_f32_16x16x32_bf16 a[56:59], v[120:123], v[36:39], a[56:59]\n"
    "v_mfma_f32_16x16x32_bf16 a[60:63], v[124:127], v[36:39], a[60:63]\n"
    "v_mfma_f32_16x16x32_bf16 a[24:27], v[120:123], v[32:35], a[24:27]\n"
    "v_mfma_f32_16x16x32_bf16 a[28:31], v[124:127], v[32:35], a[28:31]\n"
    "s_waitcnt vmcnt(0) lgkmcnt(0)\n"
    "s_barrier\n"
    "v_mfma_f32_16x16x32_bf16 a[0:3], v[64:67], v[0:3], a[0:3]\n"
    "ds_read_b64_tr_b16 v[96:97], v144 offset:32768\n"
    "v_mfma_f32_16x16x32_bf16 a[32:35], v[64:67], v[4:7], a[32:35]\n"
    "ds_read_b64_tr_b16 v[98:99], v144 offset:33280\n"
    "v_readfirstlane_b32 s55, v180\n"
    "v_mfma_f32_16x16x32_bf16 a[64:67], v[64:67], v[8:11], a[64:67]\n"
    "ds_read_b128 v[32:35], v137 offset:32768\n"
    "s_lshr_b32 s54, s24, 4\n"
    "v_mfma_f32_16x16x32_bf16 a[96:99], v[64:67], v[12:15], a[96:99]\n"
    "ds_read_b128 v[36:39], v137 offset:33792\n"
    "v_mfma_f32_16x16x32_bf16 a[4:7], v[68:71], v[0:3], a[4:7]\n"
    "ds_read_b128 v[40:43], v137 offset:34816\n"
    "v_mfma_f32_16x16x32_bf16 a[36:39], v[68:71], v[4:7], a[36:39]\n"
    "ds_read_b128 v[44:47], v137 offset:35840\n"
    "s_add_u32 s33, s33, s54\n"
    "v_mfma_f32_16x16x32_bf16 a[68:71], v[68:71], v[8:11], a[68:71]\n"
    "ds_read_b128 v[48:51], v137 offset:36864\n"
    "s_sub_u32 s54, s33, s28\n"
    "v_mfma_f32_16x16x32_bf16 a[100:103], v[68:71], v[12:15], a[100:103]\n"
    "ds_read_b128 v[52:55], v137 offset:37888\n"
    "v_mfma_f32_16x16x32_bf16 a[8:11], v[72:75], v[0:3], a[8:11]\n"
    "ds_read_b128 v[56:59], v137 offset:38912\n"
    "v_mfma_f32_16x16x32_bf16 a[40:43], v[72:75], v[4:7], a[40:43]\n"
    "ds_read_b128 v[60:63], v137 offset:39936\n"
    "v_mfma_f32_16x16x32_bf16 a[72:75], v[72:75], v[8:11], a[72:75]\n"
    "ds_read_b64_tr_b16 v[100:101], v145 offset:32768\n"
    "s_cmp_gt_i32 s54, 0\n"
    "s_cbranch_scc1 PACEL2_%=\n"
    "PACEBL2_%=:\n"
    "v_mfma_f32_16x16x32_bf16 a[104:107], v[72:75], v[12:15], a[104:107]\n"
    "ds_read_b64_tr_b16 v[102:103], v145 offset:33280\n"
    "v_mfma_f32_16x16x32_bf16 a[12:15], v[76:79], v[0:3], a[12:15]\n"
    "ds_read_b64_tr_b16 v[104:105], v146 offset:32768\n"
    "s_cmp_eq_u32 s55, s68\n"
    "s_cbranch_scc1 POISON_%=\n"
    "v_mfma_f32_16x16x32_bf16 a[44:47], v[76:79], v[4:7], a[44:47]\n"
    "ds_read_b64_tr_b16 v[106:107], v146 offset:33280\n"
    "v_mfma_f32_16x16x32_bf16 a[76:79], v[76:79], v[8:11], a[76:79]\n"
    "ds_read_b64_tr_b16 v[108:109], v147 offset:32768\n"
    "v_mfma_f32_16x16x32_bf16 a[108:111], v[76:79], v[12:15], a[108:111]\n"
    "ds_read_b64_tr_b16 v[110:111], v147 offset:33280\n"
    "v_mfma_f32_16x16x32_bf16 a[16:19], v[80:83], v[0:3], a[16:19]\n"
    "ds_read_b64_tr_b16 v[112:113], v144 offset:36864\n"
    "v_mfma_f32_16x16x32_bf16 a[48:51], v[80:83], v[4:7], a[48:51]\n"
    "ds_read_b64_tr_b16 v[114:115], v144 offset:37376\n"
    "v_mfma_f32_16x16x32_bf16 a[80:83], v[80:83], v[8:11], a[80:83]\n"
    "ds_read_b64_tr_b16 v[116:117], v145 offset:36864\n"
    "v_mfma_f32_16x16x32_bf16 a[112:115], v[80:83], v[12:15], a[112:115]\n"
    "ds_read_b64_tr_b16 v[118:119], v145 offset:37376\n"
    "v_mfma_f32_16x16x32_bf16 a[20:23], v[84:87], v[0:3], a[20:23]\n"
    "ds_read_b64_tr_b16 v[120:121], v146 offset:36864\n"
    "v_mfma_f32_16x16x32_bf16 a[52:55], v[84:87], v[4:7], a[52:55]\n"
    "ds_read_b64_tr_b16 v[122:123], v146 offset:37376\n"
    "v_mfma_f32_16x16x32_bf16 a[84:87], v[84:87], v[8:11], a[84:87]\n"
    "ds_read_b64_tr_b16 v[124:125], v147 offset:36864\n"
    "v_mfma_f32_16x16x32_bf16 a[116:119], v[84:87], v[12:15], a[116:119]\n"
    "ds_read_b64_tr_b16 v[126:127], v147 offset:37376\n"
    "v_mfma_f32_16x16x32_bf16 a[24:27], v[88:91], v[0:3], a[24:27]\n"
    "v_mfma_f32_16x16x32_bf16 a[56:59], v[88:91], v[4:7], a[56:59]\n"
    "v_mfma_f32_16x16x32_bf16 a[88:91], v[88:91], v[8:11], a[88:91]\n"
    "v_mfma_f32_16x16x32_bf16 a[120:123], v[88:91], v[12:15], a[120:123]\n"
    "v_mfma_f32_16x16x32_bf16 a[28:31], v[92:95], v[0:3], a[28:31]\n"
    "v_mfma_f32_16x16x32_bf16 a[60:63], v[92:95], v[4:7], a[60:63]\n"
    "v_mfma_f32_16x16x32_bf16 a[92:95], v[92:95], v[8:11], a[92:95]\n"
    "v_mfma_f32_16x16x32_bf16 a[124:127], v[92:95], v[12:15], a[124:127]\n"
    "s_waitcnt lgkmcnt(0)\n"
    "v_mfma_f32_16x16x32_bf16 a[0:3], v[96:99], v[32:35], a[0:3]\n"
    "v_mfma_f32_16x16x32_bf16 a[4:7], v[100:103], v[32:35], a[4:7]\n"
    "v_mfma_f32_16x16x32_bf16 a[8:11], v[104:107], v[32:35], a[8:11]\n"
    "v_mfma_f32_16x16x32_bf16 a[12:15], v[108:111], v[32:35], a[12:15]\n"
    "v_mfma_f32_16x16x32_bf16 a[16:19], v[112:115], v[32:35], a[16:19]\n"
    "v_mfma_f32_16x16x32_bf16 a[20:23], v[116:119], v[32:35], a[20:23]\n"
    "v_mfma_f32_16x16x32_bf16 a[24:27], v[120:123], v[32:35], a[24:27]\n"
    "v_mfma_f32_16x16x32_bf16 a[28:31], v[124:127], v[32:35], a[28:31]\n"
    "v_mfma_f32_16x16x32_bf16 a[32:35], v[96:99], v[36:39], a[32:35]\n"
    "v_mfma_f32_16x16x32_bf16 a[36:39], v[100:103], v[36:39], a[36:39]\n"
    "v_accvgpr_read_b32 v200, a0\n"
    "v_accvgpr_read_b32 v201, a1\n"
    "v_mfma_f32_16x16x32_bf16 a[40:43], v[104:107], v[36:39], a[40:43]\n"
    "v_accvgpr_read_b32 v202, a2\n"
    "v_accvgpr_read_b32 v203, a3\n"
    "v_mfma_f32_16x16x32_bf16 a[44:47], v[108:111], v[36:39], a[44:47]\n"
    "v_accvgpr_read_b32 v204, a4\n"
    "v_accvgpr_read_b32 v205, a5\n"
    "v_mfma_f32_16x16x32_bf16 a[48:51], v[112:115], v[36:39], a[48:51]\n"
    "v_accvgpr_read_b32 v206, a6\n"
    "v_accvgpr_read_b32 v207, a7\n"
    "v_mfma_f32_16x16x32_bf16 a[52:55], v[116:119], v[36:39], a[52:55]\n"
    "v_accvgpr_read_b32 v208, a8\n"
    "v_accvgpr_read_b32 v209, a9\n"
    "v_mfma_f32_16x16x32_bf16 a[56:59], v[120:123], v[36:39], a[56:59]\n"
    "v_accvgpr_read_b32 v210, a10\n"
    "v_accvgpr_read_b32 v211, a11\n"
    "v_mfma_f32_16x16x32_bf16 a[60:63], v[124:127], v[36:39], a[60:63]\n"
    "v_accvgpr_read_b32 v212, a12\n"
    "v_accvgpr_read_b32 v213, a13\n"
    "v_mfma_f32_16x16x32_bf16 a[64:67], v[96:99], v[40:43], a[64:67]\n"
    "v_accvgpr_read_b32 v214, a14\n"
    "v_accvgpr_read_b32 v215, a15\n"
    "v_mfma_f32_16x16x32_bf16 a[68:71], v[100:103], v[40:43], a[68:71]\n"
    "v_cvt_pk_bf16_f32 v232, v200, v201\n"
    "v_cvt_pk_bf16_f32 v233, v202, v203\n"
    "v_mfma_f32_16x16x32_bf16 a[72:75], v[104:107], v[40:43], a[72:75]\n"
    "v_cvt_pk_bf16_f32 v234, v204, v205\n"
    "v_cvt_pk_bf16_f32 v235, v206, v207\n"
    "v_mfma_f32_16x16x32_bf16 a[76:79], v[108:111], v[40:43], a[76:79]\n"
    "v_cvt_pk_bf16_f32 v236, v208, v209\n"
    "v_cvt_pk_bf16_f32 v237, v210, v211\n"
    "v_mfma_f32_16x16x32_bf16 a[80:83], v[112:115], v[40:43], a[80:83]\n"
    "v_cvt_pk_bf16_f32 v238, v212, v213\n"
    "v_cvt_pk_bf16_f32 v239, v214, v215\n"
    "v_mfma_f32_16x16x32_bf16 a[84:87], v[116:119], v[40:43], a[84:87]\n"
    "v_pk_max_i16 v232, v232, 0\n"
    "v_pk_max_i16 v234, v234, 0\n"
    "v_mfma_f32_16x16x32_bf16 a[88:91], v[120:123], v[40:43], a[88:91]\n"
    "v_pk_max_i16 v233, v233, 0\n"
    "v_pk_max_i16 v235, v235, 0\n"
    "v_mfma_f32_16x16x32_bf16 a[92:95], v[124:127], v[40:43], a[92:95]\n"
    "v_pk_max_i16 v236, v236, 0\n"
    "v_pk_max_i16 v238, v238, 0\n"
    "v_mfma_f32_16x16x32_bf16 a[96:99], v[96:99], v[44:47], a[96:99]\n"
    "v_pk_max_i16 v237, v237, 0\n"
    "v_pk_max_i16 v239, v239, 0\n"
    "v_mfma_f32_16x16x32_bf16 a[100:103], v[100:103], v[44:47], a[100:103]\n"
    "v_permlane16_swap_b32 v232, v234\n"
    "v_permlane16_swap_b32 v233, v235\n"
    "v_mfma_f32_16x16x32_bf16 a[104:107], v[104:107], v[44:47], a[104:107]\n"
    "v_permlane16_swap_b32 v236, v238\n"
    "v_permlane16_swap_b32 v237, v239\n"
    "v_mfma_f32_16x16x32_bf16 a[108:111], v[108:111], v[44:47], a[108:111]\n"
    "v_mov_b32 v164, v232\n"
    "v_mov_b32 v165, v233\n"
    "v_mfma_f32_16x16x32_bf16 a[112:115], v[112:115], v[44:47], a[112:115]\n"
    "v_mov_b32 v166, v234\n"
    "v_mov_b32 v167, v235\n"
    "v_mfma_f32_16x16x32_bf16 a[116:119], v[116:119], v[44:47], a[116:119]\n"
    "v_mov_b32_dpp v232, v236 row_ror:8 row_mask:0xf bank_mask:0xc\n"
    "v_mov_b32_dpp v233, v237 row_ror:8 row_mask:0xf bank_mask:0xc\n"
    "v_mfma_f32_16x16x32_bf16 a[120:123], v[120:123], v[44:47], a[120:123]\n"
    "v_mov_b32_dpp v234, v238 row_ror:8 row_mask:0xf bank_mask:0xc\n"
    "v_mov_b32_dpp v235, v239 row_ror:8 row_mask:0xf bank_mask:0xc\n"
    "v_mfma_f32_16x16x32_bf16 a[124:127], v[124:127], v[44:47], a[124:127]\n"
    "v_mov_b32_dpp v236, v164 row_ror:8 row_mask:0xf bank_mask:0x3\n"
    "v_mov_b32_dpp v237, v165 row_ror:8 row_mask:0xf bank_mask:0x3\n"
    "v_mfma_f32_16x16x32_bf16 a[128:131], v[64:67], v[16:19], a[128:131]\n"
    "v_mov_b32_dpp v238, v166 row_ror:8 row_mask:0xf bank_mask:0x3\n"
    "v_mov_b32_dpp v239, v167 row_ror:8 row_mask:0xf bank_mask:0x3\n"
    "v_mfma_f32_16x16x32_bf16 a[132:135], v[68:71], v[16:19], a[132:135]\n"
    "global_store_dwordx4 v198, v[232:235], s[96:97] offset:0 nt\n"
    "global_store_dwordx4 v199, v[236:239], s[96:97] offset:0 nt\n"
    "v_mfma_f32_16x16x32_bf16 a[136:139], v[72:75], v[16:19], a[136:139]\n"
    "v_accvgpr_read_b32 v216, a16\n"
    "v_accvgpr_read_b32 v217, a17\n"
    "v_mfma_f32_16x16x32_bf16 a[140:143], v[76:79], v[16:19], a[140:143]\n"
    "v_accvgpr_read_b32 v218, a18\n"
    "v_accvgpr_read_b32 v219, a19\n"
    "v_mfma_f32_16x16x32_bf16 a[144:147], v[80:83], v[16:19], a[144:147]\n"
    "v_accvgpr_read_b32 v220, a20\n"
    "v_accvgpr_read_b32 v221, a21\n"
    "v_mfma_f32_16x16x32_bf16 a[148:151], v[84:87], v[16:19], a[148:151]\n"
    "v_accvgpr_read_b32 v222, a22\n"
    "v_accvgpr_read_b32 v223, a23\n"
    "v_mfma_f32_16x16x32_bf16 a[152:155], v[88:91], v[16:19], a[152:155]\n"
    "v_accvgpr_read_b32 v224, a24\n"
    "v_accvgpr_read_b32 v225, a25\n"
    "v_mfma_f32_16x16x32_bf16 a[156:159], v[92:95], v[16:19], a[156:159]\n"
    "v_accvgpr_read_b32 v226, a26\n"
    "v_accvgpr_read_b32 v227, a27\n"
    "v_mfma_f32_16x16x32_bf16 a[128:131], v[96:99], v[48:51], a[128:131]\n"
    "v_accvgpr_read_b32 v228, a28\n"
    "v_accvgpr_read_b32 v229, a29\n"
    "v_mfma_f32_16x16x32_bf16 a[132:135], v[100:103], v[48:51], a[132:135]\n"
    "v_accvgpr_read_b32 v230, a30\n"
    "v_accvgpr_read_b32 v231, a31\n"
    "v_mfma_f32_16x16x32_bf16 a[136:139], v[104:107], v[48:51], a[136:139]\n"
    "v_cvt_pk_bf16_f32 v240, v216, v217\n"
    "v_cvt_pk_bf16_f32 v241, v218, v219\n"
    "v_mfma_f32_16x16x32_bf16 a[140:143], v[108:111], v[48:51], a[140:143]\n"
    "v_cvt_pk_bf16_f32 v242, v220, v221\n"
    "v_cvt_pk_bf16_f32 v243, v222, v223\n"
    "v_mfma_f32_16x16x32_bf16 a[144:147], v[112:115], v[48:51], a[144:147]\n"
    "v_cvt_pk_bf16_f32 v244, v224, v225\n"
    "v_cvt_pk_bf16_f32 v245, v226, v227\n"
    "v_mfma_f32_16x16x32_bf16 a[148:151], v[116:119], v[48:51], a[148:151]\n"
    "v_cvt_pk_bf16_f32 v246, v228, v229\n"
    "v_cvt_pk_bf16_f32 v247, v230, v231\n"
    "v_mfma_f32_16x16x32_bf16 a[152:155], v[120:123], v[48:51], a[152:155]\n"
    "v_pk_max_i16 v240, v240, 0\n"
    "v_pk_max_i16 v242, v242, 0\n"
    "v_mfma_f32_16x16x32_bf16 a[156:159], v[124:127], v[48:51], a[156:159]\n"
    "v_pk_max_i16 v241, v241, 0\n"
    "v_pk_max_i16 v243, v243, 0\n"
    "v_mfma_f32_16x16x32_bf16 a[160:163], v[64:67], v[20:23], a[160:163]\n"
    "v_pk_max_i16 v244, v244, 0\n"
    "v_pk_max_i16 v246, v246, 0\n"
    "v_mfma_f32_16x16x32_bf16 a[164:167], v[68:71], v[20:23], a[164:167]\n"
    "v_pk_max_i16 v245, v245, 0\n"
    "v_pk_max_i16 v247, v247, 0\n"
    "v_mfma_f32_16x16x32_bf16 a[168:171], v[72:75], v[20:23], a[168:171]\n"
    "v_permlane16_swap_b32 v240, v242\n"
    "v_permlane16_swap_b32 v241, v243\n"
    "v_mfma_f32_16x16x32_bf16 a[172:175], v[76:79], v[20:23], a[172:175]\n"
    "v_permlane16_swap_b32 v244, v246\n"
    "v_permlane16_swap_b32 v245, v247\n"
    "v_mfma_f32_16x16x32_bf16 a[176:179], v[80:83], v[20:23], a[176:179]\n"
    "v_mov_b32 v160, v240\n"
    "v_mov_b32 v161, v241\n"
    "v_mfma_f32_16x16x32_bf16 a[180:183], v[84:87], v[20:23], a[180:183]\n"
    "v_mov_b32 v162, v242\n"
    "v_mov_b32 v163, v243\n"
    "v_mfma_f32_16x16x32_bf16 a[184:187], v[88:91], v[20:23], a[184:187]\n"
    "v_mov_b32_dpp v240, v244 row_ror:8 row_mask:0xf bank_mask:0xc\n"
    "v_mov_b32_dpp v241, v245 row_ror:8 row_mask:0xf bank_mask:0xc\n"
    "v_mfma_f32_16x16x32_bf16 a[188:191], v[92:95], v[20:23], a[188:191]\n"
    "v_mov_b32_dpp v242, v246 row_ror:8 row_mask:0xf bank_mask:0xc\n"
    "v_mov_b32_dpp v243, v247 row_ror:8 row_mask:0xf bank_mask:0xc\n"
    "v_mfma_f32_16x16x32_bf16 a[160:163], v[96:99], v[52:55], a[160:163]\n"
    "v_mov_b32_dpp v244, v160 row_ror:8 row_mask:0xf bank_mask:0x3\n"
    "v_mov_b32_dpp v245, v161 row_ror:8 row_mask:0xf bank_mask:0x3\n"
    "v_mfma_f32_16x16x32_bf16 a[164:167], v[100:103], v[52:55], a[164:167]\n"
    "v_mov_b32_dpp v246, v162 row_ror:8 row_mask:0xf bank_mask:0x3\n"
    "v_mov_b32_dpp v247, v163 row_ror:8 row_mask:0xf bank_mask:0x3\n"
    "v_mfma_f32_16x16x32_bf16 a[168:171], v[104:107], v[52:55], a[168:171]\n"
    "global_store_dwordx4 v198, v[240:243], s[96:97] offset:128 nt\n"
    "global_store_dwordx4 v199, v[244:247], s[96:97] offset:128 nt\n"
    "v_mfma_f32_16x16x32_bf16 a[172:175], v[108:111], v[52:55], a[172:175]\n"
    "s_add_u32 s96, s96, 0x100000\n"
    "s_addc_u32 s97, s97, 0\n"
    "v_accvgpr_read_b32 v200, a32\n"
    "v_accvgpr_read_b32 v201, a33\n"
    "v_mfma_f32_16x16x32_bf16 a[176:179], v[112:115], v[52:55], a[176:179]\n"
    "v_accvgpr_read_b32 v202, a34\n"
    "v_accvgpr_read_b32 v203, a35\n"
    "v_mfma_f32_16x16x32_bf16 a[180:183], v[116:119], v[52:55], a[180:183]\n"
    "v_accvgpr_read_b32 v204, a36\n"
    "v_accvgpr_read_b32 v205, a37\n"
    "v_mfma_f32_16x16x32_bf16 a[184:187], v[120:123], v[52:55], a[184:187]\n"
    "v_accvgpr_read_b32 v206, a38\n"
    "v_accvgpr_read_b32 v207, a39\n"
    "v_mfma_f32_16x16x32_bf16 a[188:191], v[124:127], v[52:55], a[188:191]\n"
    "v_accvgpr_read_b32 v208, a40\n"
    "v_accvgpr_read_b32 v209, a41\n"
    "v_mfma_f32_16x16x32_bf16 a[192:195], v[64:67], v[24:27], a[192:195]\n"
    "v_accvgpr_read_b32 v210, a42\n"
    "v_accvgpr_read_b32 v211, a43\n"
    "v_mfma_f32_16x16x32_bf16 a[196:199], v[68:71], v[24:27], a[196:199]\n"
    "v_accvgpr_read_b32 v212, a44\n"
    "v_accvgpr_read_b32 v213, a45\n"
    "v_mfma_f32_16x16x32_bf16 a[200:203], v[72:75], v[24:27], a[200:203]\n"
    "v_accvgpr_read_b32 v214, a46\n"
    "v_accvgpr_read_b32 v215, a47\n"
    "v_mfma_f32_16x16x32_bf16 a[204:207], v[76:79], v[24:27], a[204:207]\n"
    "v_cvt_pk_bf16_f32 v248, v200, v201\n"
    "v_cvt_pk_bf16_f32 v249, v202, v203\n"
    "v_mfma_f32_16x16x32_bf16 a[208:211], v[80:83], v[24:27], a[208:211]\n"
    "v_cvt_pk_bf16_f32 v250, v204, v205\n"
    "v_cvt_pk_bf16_f32 v251, v206, v207\n"
    "v_mfma_f32_16x16x32_bf16 a[212:215], v[84:87], v[24:27], a[212:215]\n"
    "v_cvt_pk_bf16_f32 v252, v208, v209\n"
    "v_cvt_pk_bf16_f32 v253, v210, v211\n"
    "v_mfma_f32_16x16x32_bf16 a[216:219], v[88:91], v[24:27], a[216:219]\n"
    "v_cvt_pk_bf16_f32 v254, v212, v213\n"
    "v_cvt_pk_bf16_f32 v255, v214, v215\n"
    "v_mfma_f32_16x16x32_bf16 a[220:223], v[92:95], v[24:27], a[220:223]\n"
    "v_pk_max_i16 v248, v248, 0\n"
    "v_pk_max_i16 v250, v250, 0\n"
    "v_mfma_f32_16x16x32_bf16 a[192:195], v[96:99], v[56:59], a[192:195]\n"
    "v_pk_max_i16 v249, v249, 0\n"
    "v_pk_max_i16 v251, v251, 0\n"
    "v_mfma_f32_16x16x32_bf16 a[196:199], v[100:103], v[56:59], a[196:199]\n"
    "v_pk_max_i16 v252, v252, 0\n"
    "v_pk_max_i16 v254, v254, 0\n"
    "v_mfma_f32_16x16x32_bf16 a[200:203], v[104:107], v[56:59], a[200:203]\n"
    "v_pk_max_i16 v253, v253, 0\n"
    "v_pk_max_i16 v255, v255, 0\n"
    "v_mfma_f32_16x16x32_bf16 a[204:207], v[108:111], v[56:59], a[204:207]\n"
    "v_permlane16_swap_b32 v248, v250\n"
    "v_permlane16_swap_b32 v249, v251\n"
    "v_mfma_f32_16x16x32_bf16 a[208:211], v[112:115], v[56:59], a[208:211]\n"
    "v_permlane16_swap_b32 v252, v254\n"
    "v_permlane16_swap_b32 v253, v255\n"
    "v_mfma_f32_16x16x32_bf16 a[212:215], v[116:119], v[56:59], a[212:215]\n"
    "v_mov_b32 v164, v248\n"
    "v_mov_b32 v165, v249\n"
    "v_mfma_f32_16x16x32_bf16 a[216:219], v[120:123], v[56:59], a[216:219]\n"
    "v_mov_b32 v166, v250\n"
    "v_mov_b32 v167, v251\n"
    "v_mfma_f32_16x16x32_bf16 a[220:223], v[124:127], v[56:59], a[220:223]\n"
    "v_mov_b32_dpp v248, v252 row_ror:8 row_mask:0xf bank_mask:0xc\n"
    "v_mov_b32_dpp v249, v253 row_ror:8 row_mask:0xf bank_mask:0xc\n"
    "v_mfma_f32_16x16x32_bf16 a[224:227], v[64:67], v[28:31], a[224:227]\n"
    "v_mov_b32_dpp v250, v254 row_ror:8 row_mask:0xf bank_mask:0xc\n"
    "v_mov_b32_dpp v251, v255 row_ror:8 row_mask:0xf bank_mask:0xc\n"
    "v_mfma_f32_16x16x32_bf16 a[228:231], v[68:71], v[28:31], a[228:231]\n"
    "v_mov_b32_dpp v252, v164 row_ror:8 row_mask:0xf bank_mask:0x3\n"
    "v_mov_b32_dpp v253, v165 row_ror:8 row_mask:0xf bank_mask:0x3\n"
    "v_mfma_f32_16x16x32_bf16 a[232:235], v[72:75], v[28:31], a[232:235]\n"
    "v_mov_b32_dpp v254, v166 row_ror:8 row_mask:0xf bank_mask:0x3\n"
    "v_mov_b32_dpp v255, v167 row_ror:8 row_mask:0xf bank_mask:0x3\n"
    "v_mfma_f32_16x16x32_bf16 a[236:239], v[76:79], v[28:31], a[236:239]\n"
    "global_store_dwordx4 v198, v[248:251], s[96:97] offset:0 nt\n"
    "global_store_dwordx4 v199, v[252:255], s[96:97] offset:0 nt\n"
    "v_mfma_f32_16x16x32_bf16 a[240:243], v[80:83], v[28:31], a[240:243]\n"
    "v_accvgpr_read_b32 v216, a48\n"
    "v_accvgpr_read_b32 v217, a49\n"
    "v_mfma_f32_16x16x32_bf16 a[244:247], v[84:87], v[28:31], a[244:247]\n"
    "v_accvgpr_read_b32 v218, a50\n"
    "v_accvgpr_read_b32 v219, a51\n"
    "v_mfma_f32_16x16x32_bf16 a[248:251], v[88:91], v[28:31], a[248:251]\n"
    "v_accvgpr_read_b32 v220, a52\n"
    "v_accvgpr_read_b32 v221, a53\n"
    "v_mfma_f32_16x16x32_bf16 a[252:255], v[92:95], v[28:31], a[252:255]\n"
    "v_accvgpr_read_b32 v222, a54\n"
    "v_accvgpr_read_b32 v223, a55\n"
    "v_mfma_f32_16x16x32_bf16 a[224:227], v[96:99], v[60:63], a[224:227]\n"
    "v_accvgpr_read_b32 v224, a56\n"
    "v_accvgpr_read_b32 v225, a57\n"
    "v_mfma_f32_16x16x32_bf16 a[228:231], v[100:103], v[60:63], a[228:231]\n"
    "v_accvgpr_read_b32 v226, a58\n"
    "v_accvgpr_read_b32 v227, a59\n"
    "v_mfma_f32_16x16x32_bf16 a[232:235], v[104:107], v[60:63], a[232:235]\n"
    "v_accvgpr_read_b32 v228, a60\n"
    "v_accvgpr_read_b32 v229, a61\n"
    "v_mfma_f32_16x16x32_bf16 a[236:239], v[108:111], v[60:63], a[236:239]\n"
    "v_accvgpr_read_b32 v230, a62\n"
    "v_accvgpr_read_b32 v231, a63\n"
    "v_mfma_f32_16x16x32_bf16 a[240:243], v[112:115], v[60:63], a[240:243]\n"
    "v_cvt_pk_bf16_f32 v232, v216, v217\n"
    "v_cvt_pk_bf16_f32 v233, v218, v219\n"
    "v_mfma_f32_16x16x32_bf16 a[244:247], v[116:119], v[60:63], a[244:247]\n"
    "v_cvt_pk_bf16_f32 v234, v220, v221\n"
    "v_cvt_pk_bf16_f32 v235, v222, v223\n"
    "v_mfma_f32_16x16x32_bf16 a[248:251], v[120:123], v[60:63], a[248:251]\n"
    "v_cvt_pk_bf16_f32 v236, v224, v225\n"
    "v_cvt_pk_bf16_f32 v237, v226, v227\n"
    "v_mfma_f32_16x16x32_bf16 a[252:255], v[124:127], v[60:63], a[252:255]\n"
    "v_cvt_pk_bf16_f32 v238, v228, v229\n"
    "v_cvt_pk_bf16_f32 v239, v230, v231\n"
    "v_pk_max_i16 v232, v232, 0\n"
    "v_pk_max_i16 v234, v234, 0\n"
    "v_pk_max_i16 v233, v233, 0\n"
    "v_pk_max_i16 v235, v235, 0\n"
    "v_pk_max_i16 v236, v236, 0\n"
    "v_pk_max_i16 v238, v238, 0\n"
    "v_pk_max_i16 v237, v237, 0\n"
    "v_pk_max_i16 v239, v239, 0\n"
    "v_permlane16_swap_b32 v232, v234\n"
    "v_permlane16_swap_b32 v233, v235\n"
    "v_permlane16_swap_b32 v236, v238\n"
    "v_permlane16_swap_b32 v237, v239\n"
    "v_mov_b32 v160, v232\n"
    "v_mov_b32 v161, v233\n"
    "v_mov_b32 v162, v234\n"
    "v_mov_b32 v163, v235\n"
    "v_mov_b32_dpp v232, v236 row_ror:8 row_mask:0xf bank_mask:0xc\n"
    "v_mov_b32_dpp v233, v237 row_ror:8 row_mask:0xf bank_mask:0xc\n"
    "v_mov_b32_dpp v234, v238 row_ror:8 row_mask:0xf bank_mask:0xc\n"
    "v_mov_b32_dpp v235, v239 row_ror:8 row_mask:0xf bank_mask:0xc\n"
    "v_mov_b32_dpp v236, v160 row_ror:8 row_mask:0xf bank_mask:0x3\n"
    "v_mov_b32_dpp v237, v161 row_ror:8 row_mask:0xf bank_mask:0x3\n"
    "v_mov_b32_dpp v238, v162 row_ror:8 row_mask:0xf bank_mask:0x3\n"
    "v_mov_b32_dpp v239, v163 row_ror:8 row_mask:0xf bank_mask:0x3\n"
    "global_store_dwordx4 v198, v[232:235], s[96:97] offset:128 nt\n"
    "global_store_dwordx4 v199, v[236:239], s[96:97] offset:128 nt\n"
    "s_add_u32 s96, s96, 0x100000\n"
    "s_addc_u32 s97, s97, 0\n"
    "v_accvgpr_read_b32 v200, a64\n"
    "v_accvgpr_read_b32 v201, a65\n"
    "v_accvgpr_read_b32 v202, a66\n"
    "v_accvgpr_read_b32 v203, a67\n"
    "v_accvgpr_read_b32 v204, a68\n"
    "v_accvgpr_read_b32 v205, a69\n"
    "v_accvgpr_read_b32 v206, a70\n"
    "v_accvgpr_read_b32 v207, a71\n"
    "v_accvgpr_read_b32 v208, a72\n"
    "v_accvgpr_read_b32 v209, a73\n"
    "v_accvgpr_read_b32 v210, a74\n"
    "v_accvgpr_read_b32 v211, a75\n"
    "v_accvgpr_read_b32 v212, a76\n"
    "v_accvgpr_read_b32 v213, a77\n"
    "v_accvgpr_read_b32 v214, a78\n"
    "v_accvgpr_read_b32 v215, a79\n"
    "v_cvt_pk_bf16_f32 v240, v200, v201\n"
    "v_cvt_pk_bf16_f32 v241, v202, v203\n"
    "v_cvt_pk_bf16_f32 v242, v204, v205\n"
    "v_cvt_pk_bf16_f32 v243, v206, v207\n"
    "v_cvt_pk_bf16_f32 v244, v208, v209\n"
    "v_cvt_pk_bf16_f32 v245, v210, v211\n"
    "v_cvt_pk_bf16_f32 v246, v212, v213\n"
    "v_cvt_pk_bf16_f32 v247, v214, v215\n"
    "v_pk_max_i16 v240, v240, 0\n"
    "v_pk_max_i16 v242, v242, 0\n"
    "v_pk_max_i16 v241, v241, 0\n"
    "v_pk_max_i16 v243, v243, 0\n"
    "v_pk_max_i16 v244, v244, 0\n"
    "v_pk_max_i16 v246, v246, 0\n"
    "v_pk_max_i16 v245, v245, 0\n"
    "v_pk_max_i16 v247, v247, 0\n"
    "v_permlane16_swap_b32 v240, v242\n"
    "v_permlane16_swap_b32 v241, v243\n"
    "v_permlane16_swap_b32 v244, v246\n"
    "v_permlane16_swap_b32 v245, v247\n"
    "v_mov_b32 v164, v240\n"
    "v_mov_b32 v165, v241\n"
    "v_mov_b32 v166, v242\n"
    "v_mov_b32 v167, v243\n"
    "v_mov_b32_dpp v240, v244 row_ror:8 row_mask:0xf bank_mask:0xc\n"
    "v_mov_b32_dpp v241, v245 row_ror:8 row_mask:0xf bank_mask:0xc\n"
    "v_mov_b32_dpp v242, v246 row_ror:8 row_mask:0xf bank_mask:0xc\n"
    "v_mov_b32_dpp v243, v247 row_ror:8 row_mask:0xf bank_mask:0xc\n"
    "v_mov_b32_dpp v244, v164 row_ror:8 row_mask:0xf bank_mask:0x3\n"
    "v_mov_b32_dpp v245, v165 row_ror:8 row_mask:0xf bank_mask:0x3\n"
    "v_mov_b32_dpp v246, v166 row_ror:8 row_mask:0xf bank_mask:0x3\n"
    "v_mov_b32_dpp v247, v167 row_ror:8 row_mask:0xf bank_mask:0x3\n"
    "global_store_dwordx4 v198, v[240:243], s[96:97] offset:0 nt\n"
    "global_store_dwordx4 v199, v[244:247], s[96:97] offset:0 nt\n"
    "v_accvgpr_read_b32 v216, a80\n"
    "v_accvgpr_read_b32 v217, a81\n"
    "v_accvgpr_read_b32 v218, a82\n"
    "v_accvgpr_read_b32 v219, a83\n"
    "v_accvgpr_read_b32 v220, a84\n"
    "v_accvgpr_read_b32 v221, a85\n"
    "v_accvgpr_read_b32 v222, a86\n"
    "v_accvgpr_read_b32 v223, a87\n"
    "v_accvgpr_read_b32 v224, a88\n"
    "v_accvgpr_read_b32 v225, a89\n"
    "v_accvgpr_read_b32 v226, a90\n"
    "v_accvgpr_read_b32 v227, a91\n"
    "v_accvgpr_read_b32 v228, a92\n"
    "v_accvgpr_read_b32 v229, a93\n"
    "v_accvgpr_read_b32 v230, a94\n"
    "v_accvgpr_read_b32 v231, a95\n"
    "v_cvt_pk_bf16_f32 v248, v216, v217\n"
    "v_cvt_pk_bf16_f32 v249, v218, v219\n"
    "v_cvt_pk_bf16_f32 v250, v220, v221\n"
    "v_cvt_pk_bf16_f32 v251, v222, v223\n"
    "v_cvt_pk_bf16_f32 v252, v224, v225\n"
    "v_cvt_pk_bf16_f32 v253, v226, v227\n"
    "v_cvt_pk_bf16_f32 v254, v228, v229\n"
    "v_cvt_pk_bf16_f32 v255, v230, v231\n"
    "v_pk_max_i16 v248, v248, 0\n"
    "v_pk_max_i16 v250, v250, 0\n"
    "v_pk_max_i16 v249, v249, 0\n"
    "v_pk_max_i16 v251, v251, 0\n"
    "v_pk_max_i16 v252, v252, 0\n"
    "v_pk_max_i16 v254, v254, 0\n"
    "v_pk_max_i16 v253, v253, 0\n"
    "v_pk_max_i16 v255, v255, 0\n"
    "v_permlane16_swap_b32 v248, v250\n"
    "v_permlane16_swap_b32 v249, v251\n"
    "v_permlane16_swap_b32 v252, v254\n"
    "v_permlane16_swap_b32 v253, v255\n"
    "v_mov_b32 v160, v248\n"
    "v_mov_b32 v161, v249\n"
    "v_mov_b32 v162, v250\n"
    "v_mov_b32 v163, v251\n"
    "v_mov_b32_dpp v248, v252 row_ror:8 row_mask:0xf bank_mask:0xc\n"
    "v_mov_b32_dpp v249, v253 row_ror:8 row_mask:0xf bank_mask:0xc\n"
    "v_mov_b32_dpp v250, v254 row_ror:8 row_mask:0xf bank_mask:0xc\n"
    "v_mov_b32_dpp v251, v255 row_ror:8 row_mask:0xf bank_mask:0xc\n"
    "v_mov_b32_dpp v252, v160 row_ror:8 row_mask:0xf bank_mask:0x3\n"
    "v_mov_b32_dpp v253, v161 row_ror:8 row_mask:0xf bank_mask:0x3\n"
    "v_mov_b32_dpp v254, v162 row_ror:8 row_mask:0xf bank_mask:0x3\n"
    "v_mov_b32_dpp v255, v163 row_ror:8 row_mask:0xf bank_mask:0x3\n"
    "global_store_dwordx4 v198, v[248:251], s[96:97] offset:128 nt\n"
    "global_store_dwordx4 v199, v[252:255], s[96:97] offset:128 nt\n"
    "s_add_u32 s96, s96, 0x100000\n"
    "s_addc_u32 s97, s97, 0\n"
    "v_accvgpr_read_b32 v200, a96\n"
    "v_accvgpr_read_b32 v201, a97\n"
    "v_accvgpr_read_b32 v202, a98\n"
    "v_accvgpr_read_b32 v203, a99\n"
    "v_accvgpr_read_b32 v204, a100\n"
    "v_accvgpr_read_b32 v205, a101\n"
    "v_accvgpr_read_b32 v206, a102\n"
    "v_accvgpr_read_b32 v207, a103\n"
    "v_accvgpr_read_b32 v208, a104\n"
    "v_accvgpr_read_b32 v209, a105\n"
    "v_accvgpr_read_b32 v210, a106\n"
    "v_accvgpr_read_b32 v211, a107\n"
    "v_accvgpr_read_b32 v212, a108\n"
    "v_accvgpr_read_b32 v213, a109\n"
    "v_accvgpr_read_b32 v214, a110\n"
    "v_accvgpr_read_b32 v215, a111\n"
    "v_cvt_pk_bf16_f32 v232, v200, v201\n"
    "v_cvt_pk_bf16_f32 v233, v202, v203\n"
    "v_cvt_pk_bf16_f32 v234, v204, v205\n"
    "v_cvt_pk_bf16_f32 v235, v206, v207\n"
    "v_cvt_pk_bf16_f32 v236, v208, v209\n"
    "v_cvt_pk_bf16_f32 v237, v210, v211\n"
    "v_cvt_pk_bf16_f32 v238, v212, v213\n"
    "v_cvt_pk_bf16_f32 v239, v214, v215\n"
    "v_pk_max_i16 v232, v232, 0\n"
    "v_pk_max_i16 v234, v234, 0\n"
    "v_pk_max_i16 v233, v233, 0\n"
    "v_pk_max_i16 v235, v235, 0\n"
    "v_pk_max_i16 v236, v236, 0\n"
    "v_pk_max_i16 v238, v238, 0\n"
    "v_pk_max_i16 v237, v237, 0\n"
    "v_pk_max_i16 v239, v239, 0\n"
    "v_permlane16_swap_b32 v232, v234\n"
    "v_permlane16_swap_b32 v233, v235\n"
    "v_permlane16_swap_b32 v236, v238\n"
    "v_permlane16_swap_b32 v237, v239\n"
    "v_mov_b32 v164, v232\n"
    "v_mov_b32 v165, v233\n"
    "v_mov_b32 v166, v234\n"
    "v_mov_b32 v167, v235\n"
    "v_mov_b32_dpp v232, v236 row_ror:8 row_mask:0xf bank_mask:0xc\n"
    "v_mov_b32_dpp v233, v237 row_ror:8 row_mask:0xf bank_mask:0xc\n"
    "v_mov_b32_dpp v234, v238 row_ror:8 row_mask:0xf bank_mask:0xc\n"
    "v_mov_b32_dpp v235, v239 row_ror:8 row_mask:0xf bank_mask:0xc\n"
    "v_mov_b32_dpp v236, v164 row_ror:8 row_mask:0xf bank_mask:0x3\n"
    "v_mov_b32_dpp v237, v165 row_ror:8 row_mask:0xf bank_mask:0x3\n"
    "v_mov_b32_dpp v238, v166 row_ror:8 row_mask:0xf bank_mask:0x3\n"
    "v_mov_b32_dpp v239, v167 row_ror:8 row_mask:0xf bank_mask:0x3\n"
    "global_store_dwordx4 v198, v[232:235], s[96:97] offset:0 nt\n"
    "global_store_dwordx4 v199, v[236:239], s[96:97] offset:0 nt\n"
    "v_accvgpr_read_b32 v216, a112\n"
    "v_accvgpr_read_b32 v217, a113\n"
    "v_accvgpr_read_b32 v218, a114\n"
    "v_accvgpr_read_b32 v219, a115\n"
    "v_accvgpr_read_b32 v220, a116\n"
    "v_accvgpr_read_b32 v221, a117\n"
    "v_accvgpr_read_b32 v222, a118\n"
    "v_accvgpr_read_b32 v223, a119\n"
    "v_accvgpr_read_b32 v224, a120\n"
    "v_accvgpr_read_b32 v225, a121\n"
    "v_accvgpr_read_b32 v226, a122\n"
    "v_accvgpr_read_b32 v227, a123\n"
    "v_accvgpr_read_b32 v228, a124\n"
    "v_accvgpr_read_b32 v229, a125\n"
    "v_accvgpr_read_b32 v230, a126\n"
    "v_accvgpr_read_b32 v231, a127\n"
    "v_cvt_pk_bf16_f32 v240, v216, v217\n"
    "v_cvt_pk_bf16_f32 v241, v218, v219\n"
    "v_cvt_pk_bf16_f32 v242, v220, v221\n"
    "v_cvt_pk_bf16_f32 v243, v222, v223\n"
    "v_cvt_pk_bf16_f32 v244, v224, v225\n"
    "v_cvt_pk_bf16_f32 v245, v226, v227\n"
    "v_cvt_pk_bf16_f32 v246, v228, v229\n"
    "v_cvt_pk_bf16_f32 v247, v230, v231\n"
    "v_pk_max_i16 v240, v240, 0\n"
    "v_pk_max_i16 v242, v242, 0\n"
    "v_pk_max_i16 v241, v241, 0\n"
    "v_pk_max_i16 v243, v243, 0\n"
    "v_pk_max_i16 v244, v244, 0\n"
    "v_pk_max_i16 v246, v246, 0\n"
    "v_pk_max_i16 v245, v245, 0\n"
    "v_pk_max_i16 v247, v247, 0\n"
    "v_permlane16_swap_b32 v240, v242\n"
    "v_permlane16_swap_b32 v241, v243\n"
    "v_permlane16_swap_b32 v244, v246\n"
    "v_permlane16_swap_b32 v245, v247\n"
    "v_mov_b32 v160, v240\n"
    "v_mov_b32 v161, v241\n"
    "v_mov_b32 v162, v242\n"
    "v_mov_b32 v163, v243\n"
    "v_mov_b32_dpp v240, v244 row_ror:8 row_mask:0xf bank_mask:0xc\n"
    "v_mov_b32_dpp v241, v245 row_ror:8 row_mask:0xf bank_mask:0xc\n"
    "v_mov_b32_dpp v242, v246 row_ror:8 row_mask:0xf bank_mask:0xc\n"
    "v_mov_b32_dpp v243, v247 row_ror:8 row_mask:0xf bank_mask:0xc\n"
    "v_mov_b32_dpp v244, v160 row_ror:8 row_mask:0xf bank_mask:0x3\n"
    "v_mov_b32_dpp v245, v161 row_ror:8 row_mask:0xf bank_mask:0x3\n"
    "v_mov_b32_dpp v246, v162 row_ror:8 row_mask:0xf bank_mask:0x3\n"
    "v_mov_b32_dpp v247, v163 row_ror:8 row_mask:0xf bank_mask:0x3\n"
    "global_store_dwordx4 v198, v[240:243], s[96:97] offset:128 nt\n"
    "global_store_dwordx4 v199, v[244:247], s[96:97] offset:128 nt\n"
    "s_add_u32 s96, s96, 0x100000\n"
    "s_addc_u32 s97, s97, 0\n"
    "v_accvgpr_read_b32 v200, a128\n"
    "v_accvgpr_read_b32 v201, a129\n"
    "v_accvgpr_read_b32 v202, a130\n"
    "v_accvgpr_read_b32 v203, a131\n"
    "v_accvgpr_read_b32 v204, a132\n"
    "v_accvgpr_read_b32 v205, a133\n"
    "v_accvgpr_read_b32 v206, a134\n"
    "v_accvgpr_read_b32 v207, a135\n"
    "v_accvgpr_read_b32 v208, a136\n"
    "v_accvgpr_read_b32 v209, a137\n"
    "v_accvgpr_read_b32 v210, a138\n"
    "v_accvgpr_read_b32 v211, a139\n"
    "v_accvgpr_read_b32 v212, a140\n"
    "v_accvgpr_read_b32 v213, a141\n"
    "v_accvgpr_read_b32 v214, a142\n"
    "v_accvgpr_read_b32 v215, a143\n"
    "v_cvt_pk_bf16_f32 v248, v200, v201\n"
    "v_cvt_pk_bf16_f32 v249, v202, v203\n"
    "v_cvt_pk_bf16_f32 v250, v204, v205\n"
    "v_cvt_pk_bf16_f32 v251, v206, v207\n"
    "v_cvt_pk_bf16_f32 v252, v208, v209\n"
    "v_cvt_pk_bf16_f32 v253, v210, v211\n"
    "v_cvt_pk_bf16_f32 v254, v212, v213\n"
    "v_cvt_pk_bf16_f32 v255, v214, v215\n"
    "v_pk_max_i16 v248, v248, 0\n"
    "v_pk_max_i16 v250, v250, 0\n"
    "v_pk_max_i16 v249, v249, 0\n"
    "v_pk_max_i16 v251, v251, 0\n"
    "v_pk_max_i16 v252, v252, 0\n"
    "v_pk_max_i16 v254, v254, 0\n"
    "v_pk_max_i16 v253, v253, 0\n"
    "v_pk_max_i16 v255, v255, 0\n"
    "v_permlane16_swap_b32 v248, v250\n"
    "v_permlane16_swap_b32 v249, v251\n"
    "v_permlane16_swap_b32 v252, v254\n"
    "v_permlane16_swap_b32 v253, v255\n"
    "v_mov_b32 v164, v248\n"
    "v_mov_b32 v165, v249\n"
    "v_mov_b32 v166, v250\n"
    "v_mov_b32 v167, v251\n"
    "v_mov_b32_dpp v248, v252 row_ror:8 row_mask:0xf bank_mask:0xc\n"
    "v_mov_b32_dpp v249, v253 row_ror:8 row_mask:0xf bank_mask:0xc\n"
    "v_mov_b32_dpp v250, v254 row_ror:8 row_mask:0xf bank_mask:0xc\n"
    "v_mov_b32_dpp v251, v255 row_ror:8 row_mask:0xf bank_mask:0xc\n"
    "v_mov_b32_dpp v252, v164 row_ror:8 row_mask:0xf bank_mask:0x3\n"
    "v_mov_b32_dpp v253, v165 row_ror:8 row_mask:0xf bank_mask:0x3\n"
    "v_mov_b32_dpp v254, v166 row_ror:8 row_mask:0xf bank_mask:0x3\n"
    "v_mov_b32_dpp v255, v167 row_ror:8 row_mask:0xf bank_mask:0x3\n"
    "global_store_dwordx4 v198, v[248:251], s[96:97] offset:0\n"
    "global_store_dwordx4 v199, v[252:255], s[96:97] offset:0\n"
    "v_accvgpr_read_b32 v216, a144\n"
    "v_accvgpr_read_b32 v217, a145\n"
    "v_accvgpr_read_b32 v218, a146\n"
    "v_accvgpr_read_b32 v219, a147\n"
    "v_accvgpr_read_b32 v220, a148\n"
    "v_accvgpr_read_b32 v221, a149\n"
    "v_accvgpr_read_b32 v222, a150\n"
    "v_accvgpr_read_b32 v223, a151\n"
    "v_accvgpr_read_b32 v224, a152\n"
    "v_accvgpr_read_b32 v225, a153\n"
    "v_accvgpr_read_b32 v226, a154\n"
    "v_accvgpr_read_b32 v227, a155\n"
    "v_accvgpr_read_b32 v228, a156\n"
    "v_accvgpr_read_b32 v229, a157\n"
    "v_accvgpr_read_b32 v230, a158\n"
    "v_accvgpr_read_b32 v231, a159\n"
    "v_cvt_pk_bf16_f32 v232, v216, v217\n"
    "v_cvt_pk_bf16_f32 v233, v218, v219\n"
    "v_cvt_pk_bf16_f32 v234, v220, v221\n"
    "v_cvt_pk_bf16_f32 v235, v222, v223\n"
    "v_cvt_pk_bf16_f32 v236, v224, v225\n"
    "v_cvt_pk_bf16_f32 v237, v226, v227\n"
    "v_cvt_pk_bf16_f32 v238, v228, v229\n"
    "v_cvt_pk_bf16_f32 v239, v230, v231\n"
    "v_pk_max_i16 v232, v232, 0\n"
    "v_pk_max_i16 v234, v234, 0\n"
    "v_pk_max_i16 v233, v233, 0\n"
    "v_pk_max_i16 v235, v235, 0\n"
    "v_pk_max_i16 v236, v236, 0\n"
    "v_pk_max_i16 v238, v238, 0\n"
    "v_pk_max_i16 v237, v237, 0\n"
    "v_pk_max_i16 v239, v239, 0\n"
    "v_permlane16_swap_b32 v232, v234\n"
    "v_permlane16_swap_b32 v233, v235\n"
    "v_permlane16_swap_b32 v236, v238\n"
    "v_permlane16_swap_b32 v237, v239\n"
    "v_mov_b32 v160, v232\n"
    "v_mov_b32 v161, v233\n"
    "v_mov_b32 v162, v234\n"
    "v_mov_b32 v163, v235\n"
    "v_mov_b32_dpp v232, v236 row_ror:8 row_mask:0xf bank_mask:0xc\n"
    "v_mov_b32_dpp v233, v237 row_ror:8 row_mask:0xf bank_mask:0xc\n"
    "v_mov_b32_dpp v234, v238 row_ror:8 row_mask:0xf bank_mask:0xc\n"
    "v_mov_b32_dpp v235, v239 row_ror:8 row_mask:0xf bank_mask:0xc\n"
    "v_mov_b32_dpp v236, v160 row_ror:8 row_mask:0xf bank_mask:0x3\n"
    "v_mov_b32_dpp v237, v161 row_ror:8 row_mask:0xf bank_mask:0x3\n"
    "v_mov_b32_dpp v238, v162 row_ror:8 row_mask:0xf bank_mask:0x3\n"
    "v_mov_b32_dpp v239, v163 row_ror:8 row_mask:0xf bank_mask:0x3\n"
    "global_store_dwordx4 v198, v[232:235], s[96:97] offset:128\n"
    "global_store_dwordx4 v199, v[236:239], s[96:97] offset:128\n"
    "s_add_u32 s96, s96, 0x100000\n"
    "s_addc_u32 s97, s97, 0\n"
    "v_accvgpr_read_b32 v200, a160\n"
    "v_accvgpr_read_b32 v201, a161\n"
    "v_accvgpr_read_b32 v202, a162\n"
    "v_accvgpr_read_b32 v203, a163\n"
    "v_accvgpr_read_b32 v204, a164\n"
    "v_accvgpr_read_b32 v205, a165\n"
    "v_accvgpr_read_b32 v206, a166\n"
    "v_accvgpr_read_b32 v207, a167\n"
    "v_accvgpr_read_b32 v208, a168\n"
    "v_accvgpr_read_b32 v209, a169\n"
    "v_accvgpr_read_b32 v210, a170\n"
    "v_accvgpr_read_b32 v211, a171\n"
    "v_accvgpr_read_b32 v212, a172\n"
    "v_accvgpr_read_b32 v213, a173\n"
    "v_accvgpr_read_b32 v214, a174\n"
    "v_accvgpr_read_b32 v215, a175\n"
    "v_cvt_pk_bf16_f32 v240, v200, v201\n"
    "v_cvt_pk_bf16_f32 v241, v202, v203\n"
    "v_cvt_pk_bf16_f32 v242, v204, v205\n"
    "v_cvt_pk_bf16_f32 v243, v206, v207\n"
    "v_cvt_pk_bf16_f32 v244, v208, v209\n"
    "v_cvt_pk_bf16_f32 v245, v210, v211\n"
    "v_cvt_pk_bf16_f32 v246, v212, v213\n"
    "v_cvt_pk_bf16_f32 v247, v214, v215\n"
    "v_pk_max_i16 v240, v240, 0\n"
    "v_pk_max_i16 v242, v242, 0\n"
    "v_pk_max_i16 v241, v241, 0\n"
    "v_pk_max_i16 v243, v243, 0\n"
    "v_pk_max_i16 v244, v244, 0\n"
    "v_pk_max_i16 v246, v246, 0\n"
    "v_pk_max_i16 v245, v245, 0\n"
    "v_pk_max_i16 v247, v247, 0\n"
    "v_permlane16_swap_b32 v240, v242\n"
    "v_permlane16_swap_b32 v241, v243\n"
    "v_permlane16_swap_b32 v244, v246\n"
    "v_permlane16_swap_b32 v245, v247\n"
    "v_mov_b32 v164, v240\n"
    "v_mov_b32 v165, v241\n"
    "v_mov_b32 v166, v242\n"
    "v_mov_b32 v167, v243\n"
    "v_mov_b32_dpp v240, v244 row_ror:8 row_mask:0xf bank_mask:0xc\n"
    "v_mov_b32_dpp v241, v245 row_ror:8 row_mask:0xf bank_mask:0xc\n"
    "v_mov_b32_dpp v242, v246 row_ror:8 row_mask:0xf bank_mask:0xc\n"
    "v_mov_b32_dpp v243, v247 row_ror:8 row_mask:0xf bank_mask:0xc\n"
    "v_mov_b32_dpp v244, v164 row_ror:8 row_mask:0xf bank_mask:0x3\n"
    "v_mov_b32_dpp v245, v165 row_ror:8 row_mask:0xf bank_mask:0x3\n"
    "v_mov_b32_dpp v246, v166 row_ror:8 row_mask:0xf bank_mask:0x3\n"
    "v_mov_b32_dpp v247, v167 row_ror:8 row_mask:0xf bank_mask:0x3\n"
    "global_store_dwordx4 v198, v[240:243], s[96:97] offset:0\n"
    "global_store_dwordx4 v199, v[244:247], s[96:97] offset:0\n"
    "v_accvgpr_read_b32 v216, a176\n"
    "v_accvgpr_read_b32 v217, a177\n"
    "v_accvgpr_read_b32 v218, a178\n"
    "v_accvgpr_read_b32 v219, a179\n"
    "v_accvgpr_read_b32 v220, a180\n"
    "v_accvgpr_read_b32 v221, a181\n"
    "v_accvgpr_read_b32 v222, a182\n"
    "v_accvgpr_read_b32 v223, a183\n"
    "v_accvgpr_read_b32 v224, a184\n"
    "v_accvgpr_read_b32 v225, a185\n"
    "v_accvgpr_read_b32 v226, a186\n"
    "v_accvgpr_read_b32 v227, a187\n"
    "v_accvgpr_read_b32 v228, a188\n"
    "v_accvgpr_read_b32 v229, a189\n"
    "v_accvgpr_read_b32 v230, a190\n"
    "v_accvgpr_read_b32 v231, a191\n"
    "v_cvt_pk_bf16_f32 v248, v216, v217\n"
    "v_cvt_pk_bf16_f32 v249, v218, v219\n"
    "v_cvt_pk_bf16_f32 v250, v220, v221\n"
    "v_cvt_pk_bf16_f32 v251, v222, v223\n"
    "v_cvt_pk_bf16_f32 v252, v224, v225\n"
    "v_cvt_pk_bf16_f32 v253, v226, v227\n"
    "v_cvt_pk_bf16_f32 v254, v228, v229\n"
    "v_cvt_pk_bf16_f32 v255, v230, v231\n"
    "v_pk_max_i16 v248, v248, 0\n"
    "v_pk_max_i16 v250, v250, 0\n"
    "v_pk_max_i16 v249, v249, 0\n"
    "v_pk_max_i16 v251, v251, 0\n"
    "v_pk_max_i16 v252, v252, 0\n"
    "v_pk_max_i16 v254, v254, 0\n"
    "v_pk_max_i16 v253, v253, 0\n"
    "v_pk_max_i16 v255, v255, 0\n"
    "v_permlane16_swap_b32 v248, v250\n"
    "v_permlane16_swap_b32 v249, v251\n"
    "v_permlane16_swap_b32 v252, v254\n"
    "v_permlane16_swap_b32 v253, v255\n"
    "v_mov_b32 v160, v248\n"
    "v_mov_b32 v161, v249\n"
    "v_mov_b32 v162, v250\n"
    "v_mov_b32 v163, v251\n"
    "v_mov_b32_dpp v248, v252 row_ror:8 row_mask:0xf bank_mask:0xc\n"
    "v_mov_b32_dpp v249, v253 row_ror:8 row_mask:0xf bank_mask:0xc\n"
    "v_mov_b32_dpp v250, v254 row_ror:8 row_mask:0xf bank_mask:0xc\n"
    "v_mov_b32_dpp v251, v255 row_ror:8 row_mask:0xf bank_mask:0xc\n"
    "v_mov_b32_dpp v252, v160 row_ror:8 row_mask:0xf bank_mask:0x3\n"
    "v_mov_b32_dpp v253, v161 row_ror:8 row_mask:0xf bank_mask:0x3\n"
    "v_mov_b32_dpp v254, v162 row_ror:8 row_mask:0xf bank_mask:0x3\n"
    "v_mov_b32_dpp v255, v163 row_ror:8 row_mask:0xf bank_mask:0x3\n"
    "global_store_dwordx4 v198, v[248:251], s[96:97] offset:128\n"
    "global_store_dwordx4 v199, v[252:255], s[96:97] offset:128\n"
    "s_add_u32 s96, s96, 0x100000\n"
    "s_addc_u32 s97, s97, 0\n"
    "v_accvgpr_read_b32 v200, a192\n"
    "v_accvgpr_read_b32 v201, a193\n"
    "v_accvgpr_read_b32 v202, a194\n"
    "v_accvgpr_read_b32 v203, a195\n"
    "v_accvgpr_read_b32 v204, a196\n"
    "v_accvgpr_read_b32 v205, a197\n"
    "v_accvgpr_read_b32 v206, a198\n"
    "v_accvgpr_read_b32 v207, a199\n"
    "v_accvgpr_read_b32 v208, a200\n"
    "v_accvgpr_read_b32 v209, a201\n"
    "v_accvgpr_read_b32 v210, a202\n"
    "v_accvgpr_read_b32 v211, a203\n"
    "v_accvgpr_read_b32 v212, a204\n"
    "v_accvgpr_read_b32 v213, a205\n"
    "v_accvgpr_read_b32 v214, a206\n"
    "v_accvgpr_read_b32 v215, a207\n"
    "v_cvt_pk_bf16_f32 v232, v200, v201\n"
    "v_cvt_pk_bf16_f32 v233, v202, v203\n"
    "v_cvt_pk_bf16_f32 v234, v204, v205\n"
    "v_cvt_pk_bf16_f32 v235, v206, v207\n"
    "v_cvt_pk_bf16_f32 v236, v208, v209\n"
    "v_cvt_pk_bf16_f32 v237, v210, v211\n"
    "v_cvt_pk_bf16_f32 v238, v212, v213\n"
    "v_cvt_pk_bf16_f32 v239, v214, v215\n"
    "v_pk_max_i16 v232, v232, 0\n"
    "v_pk_max_i16 v234, v234, 0\n"
    "v_pk_max_i16 v233, v233, 0\n"
    "v_pk_max_i16 v235, v235, 0\n"
    "v_pk_max_i16 v236, v236, 0\n"
    "v_pk_max_i16 v238, v238, 0\n"
    "v_pk_max_i16 v237, v237, 0\n"
    "v_pk_max_i16 v239, v239, 0\n"
    "v_permlane16_swap_b32 v232, v234\n"
    "v_permlane16_swap_b32 v233, v235\n"
    "v_permlane16_swap_b32 v236, v238\n"
    "v_permlane16_swap_b32 v237, v239\n"
    "v_mov_b32 v164, v232\n"
    "v_mov_b32 v165, v233\n"
    "v_mov_b32 v166, v234\n"
    "v_mov_b32 v167, v235\n"
    "v_mov_b32_dpp v232, v236 row_ror:8 row_mask:0xf bank_mask:0xc\n"
    "v_mov_b32_dpp v233, v237 row_ror:8 row_mask:0xf bank_mask:0xc\n"
    "v_mov_b32_dpp v234, v238 row_ror:8 row_mask:0xf bank_mask:0xc\n"
    "v_mov_b32_dpp v235, v239 row_ror:8 row_mask:0xf bank_mask:0xc\n"
    "v_mov_b32_dpp v236, v164 row_ror:8 row_mask:0xf bank_mask:0x3\n"
    "v_mov_b32_dpp v237, v165 row_ror:8 row_mask:0xf bank_mask:0x3\n"
    "v_mov_b32_dpp v238, v166 row_ror:8 row_mask:0xf bank_mask:0x3\n"
    "v_mov_b32_dpp v239, v167 row_ror:8 row_mask:0xf bank_mask:0x3\n"
    "global_store_dwordx4 v198, v[232:235], s[96:97] offset:0\n"
    "global_store_dwordx4 v199, v[236:239], s[96:97] offset:0\n"
    "v_accvgpr_read_b32 v216, a208\n"
    "v_accvgpr_read_b32 v217, a209\n"
    "v_accvgpr_read_b32 v218, a210\n"
    "v_accvgpr_read_b32 v219, a211\n"
    "v_accvgpr_read_b32 v220, a212\n"
    "v_accvgpr_read_b32 v221, a213\n"
    "v_accvgpr_read_b32 v222, a214\n"
    "v_accvgpr_read_b32 v223, a215\n"
    "v_accvgpr_read_b32 v224, a216\n"
    "v_accvgpr_read_b32 v225, a217\n"
    "v_accvgpr_read_b32 v226, a218\n"
    "v_accvgpr_read_b32 v227, a219\n"
    "v_accvgpr_read_b32 v228, a220\n"
    "v_accvgpr_read_b32 v229, a221\n"
    "v_accvgpr_read_b32 v230, a222\n"
    "v_accvgpr_read_b32 v231, a223\n"
    "v_cvt_pk_bf16_f32 v240, v216, v217\n"
    "v_cvt_pk_bf16_f32 v241, v218, v219\n"
    "v_cvt_pk_bf16_f32 v242, v220, v221\n"
    "v_cvt_pk_bf16_f32 v243, v222, v223\n"
    "v_cvt_pk_bf16_f32 v244, v224, v225\n"
    "v_cvt_pk_bf16_f32 v245, v226, v227\n"
    "v_cvt_pk_bf16_f32 v246, v228, v229\n"
    "v_cvt_pk_bf16_f32 v247, v230, v231\n"
    "v_pk_max_i16 v240, v240, 0\n"
    "v_pk_max_i16 v242, v242, 0\n"
    "v_pk_max_i16 v241, v241, 0\n"
    "v_pk_max_i16 v243, v243, 0\n"
    "v_pk_max_i16 v244, v244, 0\n"
    "v_pk_max_i16 v246, v246, 0\n"
    "v_pk_max_i16 v245, v245, 0\n"
    "v_pk_max_i16 v247, v247, 0\n"
    "v_permlane16_swap_b32 v240, v242\n"
    "v_permlane16_swap_b32 v241, v243\n"
    "v_permlane16_swap_b32 v244, v246\n"
    "v_permlane16_swap_b32 v245, v247\n"
    "v_mov_b32 v160, v240\n"
    "v_mov_b32 v161, v241\n"
    "v_mov_b32 v162, v242\n"
    "v_mov_b32 v163, v243\n"
    "v_mov_b32_dpp v240, v244 row_ror:8 row_mask:0xf bank_mask:0xc\n"
    "v_mov_b32_dpp v241, v245 row_ror:8 row_mask:0xf bank_mask:0xc\n"
    "v_mov_b32_dpp v242, v246 row_ror:8 row_mask:0xf bank_mask:0xc\n"
    "v_mov_b32_dpp v243, v247 row_ror:8 row_mask:0xf bank_mask:0xc\n"
    "v_mov_b32_dpp v244, v160 row_ror:8 row_mask:0xf bank_mask:0x3\n"
    "v_mov_b32_dpp v245, v161 row_ror:8 row_mask:0xf bank_mask:0x3\n"
    "v_mov_b32_dpp v246, v162 row_ror:8 row_mask:0xf bank_mask:0x3\n"
    "v_mov_b32_dpp v247, v163 row_ror:8 row_mask:0xf bank_mask:0x3\n"
    "global_store_dwordx4 v198, v[240:243], s[96:97] offset:128\n"
    "global_store_dwordx4 v199, v[244:247], s[96:97] offset:128\n"
    "s_add_u32 s96, s96, 0x100000\n"
    "s_addc_u32 s97, s97, 0\n"
    "v_accvgpr_read_b32 v200, a224\n"
    "v_accvgpr_read_b32 v201, a225\n"
    "v_accvgpr_read_b32 v202, a226\n"
    "v_accvgpr_read_b32 v203, a227\n"
    "v_accvgpr_read_b32 v204, a228\n"
    "v_accvgpr_read_b32 v205, a229\n"
    "v_accvgpr_read_b32 v206, a230\n"
    "v_accvgpr_read_b32 v207, a231\n"
    "v_accvgpr_read_b32 v208, a232\n"
    "v_accvgpr_read_b32 v209, a233\n"
    "v_accvgpr_read_b32 v210, a234\n"
    "v_accvgpr_read_b32 v211, a235\n"
    "v_accvgpr_read_b32 v212, a236\n"
    "v_accvgpr_read_b32 v213, a237\n"
    "v_accvgpr_read_b32 v214, a238\n"
    "v_accvgpr_read_b32 v215, a239\n"
    "v_cvt_pk_bf16_f32 v248, v200, v201\n"
    "v_cvt_pk_bf16_f32 v249, v202, v203\n"
    "v_cvt_pk_bf16_f32 v250, v204, v205\n"
    "v_cvt_pk_bf16_f32 v251, v206, v207\n"
    "v_cvt_pk_bf16_f32 v252, v208, v209\n"
    "v_cvt_pk_bf16_f32 v253, v210, v211\n"
    "v_cvt_pk_bf16_f32 v254, v212, v213\n"
    "v_cvt_pk_bf16_f32 v255, v214, v215\n"
    "v_pk_max_i16 v248, v248, 0\n"
    "v_pk_max_i16 v250, v250, 0\n"
    "v_pk_max_i16 v249, v249, 0\n"
    "v_pk_max_i16 v251, v251, 0\n"
    "v_pk_max_i16 v252, v252, 0\n"
    "v_pk_max_i16 v254, v254, 0\n"
    "v_pk_max_i16 v253, v253, 0\n"
    "v_pk_max_i16 v255, v255, 0\n"
    "v_permlane16_swap_b32 v248, v250\n"
    "v_permlane16_swap_b32 v249, v251\n"
    "v_permlane16_swap_b32 v252, v254\n"
    "v_permlane16_swap_b32 v253, v255\n"
    "v_mov_b32 v164, v248\n"
    "v_mov_b32 v165, v249\n"
    "v_mov_b32 v166, v250\n"
    "v_mov_b32 v167, v251\n"
    "v_mov_b32_dpp v248, v252 row_ror:8 row_mask:0xf bank_mask:0xc\n"
    "v_mov_b32_dpp v249, v253 row_ror:8 row_mask:0xf bank_mask:0xc\n"
    "v_mov_b32_dpp v250, v254 row_ror:8 row_mask:0xf bank_mask:0xc\n"
    "v_mov_b32_dpp v251, v255 row_ror:8 row_mask:0xf bank_mask:0xc\n"
    "v_mov_b32_dpp v252, v164 row_ror:8 row_mask:0xf bank_mask:0x3\n"
    "v_mov_b32_dpp v253, v165 row_ror:8 row_mask:0xf bank_mask:0x3\n"
    "v_mov_b32_dpp v254, v166 row_ror:8 row_mask:0xf bank_mask:0x3\n"
    "v_mov_b32_dpp v255, v167 row_ror:8 row_mask:0xf bank_mask:0x3\n"
    "global_store_dwordx4 v198, v[248:251], s[96:97] offset:0\n"
    "global_store_dwordx4 v199, v[252:255], s[96:97] offset:0\n"
    "v_accvgpr_read_b32 v216, a240\n"
    "v_accvgpr_read_b32 v217, a241\n"
    "v_accvgpr_read_b32 v218, a242\n"
    "v_accvgpr_read_b32 v219, a243\n"
    "v_accvgpr_read_b32 v220, a244\n"
    "v_accvgpr_read_b32 v221, a245\n"
    "v_accvgpr_read_b32 v222, a246\n"
    "v_accvgpr_read_b32 v223, a247\n"
    "v_accvgpr_read_b32 v224, a248\n"
    "v_accvgpr_read_b32 v225, a249\n"
    "v_accvgpr_read_b32 v226, a250\n"
    "v_accvgpr_read_b32 v227, a251\n"
    "v_accvgpr_read_b32 v228, a252\n"
    "v_accvgpr_read_b32 v229, a253\n"
    "v_accvgpr_read_b32 v230, a254\n"
    "v_accvgpr_read_b32 v231, a255\n"
    "v_cvt_pk_bf16_f32 v232, v216, v217\n"
    "v_cvt_pk_bf16_f32 v233, v218, v219\n"
    "v_cvt_pk_bf16_f32 v234, v220, v221\n"
    "v_cvt_pk_bf16_f32 v235, v222, v223\n"
    "v_cvt_pk_bf16_f32 v236, v224, v225\n"
    "v_cvt_pk_bf16_f32 v237, v226, v227\n"
    "v_cvt_pk_bf16_f32 v238, v228, v229\n"
    "v_cvt_pk_bf16_f32 v239, v230, v231\n"
    "v_pk_max_i16 v232, v232, 0\n"
    "v_pk_max_i16 v234, v234, 0\n"
    "v_pk_max_i16 v233, v233, 0\n"
    "v_pk_max_i16 v235, v235, 0\n"
    "v_pk_max_i16 v236, v236, 0\n"
    "v_pk_max_i16 v238, v238, 0\n"
    "v_pk_max_i16 v237, v237, 0\n"
    "v_pk_max_i16 v239, v239, 0\n"
    "v_permlane16_swap_b32 v232, v234\n"
    "v_permlane16_swap_b32 v233, v235\n"
    "v_permlane16_swap_b32 v236, v238\n"
    "v_permlane16_swap_b32 v237, v239\n"
    "v_mov_b32 v160, v232\n"
    "v_mov_b32 v161, v233\n"
    "v_mov_b32 v162, v234\n"
    "v_mov_b32 v163, v235\n"
    "v_mov_b32_dpp v232, v236 row_ror:8 row_mask:0xf bank_mask:0xc\n"
    "v_mov_b32_dpp v233, v237 row_ror:8 row_mask:0xf bank_mask:0xc\n"
    "v_mov_b32_dpp v234, v238 row_ror:8 row_mask:0xf bank_mask:0xc\n"
    "v_mov_b32_dpp v235, v239 row_ror:8 row_mask:0xf bank_mask:0xc\n"
    "v_mov_b32_dpp v236, v160 row_ror:8 row_mask:0xf bank_mask:0x3\n"
    "v_mov_b32_dpp v237, v161 row_ror:8 row_mask:0xf bank_mask:0x3\n"
    "v_mov_b32_dpp v238, v162 row_ror:8 row_mask:0xf bank_mask:0x3\n"
    "v_mov_b32_dpp v239, v163 row_ror:8 row_mask:0xf bank_mask:0x3\n"
    "global_store_dwordx4 v198, v[232:235], s[96:97] offset:128\n"
    "global_store_dwordx4 v199, v[236:239], s[96:97] offset:128\n"
    "s_branch END_%=\n"
    "POISON_%=:\n"
    "v_mov_b32 v232, 0x7fc07fc0\n"
    "v_mov_b32 v233, 0x7fc07fc0\n"
    "v_mov_b32 v234, 0x7fc07fc0\n"
    "v_mov_b32 v235, 0x7fc07fc0\n"
    "s_mov_b32 s54, 8\n"
    "POISONL_%=:\n"
    "global_store_dwordx4 v198, v[232:235], s[96:97] offset:0\n"
    "global_store_dwordx4 v199, v[232:235], s[96:97] offset:0\n"
    "global_store_dwordx4 v198, v[232:235], s[96:97] offset:128\n"
    "global_store_dwordx4 v199, v[232:235], s[96:97] offset:128\n"
    "s_add_u32 s96, s96, 0x100000\n"
    "s_addc_u32 s97, s97, 0\n"
    "s_sub_u32 s54, s54, 1\n"
    "s_cmp_lg_u32 s54, 0\n"
    "s_cbranch_scc1 POISONL_%=\n"
    "s_branch END_%=\n"
    "PACEI02_%=:\n"
    "s_memrealtime s[28:29]\n"
    "s_waitcnt lgkmcnt(0)\n"
    "s_sub_u32 s54, s33, s28\n"
    "s_cmp_gt_i32 s54, 0\n"
    "s_cbranch_scc1 PACEI02_%=\n"
    "s_branch PACEBI02_%=\n"
    "PACEI10_%=:\n"
    "s_memrealtime s[28:29]\n"
    "s_waitcnt lgkmcnt(0)\n"
    "s_sub_u32 s54, s33, s28\n"
    "s_cmp_gt_i32 s54, 0\n"
    "s_cbranch_scc1 PACEI10_%=\n"
    "s_branch PACEBI10_%=\n"
    "PACEI12_%=:\n"
    "s_memrealtime s[28:29]\n"
    "s_waitcnt lgkmcnt(0)\n"
    "s_sub_u32 s54, s33, s28\n"
    "s_cmp_gt_i32 s54, 0\n"
    "s_cbranch_scc1 PACEI12_%=\n"
    "s_branch PACEBI12_%=\n"
    "PACEI20_%=:\n"
    "s_memrealtime s[28:29]\n"
    "s_waitcnt lgkmcnt(0)\n"
    "s_sub_u32 s54, s33, s28\n"
    "s_cmp_gt_i32 s54, 0\n"
    "s_cbranch_scc1 PACEI20_%=\n"
    "s_branch PACEBI20_%=\n"
    "PACEI22_%=:\n"
    "s_memrealtime s[28:29]\n"
    "s_waitcnt lgkmcnt(0)\n"
    "s_sub_u32 s54, s33, s28\n"
    "s_cmp_gt_i32 s54, 0\n"
    "s_cbranch_scc1 PACEI22_%=\n"
    "s_branch PACEBI22_%=\n"
    "PACEI30_%=:\n"
    "s_memrealtime s[28:29]\n"
    "s_waitcnt lgkmcnt(0)\n"
    "s_sub_u32 s54, s33, s28\n"
    "s_cmp_gt_i32 s54, 0\n"
    "s_cbranch_scc1 PACEI30_%=\n"
    "s_branch PACEBI30_%=\n"
    "PACEI32_%=:\n"
    "s_memrealtime s[28:29]\n"
    "s_waitcnt lgkmcnt(0)\n"
    "s_sub_u32 s54, s33, s28\n"
    "s_cmp_gt_i32 s54, 0\n"
    "s_cbranch_scc1 PACEI32_%=\n"
    "s_branch PACEBI32_%=\n"
    "PACEA0_%=:\n"
    "s_memrealtime s[28:29]\n"
    "s_waitcnt lgkmcnt(0)\n"
    "s_sub_u32 s54, s33, s28\n"
    "s_cmp_gt_i32 s54, 0\n"
    "s_cbranch_scc1 PACEA0_%=\n"
    "s_branch PACEBA0_%=\n"
    "PACEA2_%=:\n"
    "s_memrealtime s[28:29]\n"
    "s_waitcnt lgkmcnt(0)\n"
    "s_sub_u32 s54, s33, s28\n"
    "s_cmp_gt_i32 s54, 0\n"
    "s_cbranch_scc1 PACEA2_%=\n"
    "s_branch PACEBA2_%=\n"
    "PACEB0_%=:\n"
    "s_memrealtime s[28:29]\n"
    "s_waitcnt lgkmcnt(0)\n"
    "s_sub_u32 s54, s33, s28\n"
    "s_cmp_gt_i32 s54, 0\n"
    "s_cbranch_scc1 PACEB0_%=\n"
    "s_branch PACEBB0_%=\n"
    "PACEB2_%=:\n"
    "s_memrealtime s[28:29]\n"
    "s_waitcnt lgkmcnt(0)\n"
    "s_sub_u32 s54, s33, s28\n"
    "s_cmp_gt_i32 s54, 0\n"
    "s_cbranch_scc1 PACEB2_%=\n"
    "s_branch PACEBB2_%=\n"
    "PACEL0_%=:\n"
    "s_memrealtime s[28:29]\n"
    "s_waitcnt lgkmcnt(0)\n"
    "s_sub_u32 s54, s33, s28\n"
    "s_cmp_gt_i32 s54, 0\n"
    "s_cbranch_scc1 PACEL0_%=\n"
    "s_branch PACEBL0_%=\n"
    "PACEL2_%=:\n"
    "s_memrealtime s[28:29]\n"
    "s_waitcnt lgkmcnt(0)\n"
    "s_sub_u32 s54, s33, s28\n"
    "s_cmp_gt_i32 s54, 0\n"
    "s_cbranch_scc1 PACEL2_%=\n"
    "s_branch PACEBL2_%=\n"
    "SLOWI2_%=:\n"
    "s_getpc_b64 s[82:83]\n"
    "s_branch SLOW_%=\n"
    "s_branch SLOWBACKI2_%=\n"
    "SLOWI3_%=:\n"
    "s_getpc_b64 s[82:83]\n"
    "s_branch SLOW_%=\n"
    "s_branch SLOWBACKI3_%=\n"
    "SLOWA_%=:\n"
    "s_getpc_b64 s[82:83]\n"
    "s_branch SLOW_%=\n"
    "s_branch SLOWBACKA_%=\n"
    "SLOWB_%=:\n"
    "s_getpc_b64 s[82:83]\n"
    "s_branch SLOW_%=\n"
    "s_branch SLOWBACKB_%=\n"
    "SLOW_%=:\n"
    "s_mov_b32 s36, 1\n"
    "s_memrealtime s[84:85]\n"
    "SLOWLOOP_%=:\n"
    "s_mov_b32 exec_lo, s70\n"
    "s_mov_b32 exec_hi, 0\n"
    "global_load_dwordx4 v[180:183], v139, s[74:75] sc0 sc1\n"
    "s_mov_b64 exec, -1\n"
    "global_load_dword v184, v148, s[74:75] offset:2048 sc0 sc1\n"
    "s_waitcnt vmcnt(0)\n"
    "v_min_u32 v180, v180, v181\n"
    "v_min3_u32 v180, v180, v182, v183\n"
    "v_cmp_le_u32 vcc, s68, v180\n"
    "v_readfirstlane_b32 s55, v184\n"
    "s_nop 3\n"
    "s_and_b32 s54, vcc_lo, s70\n"
    "s_or_b32 s69, s69, s54\n"
    "s_andn2_b32 s70, s70, s54\n"
    "s_bitcmp1_b32 s69, s72\n"
    "s_cbranch_scc1 SLOWOK_%=\n"
    "s_cmp_eq_u32 s55, s68\n"
    "s_cbranch_scc1 FAIL_%=\n"
    "s_memrealtime s[86:87]\n"
    "s_waitcnt lgkmcnt(0)\n"
    "s_sub_u32 s86, s86, s84\n"
    "s_subb_u32 s87, s87, s85\n"
    "s_cmp_lg_u32 s87, 0\n"
    "s_cbranch_scc1 FAIL_%=\n"
    "s_cmp_gt_u32 s86, 50000000\n"
    "s_cbranch_scc1 FAIL_%=\n"
    "s_sleep 4\n"
    "s_branch SLOWLOOP_%=\n"
    "FAIL_%=:\n"
    "s_mov_b64 exec, 1\n"
    "global_store_dword v148, v152, s[74:75] offset:2048 sc0 sc1\n"
    "s_mov_b64 exec, -1\n"
    "s_waitcnt vmcnt(0)\n"
    "s_mov_b32 s69, -1\n"
    "s_mov_b32 s70, 0\n"
    "SLOWOK_%=:\n"
    "s_add_u32 s82, s82, 4\n"
    "s_addc_u32 s83, s83, 0\n"
    "s_setpc_b64 s[82:83]\n"
    "END_%=:\n"

    : "={a[0:15]}"(c[0]), "={a[16:31]}"(c[1]), "={a[32:47]}"(c[2]), "={a[48:63]}"(c[3]),
      "={a[64:79]}"(c[4]), "={a[80:95]}"(c[5]), "={a[96:111]}"(c[6]), "={a[112:127]}"(c[7]),
      "={a[128:143]}"(c[8]), "={a[144:159]}"(c[9]), "={a[160:175]}"(c[10]), "={a[176:191]}"(c[11]),
      "={a[192:207]}"(c[12]), "={a[208:223]}"(c[13]), "={a[224:239]}"(c[14]), "={a[240:255]}"(c[15]),
      "+{s[24:39]}"(sd), "+{s[40:55]}"(sa), "+{s[56:71]}"(sb), "+{s[72:87]}"(sc), "={s[88:95]}"(st), "+{v[128:143]}"(va), "={v180}"(fail), "={v188}"(prof), "+{s[96:97]}"(so)
    : "{v[198:199]}"(ve), "{v[144:151]}"(vb), "{v[154:157]}"(vc), "{v[190:197]}"(vd)
    : "memory", "scc", "vcc",
#define V10(x) "v" #x "0", "v" #x "1", "v" #x "2", "v" #x "3", "v" #x "4", "v" #x "5", "v" #x "6", "v" #x "7", "v" #x "8", "v" #x "9"
      "v0","v1","v2","v3","v4","v5","v6","v7","v8","v9", V10(1), V10(2), V10(3), V10(4), V10(5), V10(6), V10(7), V10(8), V10(9), V10(10), V10(11),
      "v120","v121","v122","v123","v124","v125","v126","v127", "v152", V10(16), V10(17), "v181", "v182", "v183", "v184", "v185", "v186", "v187", V10(20), V10(21), V10(22), V10(23), V10(24), "v250", "v251", "v252", "v253", "v254", "v255");
}

extern "C" void kernel_launch(int rank, int nranks, ncclComm_t comm, hipStream_t stream,
                              void* const* d_in, const int* in_sizes, int n_in,
                              void* d_out, int out_size, const int* in_dtypes, int out_dtype,
                              void* const* d_out_all, void* const* d_ws_all, size_t ws_bytes) {
  if (nranks != 8 || (unsigned)rank >= 8u || n_in < 2 || ws_bytes < WS_NEED || in_sizes[0] != 4096 * 512 ||
      in_sizes[1] != 4096 * 32768 || out_size != 512 * 32768) {
    fprintf(stderr, "a2a_gemm: unexpected problem shape\n");
    return;
  }
  WsTab t;
  for (int j = 0; j < 8; j++) t.p[j] = (char*)d_ws_all[(rank - j) & 7];
  a2a_gemm<<<256, 256, 0, stream>>>((const u16*)d_in[0], (const u16*)d_in[1], (u16*)d_out, t, rank);
}
```
